```python
import numpy as np

kernel_name = "gpu_gemm_rs_m4096_k4096_n4096_bf16_none"


ABSMAX_RTOL = 0.008

COMPUTE_DTYPE = "bfloat16"
EPILOGUE = "none"
IN_QUANT = None
OUT_QUANT = None
IN_DTYPES = {"x": "bfloat16", "w_mat": "bfloat16"}
WORLD_SIZE = 8

def _to_bf16(x):
    u = np.ascontiguousarray(x, dtype=np.float32).view(np.uint32)
    return ((u + 0x7FFF + ((u >> 16) & 1)) & 0xFFFF0000).view(np.float32)

def setup_inputs():

    x = _to_bf16(np.random.randn(4096, 4096).astype(np.float32))
    w_mat = _to_bf16((np.random.randn(4096, 4096) / np.sqrt(4096)).astype(np.float32))
    return {"x": x, "w_mat": w_mat}

def _epilogue(y):
    return y

def reference(x, w_mat):
    y = (x @ w_mat).astype(np.float32)
    return _epilogue(y)
```

```cpp
#include <hip/hip_runtime.h>
#include <rccl/rccl.h>
#include <cstdio>

namespace {

typedef unsigned u32;
typedef unsigned long long u64;
typedef __attribute__((ext_vector_type(4))) unsigned u32x4;
typedef __attribute__((ext_vector_type(4))) float f32x4;
typedef __attribute__((ext_vector_type(2))) float f32x2;
typedef __attribute__((ext_vector_type(2))) __bf16 bf16x2;
typedef __attribute__((ext_vector_type(8))) __bf16 bf16x8;
typedef __attribute__((ext_vector_type(4))) short s16x4;
typedef __attribute__((ext_vector_type(8))) short s16x8;
#define LAS __attribute__((address_space(3)))
#define GAS __attribute__((address_space(1)))
typedef GAS u32 gu32;

constexpr int NR = 8;
constexpr int NWG = 256;
constexpr int NT = 512;
constexpr u32 MiB = 1u << 20;
constexpr u32 CHUNK = 65536;
constexpr u32 IN_BYTES = 4 * MiB;

constexpr u32 OFF_CNT = 0;
constexpr u32 OFF_TMO = 4096;
constexpr u32 OFF_GATE = 8192;
constexpr u32 OFF_IFLAG = 16384;
constexpr u32 OFF_YFLAG = 32768;
constexpr u32 OFF_STAMP = 65536;
constexpr u32 OFF_STAGE = 1 * MiB;
constexpr u32 YS_OFF = 12 * MiB;
constexpr u32 PAR_BYTES = 18 * MiB;
constexpr u32 WS_NEED = OFF_STAGE + 2 * PAR_BYTES;

constexpr u64 DEADLINE = 40000000ull;

constexpr u32 STAGE_B = 32768;
constexpr u32 LDS_CTL = 4 * STAGE_B;
constexpr u32 LDS_BYTES = LDS_CTL + 512;

#define XT(i) (i)
#define WT(i) (16 + (i))
constexpr unsigned char SEQ[32] = {
    XT(0), WT(0), WT(1), XT(1), WT(2), XT(2), XT(3), WT(3),
    XT(4), WT(4), XT(6), WT(5), XT(8), WT(6), XT(10), WT(7),
    XT(5), WT(8), XT(7), WT(9), XT(9), WT(10), XT(11), WT(11),
    WT(12), WT(13), WT(14), WT(15), XT(12), XT(13), XT(14), XT(15)};
constexpr u64 seq_nibbles(int h) { u64 r = 0; for (int t = 0; t < 16; ++t) r |= (u64)(SEQ[2 * t + h] & 15) << (4 * t); return r; }
constexpr u32 seq_types(int h) { u32 r = 0; for (int t = 0; t < 16; ++t) r |= (u32)(SEQ[2 * t + h] >> 4) << t; return r; }
#define GATE_T 16
#define YPUSH_DEPTH 1
#define DELAY_GO 600
#define DELAY_P 400

struct Params {
  const char* x;
  const char* w;
  char* out;
  char* ws[NR];
  int rank;
};

#define RSRC(p, n) __builtin_amdgcn_make_buffer_rsrc((void*)(p), 0, (n), 0x00020000)
#define WAIT_VM(n) asm volatile("s_waitcnt vmcnt(%0)" ::"n"(n) : "memory")
#define BARRIER() do { asm volatile("s_waitcnt lgkmcnt(0)" ::: "memory"); __builtin_amdgcn_s_barrier(); asm volatile("" ::: "memory"); } while (0)
constexpr int SYS = 17;

__device__ __forceinline__ u32 rfl(u32 v) { return __builtin_amdgcn_readfirstlane(v); }

__device__ __forceinline__ u32 row0_of(u32 q, u32 rt) {
  const bool g = rt - 4u < 8u;
  const u32 d = g ? ((rt - 4u) | 1u) : (rt < 4u ? 4u : 0u);
  const u32 t = g ? (d & 2u) + (rt & 1u) : (rt & 3u);
  return (q ^ d) * 512u + t * 128u;
}

struct Ctx {
  LAS char* lds;
  char* wsl;
  u32 call, lane, wave;
};
#define STAMP(i)

__device__ __forceinline__ bool wait_words(const Ctx& c, u32 off0, u32 n0, u32 off1, u32 n1, u32 stride, int slot) {
  LAS u32* res = (LAS u32*)(c.lds + LDS_CTL + 256) + slot;
  if (c.wave == 0) {
    gu32* tmo = (gu32*)(c.wsl + OFF_TMO);
    gu32* p0 = c.lane < n0 ? (gu32*)(c.wsl + off0 + c.lane * stride) : tmo;
    gu32* p1 = c.lane < n1 ? (gu32*)(c.wsl + off1 + c.lane * stride) : tmo;
    const u64 t0 = __builtin_amdgcn_s_memrealtime();
    u32 bad = 0;
    for (;;) {
      const u32 v0 = __hip_atomic_load(p0, __ATOMIC_RELAXED, __HIP_MEMORY_SCOPE_SYSTEM);
      const u32 v1 = __hip_atomic_load(p1, __ATOMIC_RELAXED, __HIP_MEMORY_SCOPE_SYSTEM);
      if (__builtin_amdgcn_readlane(v1, 63) == c.call) { bad = 1; break; }
      const bool ok = (c.lane >= n0 || v0 >= c.call) && (c.lane >= n1 || v1 >= c.call);
      if (__all(ok)) break;
      if (__builtin_amdgcn_s_memrealtime() - t0 > DEADLINE) {
        if (c.lane == 0) __hip_atomic_store(tmo, c.call, __ATOMIC_RELAXED, __HIP_MEMORY_SCOPE_SYSTEM);
        bad = 1;
        break;
      }
      __builtin_amdgcn_s_sleep(2);
    }
    if (c.lane == 0) *res = bad;
  }
  BARRIER();
  return rfl(*res) != 0;
}

__device__ __forceinline__ bool last_wave(const Ctx& c, int idx) {
  LAS u32* cnt = (LAS u32*)(c.lds + LDS_CTL) + idx;
  u32 old = 0;
  if (c.lane == 0) old = __hip_atomic_fetch_add(cnt, 1u, __ATOMIC_RELAXED, __HIP_MEMORY_SCOPE_WORKGROUP);
  return rfl(old) == 7u;
}

__device__ __forceinline__ void send_inputs(Ctx& c, const Params& p, u32 d, u32 sidx, u32 cnt_v) {
  const u32 q = ((u32)p.rank ^ d) & (NR - 1);
  char* dst = p.ws[q];
  const u32 slot = d < 4 ? d - 1 : d - 2;
  const u32 half = c.wave >> 2;
  const u32 iq = (sidx & 1) * 8 + (c.wave & 3) * 2;
  const u32 k0 = (q >> 2) * 256 + (sidx >> 1) * 64;
  const u32 xr = iq * 8 + (c.lane >> 3);
  const u32 vx0 = xr * 1024 + k0 * 2 + (((c.lane & 7) ^ ((xr >> 1) & 7)) << 4);
  const u32 vx1 = (xr + 8) * 1024 + k0 * 2 + (((c.lane & 7) ^ (((xr + 8) >> 1) & 7)) << 4);
  const u32 wk = iq * 4 + (c.lane >> 4);
  const u32 vw0 = (k0 + wk) * 8192 + (q & 1) * 4096 + (((c.lane & 15) ^ (((wk & 3) << 2) | ((wk >> 3) & 1))) << 4);
  const u32 vw1 = vw0 + 4 * 8192;
  const u32 vd = sidx * 8192 + (c.wave & 3) * 2048 + c.lane * 16;
  const auto rd = RSRC(dst, WS_NEED);
  const u64 nib = half ? seq_nibbles(1) : seq_nibbles(0);
  const u32 typ = half ? seq_types(1) : seq_types(0);
  u32x4 v0, v1;
  auto chunk_id = [&](int t) { return (u32)((nib >> (4 * t)) & 15) + ((typ >> t) & 1) * 16; };
  auto load = [&](int t) {
    const u32 id = chunk_id(t);
    const bool isw = id >= 16;
    const auto rs = RSRC(isw ? p.w : p.x, IN_BYTES);
    const u32 so = rfl(isw ? (id - 16) * 256 : row0_of(q, id) * 1024);
    v0 = __builtin_amdgcn_raw_buffer_load_b128(rs, isw ? vw0 : vx0, so, 0);
    v1 = __builtin_amdgcn_raw_buffer_load_b128(rs, isw ? vw1 : vx1, so, 0);
  };
  load(0);
  c.call = rfl(cnt_v) + 1;
  const u32 sd = OFF_STAGE + (c.call & 1) * PAR_BYTES + slot * CHUNK;
  gu32* iflag = (gu32*)(dst + OFF_IFLAG) + slot * 8 + sidx;
  gu32* gate = (gu32*)(c.wsl + OFF_GATE + d * 128) + sidx;
  const bool leader = (threadIdx.x & 255) == 0;
  auto flag = [&](int t) {
    if (leader) {
      __hip_atomic_store(iflag + chunk_id(t) * 64, c.call, __ATOMIC_RELAXED, __HIP_MEMORY_SCOPE_SYSTEM);
      if (t == GATE_T - 1 && half == 0) __hip_atomic_store(gate, c.call, __ATOMIC_RELAXED, __HIP_MEMORY_SCOPE_SYSTEM);
    }
  };
  if (half) BARRIER();
#pragma unroll 1
  for (int t = 0; t < 16; ++t) {
    WAIT_VM(0);
    BARRIER();
    if (t > 0) flag(t - 1);
    const u32 so = sd + chunk_id(t) * (6 * CHUNK);
    __builtin_amdgcn_raw_buffer_store_b128(v0, rd, vd + so, 0, SYS);
    __builtin_amdgcn_raw_buffer_store_b128(v1, rd, vd + so + 1024, 0, SYS);
    load(t < 15 ? t + 1 : t);
    BARRIER();
  }
  WAIT_VM(0);
  BARRIER();
  flag(15);
  if (!half) BARRIER();
}

struct Frag { bf16x8 x[4]; bf16x8 w[2]; };

template <int NS, bool STAGED>
__device__ __forceinline__ void kloop(const Ctx& c, f32x4 (&acc)[4][2], __amdgpu_buffer_rsrc_t ra, __amdgpu_buffer_rsrc_t rb,
                                      u32 va0, u32 va1, u32 vb0, u32 vb1, u32 sa, u32 sb, u32 stepa, u32 stepb) {
  const u32 lbase = (u32)(size_t)c.lds;
  const u32 m0a = lbase + c.wave * 2048;
  const u32 wm = c.wave >> 2, wn = c.wave & 3, fr = c.lane & 15, fq = c.lane >> 4;
  const u32 xa0 = (wm * 64 + fr) * 128 + ((fq ^ (fr >> 1)) << 4);
  const u32 tq = (c.lane >> 2) & 3, tp = c.lane & 3;
  const u32 wa0 = 16384 + (fq * 8 + tq) * 256 + (((wn * 4 + tp) ^ ((tq << 2) | (fq & 1))) << 4) + ((tp & 1) << 3);

  auto issue = [&](int s) {
    const u32 st = m0a + (u32)(s & 3) * STAGE_B;
    const u32 oa = sa + (u32)s * stepa, ob = sb + (u32)s * stepb;
    if constexpr (STAGED) {
      asm volatile(
          "s_mov_b32 m0, %0\n\ts_nop 3\n\t"
          "buffer_load_dwordx4 %2, %6, %8 offen sc0 sc1 lds\n\t"
          "buffer_load_dwordx4 %3, %6, %8 offen offset:1024 sc0 sc1 lds\n\t"
          "s_mov_b32 m0, %1\n\ts_nop 0\n\t"
          "buffer_load_dwordx4 %4, %7, %9 offen sc0 sc1 lds\n\t"
          "buffer_load_dwordx4 %5, %7, %9 offen offset:1024 sc0 sc1 lds"
          ::"s"(st), "s"(st + 16384), "v"(va0), "v"(va1), "v"(vb0), "v"(vb1), "s"(ra), "s"(rb), "s"(oa), "s"(ob) : "memory");
    } else {
      asm volatile(
          "s_mov_b32 m0, %0\n\ts_nop 3\n\t"
          "buffer_load_dwordx4 %2, %6, %8 offen lds\n\t"
          "buffer_load_dwordx4 %3, %6, %8 offen offset:1024 lds\n\t"
          "s_mov_b32 m0, %1\n\ts_nop 0\n\t"
          "buffer_load_dwordx4 %4, %7, %9 offen lds\n\t"
          "buffer_load_dwordx4 %5, %7, %9 offen offset:1024 lds"
          ::"s"(st), "s"(st + 16384), "v"(va0), "v"(va1), "v"(vb0), "v"(vb1), "s"(ra), "s"(rb), "s"(oa), "s"(ob) : "memory");
    }
  };
  auto read = [&](Frag& f, int s, int kh) {
    LAS char* st = c.lds + (u32)(s & 3) * STAGE_B;
#pragma unroll
    for (int n = 0; n < 2; ++n) {
      LAS char* pw = st + (wa0 ^ (u32)(n << 3)) + kh * 8192;
      const s16x4 lo = __builtin_amdgcn_ds_read_tr16_b64_v4i16((LAS s16x4*)pw);
      const s16x4 hi = __builtin_amdgcn_ds_read_tr16_b64_v4i16((LAS s16x4*)(pw + 1024));
      f.w[n] = __builtin_bit_cast(bf16x8, __builtin_shufflevector(lo, hi, 0, 1, 2, 3, 4, 5, 6, 7));
    }
#pragma unroll
    for (int m = 0; m < 4; ++m) f.x[m] = *(LAS bf16x8*)(st + (xa0 ^ (u32)(kh << 6)) + m * 2048);
  };
  auto mma = [&](const Frag& f) {
#pragma unroll
    for (int m = 0; m < 4; ++m)
#pragma unroll
      for (int n = 0; n < 2; ++n) acc[m][n] = __builtin_amdgcn_mfma_f32_16x16x32_bf16(f.w[n], f.x[m], acc[m][n], 0, 0, 0);
  };

  Frag f0, f1;
#pragma unroll
  for (int m = 0; m < 4; ++m) f1.x[m] = (bf16x8)(__bf16)0.0f;
  f1.w[0] = f1.w[1] = (bf16x8)(__bf16)0.0f;
  issue(0); issue(1); issue(2);
  auto step = [&](int s, bool more) {
    BARRIER();
    if (more) issue(s + 3);
    read(f0, s, 0);
    __builtin_amdgcn_sched_barrier(0);
    mma(f1);
    __builtin_amdgcn_sched_barrier(0);
    read(f1, s, 1);
    __builtin_amdgcn_sched_barrier(0);
    mma(f0);
    __builtin_amdgcn_sched_barrier(0);
  };
#pragma unroll 1
  for (int s = 0; s < NS - 3; ++s) { WAIT_VM(8); step(s, true); }
  WAIT_VM(8); step(NS - 3, false);
  WAIT_VM(4); step(NS - 2, false);
  WAIT_VM(0); step(NS - 1, false);
  mma(f1);
  BARRIER();
}

__device__ __forceinline__ u32 pack2(float a, float b) {
  return __builtin_bit_cast(u32, __builtin_convertvector((f32x2){a, b}, bf16x2));
}
__device__ __forceinline__ float bf_lo(u32 v) { return __builtin_bit_cast(float, v << 16); }
__device__ __forceinline__ float bf_hi(u32 v) { return __builtin_bit_cast(float, v & 0xffff0000u); }

__global__ void __launch_bounds__(NT, 2) gemm_rs(Params p) {
  __shared__ __attribute__((aligned(1024))) char smem[LDS_BYTES];
  Ctx c;
  c.lds = (LAS char*)smem;
  c.lane = threadIdx.x & 63;
  c.wave = rfl(threadIdx.x >> 6);
  const u32 rank = (u32)p.rank & (NR - 1);
  c.wsl = p.ws[rank];
  STAMP(0);
  gu32* cnt = (gu32*)(c.wsl + OFF_CNT) + blockIdx.x;
  const u32 cnt_v = __hip_atomic_load(cnt, __ATOMIC_RELAXED, __HIP_MEMORY_SCOPE_AGENT);
  if (threadIdx.x < 128) ((LAS u32*)(c.lds + LDS_CTL))[threadIdx.x] = 0;

  const u32 grp = blockIdx.x & 7, u = blockIdx.x >> 3;
  const bool isG = grp & 1, isO = (grp & 3) == 2;
  u32 j, ct;
  if (isG) { j = u & 1; ct = u >> 1; }
  else { j = u & 3; ct = (u >> 2) * 2 + (grp >> 2); }
  const u32 rt0 = isG ? 4 + (grp - 1) : (isO ? 12 : 0);
  const bool sender = isG ? u >= 24 : (isO && j == 3);
  const int ntiles = isG ? (sender ? 0 : ((u < 16 && j == 0) ? 2 : 1)) : 1;

  if (sender) {
    send_inputs(c, p, grp, isG ? u & 7 : u >> 2, cnt_v);
  } else {
    const u64 t0 = __builtin_amdgcn_s_memrealtime();
    const u64 hold = (isG || isO) ? DELAY_GO : (ct >= 4 ? DELAY_P : 0);
    while (__builtin_amdgcn_s_memrealtime() - t0 < hold) __builtin_amdgcn_s_sleep(4);
    c.call = rfl(cnt_v) + 1;
    BARRIER();
  }
  const u32 par_off = OFF_STAGE + (c.call & 1) * PAR_BYTES;
  if (threadIdx.x == 0) __hip_atomic_store(cnt, c.call, __ATOMIC_RELAXED, __HIP_MEMORY_SCOPE_AGENT);
  STAMP(1);

  const auto rws = RSRC(c.wsl, WS_NEED);
  const u32 NANS = 0x7fc07fc0u;
  bool bad = false;
  f32x4 lo[4], hi[4];
  u32x4 y[4], y2[4];
  u32 tj = j, tct = ct;
#pragma unroll 1
  for (int it = 0; it < ntiles; ++it) {
    if (it == 1) {
      tj = (u >> 1) & 1; tct = 12 + (u >> 2);
#pragma unroll
      for (int m = 0; m < 4; ++m) y2[m] = y[m];
    }
    const u32 rt = rt0 + tj;
    f32x4 acc[4][2];
#pragma unroll
    for (int m = 0; m < 4; ++m) acc[m][0] = acc[m][1] = (f32x4){0.f, 0.f, 0.f, 0.f};
    {
      const u32 xr = c.wave * 16 + (c.lane >> 3);
      const u32 va0 = xr * 1024 + (((c.lane & 7) ^ ((xr >> 1) & 7)) << 4);
      const u32 va1 = (xr + 8) * 1024 + (((c.lane & 7) ^ (((xr + 8) >> 1) & 7)) << 4) - 1024;
      const u32 wk = c.wave * 8 + (c.lane >> 4);
      const u32 vb0 = wk * 8192 + (((c.lane & 15) ^ (((wk & 3) << 2) | ((wk >> 3) & 1))) << 4);
      const u32 vb1 = vb0 + 4 * 8192 - 1024;
      kloop<8, false>(c, acc, RSRC(p.x, IN_BYTES), RSRC(p.w, IN_BYTES), va0, va1, vb0, vb1,
                      row0_of(rank, rt) * 1024, (rank & 1) * 4096 + tct * 256, 128, 64 * 8192);
    }
    if (it == 0) { STAMP(2); }
    bad |= wait_words(c, OFF_IFLAG + rt * 256, 48, OFF_IFLAG + (16 + tct) * 256, 48, 4, it);
    if (it == 0) { STAMP(3); }
    {
      const u32 v = c.wave * 2048 + c.lane * 16;
      kloop<24, true>(c, acc, rws, rws, v, v, v, v, par_off + rt * 6 * CHUNK, par_off + (16 + tct) * 6 * CHUNK, 16384, 16384);
    }
    if (it == 0) { STAMP(4); }
    const bool odd = (c.lane >> 4) & 1;
#pragma unroll
    for (int m = 0; m < 4; ++m) {
      lo[m] = odd ? acc[m][1] : acc[m][0]; hi[m] = odd ? acc[m][0] : acc[m][1];
      y[m].x = pack2(lo[m][0], lo[m][1]); y[m].y = pack2(lo[m][2], lo[m][3]);
      y[m].z = pack2(hi[m][0], hi[m][1]); y[m].w = pack2(hi[m][2], hi[m][3]);
      if (bad) y[m] = (u32x4){NANS, NANS, NANS, NANS};
    }
  }
  STAMP(5);

  const auto rout = RSRC(p.out, IN_BYTES);
  const u32 vy = c.wave * 4096 + c.lane * 16;

  if (isO) {
    const u32 yt = j * 16 + ct;
    bad |= wait_words(c, OFF_YFLAG + yt * 128, 1, 0, 0, 4, 2);
    STAMP(6);
#pragma unroll
    for (int m = 0; m < 4; ++m) y[m] = __builtin_amdgcn_raw_buffer_load_b128(rws, vy + m * 1024, par_off + YS_OFF + yt * 32768, SYS);
    const u32 vo = (j * 128 + (c.wave >> 2) * 64 + (c.lane & 15)) * 8192 + (rank & 1) * 4096 + ct * 256 + (c.wave & 3) * 64 + (c.lane >> 4) * 16;
#pragma unroll
    for (int m = 0; m < 4; ++m) {
      u32x4 o;
      o.x = pack2(lo[m][0] + bf_lo(y[m].x), lo[m][1] + bf_hi(y[m].x));
      o.y = pack2(lo[m][2] + bf_lo(y[m].y), lo[m][3] + bf_hi(y[m].y));
      o.z = pack2(hi[m][0] + bf_lo(y[m].z), hi[m][1] + bf_hi(y[m].z));
      o.w = pack2(hi[m][2] + bf_lo(y[m].w), hi[m][3] + bf_hi(y[m].w));
      if (bad) o = (u32x4){NANS, NANS, NANS, NANS};
      __builtin_amdgcn_raw_buffer_store_b128(o, rout, vo + m * 16 * 8192, 0, 0);
    }
  } else {
    if (ntiles > 0) {
      const u32 d = isG ? grp : 4u;
      const u32 ytb = isG ? 64 + (grp >> 1) * 32 : 0;
      const u32 yt = rfl(ytb + tj * 16 + tct), yt2 = ytb + j * 16 + ct;
      if (isG) (void)wait_words(c, OFF_GATE + grp * 128, 8, 0, 0, 4, 3);
      STAMP(6);
      char* dst = p.ws[(rank ^ d) & (NR - 1)];
      const auto rd = RSRC(dst, WS_NEED);
#pragma unroll
      for (int m = 0; m < 4; ++m) {
        __builtin_amdgcn_raw_buffer_store_b128(y[m], rd, vy + m * 1024 + par_off + YS_OFF + yt * 32768, 0, SYS);
        if (ntiles == 2) __builtin_amdgcn_raw_buffer_store_b128(y2[m], rd, vy + m * 1024 + par_off + YS_OFF + yt2 * 32768, 0, SYS);
        if (m < 3) WAIT_VM(YPUSH_DEPTH - 1);
      }
      WAIT_VM(0);
      if (last_wave(c, 0) && c.lane == 0) {
        __hip_atomic_store((gu32*)(dst + OFF_YFLAG + yt * 128), c.call, __ATOMIC_RELAXED, __HIP_MEMORY_SCOPE_SYSTEM);
        if (ntiles == 2) __hip_atomic_store((gu32*)(dst + OFF_YFLAG + yt2 * 128), c.call, __ATOMIC_RELAXED, __HIP_MEMORY_SCOPE_SYSTEM);
      }
    }
    STAMP(7);
    if (isG) {
      const u32 ya = 64 + ((grp >> 1) & 1) * 32 + j * 16 + ct, yb = ya + 64;
      bad |= wait_words(c, OFF_YFLAG + ya * 128, 2, 0, 0, 64 * 128, 2);
      STAMP(8);
      const u32 pw = (grp >> 2) * 4 + (c.wave >> 1), m0 = (c.wave & 1) * 2;
      const u32 vr = pw * 4096 + m0 * 1024 + c.lane * 16;
      u32x4 ra[2], rb[2];
#pragma unroll
      for (int m = 0; m < 2; ++m) {
        ra[m] = __builtin_amdgcn_raw_buffer_load_b128(rws, vr + m * 1024, par_off + YS_OFF + ya * 32768, SYS);
        rb[m] = __builtin_amdgcn_raw_buffer_load_b128(rws, vr + m * 1024, par_off + YS_OFF + yb * 32768, SYS);
      }
      const u32 vo = (((grp >> 1) & 1) * 256 + j * 128 + (pw >> 2) * 64 + m0 * 16 + (c.lane & 15)) * 8192 +
                     ((rank & 1) ^ 1) * 4096 + ct * 256 + (pw & 3) * 64 + (c.lane >> 4) * 16;
#pragma unroll
      for (int m = 0; m < 2; ++m) {
        u32x4 o;
        o.x = pack2(bf_lo(ra[m].x) + bf_lo(rb[m].x), bf_hi(ra[m].x) + bf_hi(rb[m].x));
        o.y = pack2(bf_lo(ra[m].y) + bf_lo(rb[m].y), bf_hi(ra[m].y) + bf_hi(rb[m].y));
        o.z = pack2(bf_lo(ra[m].z) + bf_lo(rb[m].z), bf_hi(ra[m].z) + bf_hi(rb[m].z));
        o.w = pack2(bf_lo(ra[m].w) + bf_lo(rb[m].w), bf_hi(ra[m].w) + bf_hi(rb[m].w));
        if (bad) o = (u32x4){NANS, NANS, NANS, NANS};
        __builtin_amdgcn_raw_buffer_store_b128(o, rout, vo + m * 16 * 8192, 0, 0);
      }
    }
  }
}

}

extern "C" void kernel_launch(int rank, int nranks, ncclComm_t comm, hipStream_t stream,
                              void* const* d_in, const int* in_sizes, int n_in,
                              void* d_out, int out_size,
                              const int* in_dtypes, int out_dtype,
                              void* const* d_out_all,
                              void* const* d_ws_all, size_t ws_bytes) {
  (void)comm; (void)in_dtypes; (void)out_dtype; (void)d_out_all;
  if (nranks != NR || (unsigned)rank >= (unsigned)NR || n_in != 2 || ws_bytes < WS_NEED ||
      in_sizes[0] != 4096 * 512 || in_sizes[1] != 512 * 4096 || out_size != 512 * 4096) {
    fprintf(stderr, "gemm_rs: unexpected problem shape\n");
    return;
  }
  Params p;
  p.x = (const char*)d_in[0];
  p.w = (const char*)d_in[1];
  p.out = (char*)d_out;
  for (int i = 0; i < NR; ++i) p.ws[i] = (char*)d_ws_all[i];
  p.rank = rank;
  hipLaunchKernelGGL(gemm_rs, dim3(NWG), dim3(NT), 0, stream, p);
}
```

```cpp
#include <hip/hip_runtime.h>
#include <rccl/rccl.h>
#include <cstdio>

namespace {

typedef unsigned u32;
typedef unsigned long long u64;
typedef __attribute__((ext_vector_type(4))) unsigned u32x4;
typedef __attribute__((ext_vector_type(4))) float f32x4;
typedef __attribute__((ext_vector_type(2))) float f32x2;
typedef __attribute__((ext_vector_type(2))) __bf16 bf16x2;
typedef __attribute__((ext_vector_type(8))) __bf16 bf16x8;
typedef __attribute__((ext_vector_type(4))) short s16x4;
#define LAS __attribute__((address_space(3)))
#define GAS __attribute__((address_space(1)))
typedef GAS u32 gu32;

constexpr int NR = 8;
constexpr int NWG = 256;
constexpr int NT = 512;
constexpr u32 MiB = 1u << 20;
constexpr u32 CHUNK = 65536;
constexpr int NSEND = 8;
constexpr u32 PIECE = CHUNK / NSEND;
constexpr u32 IN_BYTES = 4 * MiB;

constexpr u32 OFF_CNT = 0;
constexpr u32 OFF_TMO = 4096;
constexpr u32 OFF_LYF = 8192;
constexpr u32 OFF_GATE = 12288;
constexpr u32 OFF_IFLAG = 16384;
constexpr u32 OFF_YFLAG = 32768;
constexpr u32 OFF_GFLAG = 40960;
constexpr u32 OFF_STAMP = 65536;
constexpr u32 OFF_STAGE = 1 * MiB;
constexpr u32 YS_OFF = 12 * MiB;
constexpr u32 PAR_BYTES = 18 * MiB;
constexpr u32 OFF_LY = OFF_STAGE + 2 * PAR_BYTES;
constexpr u32 WS_NEED = OFF_LY + 4 * MiB;

constexpr u64 DEADLINE = 40000000ull;

constexpr u32 STAGE_B = 32768;
constexpr u32 LDS_CTL = 4 * STAGE_B;
constexpr u32 LDS_BYTES = LDS_CTL + 2048;

enum { K_X = 0, K_W = 1, K_Y = 2, K_H = 3, K_R = 4 };
#define X(i) (K_X * 16 + (i))
#define W(i) (K_W * 16 + (i))
#define Y(i) (K_Y * 16 + (i))
#define H(i) (K_H * 16 + (i))
#define R(i) (K_R * 16 + (i))
#define NREL 6
#define SEQ_HEAD X(0), W(0), W(1), X(1), W(2), X(2), X(3), W(3), X(4), X(6), W(4), X(8), W(5), X(10), W(6), X(5), \
                 W(7), X(7), W(8), X(9), W(9), X(11)
constexpr int NFWD = 4;
constexpr int NPOS_G = 32 - NREL / 2 + NFWD, NPOS_O = 32 + NREL;
constexpr unsigned char SEQ_G[48] = {SEQ_HEAD,
    H(10), H(12), H(14),
    X(12), X(13), X(14), X(15), Y(0), Y(1), Y(2), Y(3)};
constexpr unsigned char SEQ_O[48] = {SEQ_HEAD,
    W(10), W(11), W(12), W(13), R(10), R(11), W(14), R(12), W(15), R(13), R(14), R(15),
    X(12), X(13), X(14), X(15)};
#undef X
#undef W
#undef Y
#undef H
#undef R
constexpr u64 seq_pack(const unsigned char* s, int from) { u64 r = 0; for (int t = 0; t < 8; ++t) r |= (u64)s[from + t] << (8 * t); return r; }

#define GATE_POS (NPOS_G - 3)
#define PACE 113
#define PACE_LEAD 40
#define DELAY_GO 600
#define DELAY_P 400

constexpr u32 L2P = 0x53764210u;
constexpr u32 X2G = 0x75314602u;

struct Params {
  const char* x;
  const char* w;
  char* out;
  char* wsl;
  char* ws[NR];
  int rank;
};

#define RSRC(p, n) __builtin_amdgcn_make_buffer_rsrc((void*)(p), 0, (n), 0x00020000)
#define WAIT_VM(n) asm volatile("s_waitcnt vmcnt(%0)" ::"n"(n) : "memory")
#define BARRIER() do { asm volatile("s_waitcnt lgkmcnt(0)" ::: "memory"); __builtin_amdgcn_s_barrier(); asm volatile("" ::: "memory"); } while (0)
#define SB() __builtin_amdgcn_sched_barrier(0)
constexpr int SYS = 17;

__device__ __forceinline__ u32 rfl(u32 v) { return __builtin_amdgcn_readfirstlane(v); }
__device__ __forceinline__ u32 rdl(u32 v, int lane) { return __builtin_amdgcn_readlane(v, lane); }
__device__ __forceinline__ void put_flag(char* addr, u32 v) { __hip_atomic_store((gu32*)addr, v, __ATOMIC_RELAXED, __HIP_MEMORY_SCOPE_SYSTEM); }
__device__ __forceinline__ u32 get_flag(const char* addr) { return __hip_atomic_load((gu32*)addr, __ATOMIC_RELAXED, __HIP_MEMORY_SCOPE_SYSTEM); }
__device__ __forceinline__ u32 lds_get(LAS u32* w) { return rfl(__hip_atomic_load(w, __ATOMIC_RELAXED, __HIP_MEMORY_SCOPE_WORKGROUP)); }
__device__ __forceinline__ void lds_inc(LAS u32* w, u32 lane) {
  if (lane == 0) __hip_atomic_fetch_add(w, 1u, __ATOMIC_RELAXED, __HIP_MEMORY_SCOPE_WORKGROUP);
}

__device__ __forceinline__ u32 row0_of(u32 q, u32 rt) {
  const bool g = rt - 4u < 8u;
  const u32 d = g ? ((rt - 4u) | 1u) : (rt < 4u ? 4u : 0u);
  const u32 t = g ? (d & 2u) + (rt & 1u) : (rt & 3u);
  return ((L2P >> ((q ^ d) * 4u)) & 7u) * 512u + t * 128u;
}
__device__ __forceinline__ u32 slot_of(u32 d) { return (d & 1u) ? d >> 1 : 4u + (d >> 2); }
__device__ __forceinline__ u32 gpos(u32 j, u32 ct) { return j * 16u + ct; }

struct Ctx {
  LAS char* lds;
  char* wsl;
  u32 call, lane, wave;
};
#define STAMP(i)

__device__ __forceinline__ u32 wait_words(const Ctx& c, u32 off0, u32 off1, u32 n, u64 need, u32 slot) {
  LAS u32* res = (LAS u32*)(c.lds + LDS_CTL + 256) + slot;
  if (c.wave == 0) {
    const char* p0 = c.wsl + (c.lane < n ? off0 + c.lane * 4 : OFF_TMO);
    const char* p1 = c.wsl + (c.lane < n ? off1 + c.lane * 4 : OFF_TMO);
    const u64 t0 = __builtin_amdgcn_s_memrealtime();
    u32 r;
    for (;;) {
      const u32 v0 = get_flag(p0), v1 = get_flag(p1);
      if (rdl(v1, 63) == c.call) { r = 3; break; }
      const u64 miss = __ballot(c.lane < n && (v0 < c.call || v1 < c.call));
      if ((miss & need) == 0) { r = miss ? 0 : 2; break; }
      if (__builtin_amdgcn_s_memrealtime() - t0 > DEADLINE) {
        if (c.lane == 0) put_flag(c.wsl + OFF_TMO, c.call);
        r = 3;
        break;
      }
      __builtin_amdgcn_s_sleep(2);
    }
    if (c.lane == 0) *res = r;
  }
  BARRIER();
  return rfl(*res);
}

__device__ __forceinline__ bool last_wave(const Ctx& c, int idx) {
  LAS u32* cnt = (LAS u32*)(c.lds + LDS_CTL) + idx;
  u32 old = 0;
  if (c.lane == 0) old = __hip_atomic_fetch_add(cnt, 1u, __ATOMIC_RELAXED, __HIP_MEMORY_SCOPE_WORKGROUP);
  return rfl(old) == 7u;
}

__device__ __forceinline__ void link_engine(Ctx& c, const Params& p, u32 d, u32 sidx, u32 cnt_v) {
  const u32 q = ((u32)p.rank ^ d) & (NR - 1);
  char* dst = p.ws[q];
  const bool odd = d & 1;
  const int npos = odd ? NPOS_G : NPOS_O;
  LAS u32* done = (LAS u32*)(c.lds + LDS_CTL + 512);
  LAS u32* full = (LAS u32*)(c.lds + LDS_CTL + 768);
  LAS u32* freed = (LAS u32*)(c.lds + LDS_CTL + 1024);
  LAS u32* dead = (LAS u32*)(c.lds + LDS_CTL + 320);
  const u32 l = c.lane;
  u64 tab = seq_pack(SEQ_G, 0);
  if (l >= 8) tab = seq_pack(SEQ_G, 8);
  if (l >= 16) tab = odd ? seq_pack(SEQ_G, 16) : seq_pack(SEQ_O, 16);
  if (l >= 24) tab = odd ? seq_pack(SEQ_G, 24) : seq_pack(SEQ_O, 24);
  if (l >= 32) tab = odd ? seq_pack(SEQ_G, 32) : seq_pack(SEQ_O, 32);
  if (l >= 40) tab = odd ? seq_pack(SEQ_G, 40) : seq_pack(SEQ_O, 40);
  const u32 ent = (u32)(tab >> (8 * (l & 7))) & 255u, kind = ent >> 4;
  const u32 ts = (sidx + l) & 7u;
  const bool isy = kind == K_Y, isr = kind == K_R, split = kind >= K_H;
  const u32 id = (ent & 15u) + (kind == K_H ? ts >> 2 : 0u);
  const u32 pc = split ? ((isr ? d : d << 1) & 4u) + (ts & 3u) : ts;
  const u32 dorg = isr ? (d >> 1) + (ts & 4u) : 0u;
  const u32 cid = (kind == K_X ? 0u : 16u) + id, yk = (d >> 1) * 16 + id;
  if (c.wave >= 4 && c.wave < 7) {
    const u32 st = pc >> 1, qa = q >> 2, qab = (qa ^ (q >> 1)) & 1u;
    const u32 kof = st < 2u ? qa * 128u + st * 64u : 128u + qab * 128u + st * 64u;
    const u32 t_src = isy ? OFF_LY + yk * CHUNK + pc * PIECE
                    : isr ? OFF_STAGE + (cid * 6 + slot_of(dorg)) * CHUNK + pc * PIECE
                    : kind == K_X ? row0_of(q, id) * 1024u + kof * 2u + (pc & 1u) * 65536u
                                  : (kof + (pc & 1u) * 32u) * 8192u + id * 256u;
    const u32 t_mx = kind == K_X ? ~0u : 0u, t_ml = (isy || isr) ? ~0u : 0u, t_mr = isr ? ~0u : 0u;
    const u32 t_dep = isy ? OFF_LYF + (yk * 2 + (pc >> 2)) * 4 : OFF_IFLAG + cid * 256 + (slot_of(dorg) * 8 + pc) * 4;
    u32 vx[4], vw[4];
#pragma unroll
    for (int i = 0; i < 4; ++i) {
      const u32 xr = i * 8 + (l >> 3), wk = i * 4 + (l >> 4);
      vx[i] = xr * 1024 + (((l & 7) ^ ((xr >> 1) & 7)) << 4) - i * 1024;
      vw[i] = wk * 8192 + (q & 1) * 4096 + (((l & 15) ^ (((wk & 3) << 2) | ((wk >> 3) & 1))) << 4) - i * 1024;
    }
    const u32 lbase = (u32)(size_t)c.lds;
    u32 par = 0;
    bool first = true;
#pragma unroll 1
    for (int pos = (int)c.wave - 4; pos < npos; pos += 3) {
      const u32 mx = rdl(t_mx, pos), ml = rdl(t_ml, pos), mw = ~(mx | ml);
      const u32 so = rdl(t_src, pos) + (rdl(t_mr, pos) & par), sstep = (mx & 32768u) | (mw & 131072u) | (ml & 4096u);
      const auto rs = RSRC(ml ? c.wsl : (mw ? p.w : p.x), ml ? WS_NEED : IN_BYTES);
      u32 vo[4];
#pragma unroll
      for (int i = 0; i < 4; ++i) vo[i] = (vx[i] & mx) | (vw[i] & mw) | ((l * 16) & ml);
      if (pos >= 8) while (lds_get(freed + (pos - 8)) < 4u) __builtin_amdgcn_s_sleep(1);
      if (ml) {
        const char* dep = c.wsl + rdl(t_dep, pos);
        const u64 t0 = __builtin_amdgcn_s_memrealtime();
        while (rfl(get_flag(dep)) < c.call) {
          if (lds_get(dead) || __builtin_amdgcn_s_memrealtime() - t0 > 2 * DEADLINE) {
            if (l == 0) {
              put_flag(c.wsl + OFF_TMO, c.call);
              __hip_atomic_store(dead, 1u, __ATOMIC_RELAXED, __HIP_MEMORY_SCOPE_WORKGROUP);
            }
            break;
          }
          __builtin_amdgcn_s_sleep(2);
        }
      }
#pragma unroll
      for (int g = 0; g < 2; ++g)
        asm volatile("s_mov_b32 m0, %0\n\ts_nop 3\n\t"
                     "buffer_load_dwordx4 %1, %5, %6 offen sc0 sc1 lds\n\t"
                     "buffer_load_dwordx4 %2, %5, %6 offen offset:1024 sc0 sc1 lds\n\t"
                     "buffer_load_dwordx4 %3, %5, %6 offen offset:2048 sc0 sc1 lds\n\t"
                     "buffer_load_dwordx4 %4, %5, %6 offen offset:3072 sc0 sc1 lds"
                     ::"s"(lbase + (u32)(pos & 7) * PIECE + g * 4096u), "v"(vo[0]), "v"(vo[1]), "v"(vo[2]), "v"(vo[3]), "s"(rs),
                       "s"(so + g * sstep) : "memory");
      if (first) { c.call = rfl(cnt_v) + 1; par = (c.call & 1) * PAR_BYTES; BARRIER(); first = false; }
      WAIT_VM(0);
      lds_inc(full + pos, l);
    }
  } else {
    c.call = rfl(cnt_v) + 1;
    BARRIER();
    const u32 oslot = slot_of(d ^ dorg);
    if (c.wave < 4) {
      const u32 tw = (c.wave + l + (l >> 2)) & 3u;
      const u32 t_dst = OFF_STAGE + (c.call & 1) * PAR_BYTES + pc * PIECE + tw * 2048u +
                        (isy ? YS_OFF + 2 * MiB + yk * CHUNK : (cid * 6 + oslot) * CHUNK);
      const u32 t_lds = (l & 7u) * PIECE + tw * 2048u;
      const auto rd = RSRC(dst, WS_NEED);
      int sig = 0;
      while (lds_get(full) == 0u) __builtin_amdgcn_s_sleep(1);
      STAMP(7);
      const u32 t0 = (u32)__builtin_amdgcn_s_memrealtime() - PACE_LEAD;
      u32 due = (sidx * 4 + c.wave) * PACE / 32;
#pragma unroll 1
      for (int pos = 0; pos < npos; ++pos) {
        if (lds_get(full + pos) == 0u) {
          WAIT_VM(0);
          for (; sig < pos; ++sig) lds_inc(done + sig, l);
          while (lds_get(full + pos) == 0u) __builtin_amdgcn_s_sleep(1);
        }
        asm volatile("" ::: "memory");
        LAS char* src = c.lds + rdl(t_lds, pos) + l * 16;
        u32x4 v0 = *(LAS u32x4*)src, v1 = *(LAS u32x4*)(src + 1024);
        const u32 vo = l * 16 + rdl(t_dst, pos);
        asm volatile("s_waitcnt lgkmcnt(0)" : "+v"(v0), "+v"(v1)::"memory");
        lds_inc(freed + pos, l);
        WAIT_VM(1);
        while (((u32)__builtin_amdgcn_s_memrealtime() - t0) * 2 < due) __builtin_amdgcn_s_sleep(1);
        __builtin_amdgcn_raw_buffer_store_b128(v0, rd, vo, 0, SYS);
        WAIT_VM(1);
        if (sig < pos) { lds_inc(done + sig, l); ++sig; }
        while (((u32)__builtin_amdgcn_s_memrealtime() - t0) * 2 < due + PACE) __builtin_amdgcn_s_sleep(1);
        __builtin_amdgcn_raw_buffer_store_b128(v1, rd, vo + 1024, 0, SYS);
        asm volatile("" ::: "memory");
        due += 2 * PACE;
      }
      WAIT_VM(0);
      lds_inc(done + sig, l);
    } else {
      const u32 t_flag = isy ? OFF_GFLAG + (yk * 4 + (pc >> 1)) * 64 + (pc & 1u) * 4 : OFF_IFLAG + cid * 256 + (oslot * 8 + pc) * 4;
#pragma unroll 1
      for (int pos = 0; pos < npos; ++pos) {
        while (lds_get(done + pos) < 4u) __builtin_amdgcn_s_sleep(1);
        if (l == (u32)pos && !lds_get(dead)) put_flag(dst + t_flag, c.call);
        if (pos == GATE_POS && l == 0) put_flag(c.wsl + OFF_GATE + d * 128 + sidx * 4, c.call);
      }
    }
  }
  BARRIER();
}

__device__ __forceinline__ void kloop(const Ctx& c, f32x4 (&acc)[4][2], int ns, u64 order, const char* pa, const char* pb, u32 nrec,
                                      u32 va0, u32 va1, u32 vb0, u32 vb1, u32 sa, u32 sb, u32 stepa, u32 stepb) {
  const u32 lbase = (u32)(size_t)c.lds;
  const u32 m0a = lbase + c.wave * 2048;
  const u32 wm = c.wave >> 2, wn = c.wave & 3, fr = c.lane & 15, fq = c.lane >> 4;
  const u32 xa0 = (wm * 64 + fr) * 128 + ((fq ^ (fr >> 1)) << 4);
  const u32 tq = (c.lane >> 2) & 3, tp = c.lane & 3;
  const u32 wa0 = 16384 + (fq * 8 + tq) * 256 + (((wn * 4 + tp) ^ ((tq << 2) | (fq & 1))) << 4) + ((tp & 1) << 3);

  auto dma = [&](int s, int i) {
    const u32 n = s < ns ? nrec : 0u;
    const u32 img = ((u32)(order >> (2 * (s & ~1))) & 15u) * 2u + (u32)(s & 1);
    const u32 m0v = m0a + (u32)(s & 3) * STAGE_B + (i >> 1) * 16384;
    const auto rs = RSRC(i < 2 ? pa : pb, n);
    const u32 so = i < 2 ? sa + img * stepa : sb + img * stepb;
    const u32 vo = i == 0 ? va0 : i == 1 ? va1 : i == 2 ? vb0 : vb1;
    if (i & 1)
      asm volatile("s_mov_b32 m0, %0\n\ts_nop 0\n\tbuffer_load_dwordx4 %1, %2, %3 offen offset:1024 sc0 sc1 lds"
                   ::"s"(m0v), "v"(vo), "s"(rs), "s"(so) : "memory");
    else
      asm volatile("s_mov_b32 m0, %0\n\ts_nop 0\n\tbuffer_load_dwordx4 %1, %2, %3 offen sc0 sc1 lds"
                   ::"s"(m0v), "v"(vo), "s"(rs), "s"(so) : "memory");
  };

  bf16x8 X[2][2][4];
  s16x4 W[2][2][2][2];
#pragma unroll
  for (int h = 0; h < 2; ++h) {
#pragma unroll
    for (int m = 0; m < 4; ++m) X[1][h][m] = (bf16x8)(__bf16)0.0f;
#pragma unroll
    for (int n = 0; n < 2; ++n) W[1][h][n][0] = W[1][h][n][1] = (s16x4){0, 0, 0, 0};
  }
  auto mm = [&](int o, int ii) {
    const int h = ii >> 3, m = (ii >> 1) & 3, n = ii & 1;
    const bf16x8 wf = __builtin_bit_cast(bf16x8, __builtin_shufflevector(W[o][h][n][0], W[o][h][n][1], 0, 1, 2, 3, 4, 5, 6, 7));
    acc[m][n] = __builtin_amdgcn_mfma_f32_16x16x32_bf16(wf, X[o][h][m], acc[m][n], 0, 0, 0);
  };
  SB();
  asm volatile("s_nop 4" ::: "memory");
  SB();
#pragma unroll
  for (int s = 0; s < 3; ++s)
#pragma unroll
    for (int i = 0; i < 4; ++i) dma(s, i);
#pragma unroll 1
  for (int s2 = 0; s2 < ns; s2 += 2) {
#pragma unroll
    for (int b = 0; b < 2; ++b) {
      const int s = s2 + b, o = b ^ 1;
      WAIT_VM(8);
      BARRIER();
      LAS char* st = c.lds + (u32)(s & 3) * STAGE_B;
      auto rd = [&](int ii) {
        const int h = ii >> 3, i = ii & 7;
        if (i < 4) W[b][h][i >> 1][i & 1] = __builtin_amdgcn_ds_read_tr16_b64_v4i16((LAS s16x4*)(st + (wa0 ^ (u32)((i >> 1) << 3)) + h * 8192 + (i & 1) * 1024));
        else X[b][h][i - 4] = *(LAS bf16x8*)(st + (xa0 ^ (u32)(h << 6)) + (i - 4) * 2048);
      };
      mm(o, 0); SB(); rd(0); rd(1); SB(); mm(o, 1); SB(); rd(4); SB(); mm(o, 2); SB(); dma(s + 3, 0); rd(2); rd(3); SB(); mm(o, 3); SB(); rd(5); SB();
      mm(o, 4); SB(); rd(6); SB(); mm(o, 5); SB(); rd(7); SB(); mm(o, 6); SB(); dma(s + 3, 1); rd(8); rd(9); SB(); mm(o, 7); SB(); rd(12); SB();
      mm(o, 8); SB(); rd(10); rd(11); SB(); mm(o, 9); SB(); rd(13); SB(); mm(o, 10); SB(); dma(s + 3, 2); rd(14); SB(); mm(o, 11); SB(); rd(15); SB();
      mm(o, 12); SB(); mm(o, 13); SB(); dma(s + 3, 3); SB(); mm(o, 14); SB(); mm(o, 15); SB();
    }
  }
#pragma unroll
  for (int i = 0; i < 16; ++i) mm(1, i);
  WAIT_VM(0);
  BARRIER();
}

__device__ __forceinline__ u32 pack2(float a, float b) {
  return __builtin_bit_cast(u32, __builtin_convertvector((f32x2){a, b}, bf16x2));
}
__device__ __forceinline__ float bf_lo(u32 v) { return __builtin_bit_cast(float, v << 16); }
__device__ __forceinline__ float bf_hi(u32 v) { return __builtin_bit_cast(float, v & 0xffff0000u); }

__global__ void __launch_bounds__(NT, 2) gemm_rs(Params p) {
  __shared__ __attribute__((aligned(1024))) char smem[LDS_BYTES];
  Ctx c;
  c.lds = (LAS char*)smem;
  c.lane = threadIdx.x & 63;
  c.wave = rfl(threadIdx.x >> 6);
  const u32 rank = (u32)p.rank & (NR - 1);
  c.wsl = p.wsl;
  STAMP(0);
  gu32* cnt = (gu32*)(c.wsl + OFF_CNT) + blockIdx.x;
  const u32 cnt_v = __hip_atomic_load(cnt, __ATOMIC_RELAXED, __HIP_MEMORY_SCOPE_AGENT);
  ((LAS u32*)(c.lds + LDS_CTL))[threadIdx.x] = 0;

  const u32 grp = (X2G >> ((blockIdx.x & 7) * 4)) & 7, u = blockIdx.x >> 3;
  const bool isG = grp & 1, isO = (grp & 3) == 2;
  u32 j, ct;
  if (isG) { j = u & 1; ct = u >> 1; }
  else { j = u & 3; ct = (u >> 2) * 2 + (grp >> 2); }
  const bool sender = isG ? u >= 24 : (isO && j == 3);
  const int ntiles = sender ? 0 : ((isG ? (u < 16 && j == 0) : (!isO && u < 8)) ? 2 : 1);

  if (sender) {
    link_engine(c, p, grp, (isG ? u : u >> 2) & 7, cnt_v);
  } else {
    const u64 t0 = __builtin_amdgcn_s_memrealtime();
    const u64 hold = (isG || isO) ? DELAY_GO : (ct >= 4 ? DELAY_P : 0);
    while (__builtin_amdgcn_s_memrealtime() - t0 < hold) __builtin_amdgcn_s_sleep(4);
    c.call = rfl(cnt_v) + 1;
    WAIT_VM(0);
    BARRIER();
  }
  __builtin_amdgcn_s_waitcnt(0x0F70);
  const u32 par_off = OFF_STAGE + (c.call & 1) * PAR_BYTES;
  if (threadIdx.x == 0) __hip_atomic_store(cnt, c.call, __ATOMIC_RELAXED, __HIP_MEMORY_SCOPE_AGENT);
  STAMP(1);

  const auto rws = RSRC(c.wsl, WS_NEED);
  const u32 NANS = 0x7fc07fc0u;
  bool bad = false;
  u32x4 y[4];
  const u32 vy = c.wave * 4096 + c.lane * 16;
  u32 tj = j, tct = ct, nwait = 0;
  bool tO = isO && !sender;
  f32x4 acc[4][2];
  constexpr u64 ORD_LIN = 0xba9876543210ull, ORD_REL = 0x65217430ba98ull;
  constexpr u64 ALL48 = (1ull << 48) - 1, NEED_O = (1ull << 32) - 1, NEED_REL = 0xfffff00ff00full;
  int tile = 0, part = 0;
#pragma unroll 1
  while (tile < ntiles) {
    const u32 rt = (isG ? 3 + grp : (tO ? 12 : 0)) + tj;
    const bool rel = !tO && tct >= 16 - NREL;
    u32 va0, va1, vb0, vb1;
    int ns = 8;
    u64 order = ORD_LIN;
    if (part == 0) {
#pragma unroll
      for (int m = 0; m < 4; ++m) acc[m][0] = acc[m][1] = (f32x4){0.f, 0.f, 0.f, 0.f};
      const u32 xr = c.wave * 16 + (c.lane >> 3);
      va0 = xr * 1024 + (((c.lane & 7) ^ ((xr >> 1) & 7)) << 4);
      va1 = (xr + 8) * 1024 + (((c.lane & 7) ^ (((xr + 8) >> 1) & 7)) << 4) - 1024;
      const u32 wk = c.wave * 8 + (c.lane >> 4);
      vb0 = wk * 8192 + (((c.lane & 15) ^ (((wk & 3) << 2) | ((wk >> 3) & 1))) << 4);
      vb1 = vb0 + 4 * 8192 - 1024;
    } else {
      if (tile == 0 && part == 1) { STAMP(2); }
      const u64 need = part == 2 ? ALL48 : (tO && NREL > 0 ? NEED_O : (rel ? NEED_REL : ALL48));
      const u32 r = wait_words(c, OFF_IFLAG + rt * 256, OFF_IFLAG + (16 + tct) * 256, 48, need, nwait++);
      bad |= r & 1;
      if (tile == 0 && part == 1) { STAMP(3); }
      if (tile == 0 && part == 2) { STAMP(10); }
      if (rel) order = ORD_REL;
      if (part == 2) order >>= 32;
      else if (r & 2) { ns = 24; part = 2; }
      else ns = 16;
      va0 = va1 = vb0 = vb1 = c.wave * 2048 + c.lane * 16;
    }
    const bool staged = part > 0;
    kloop(c, acc, ns, order, staged ? c.wsl : p.x, staged ? c.wsl : p.w, staged ? WS_NEED : IN_BYTES, va0, va1, vb0, vb1,
          rfl(staged ? par_off + rt * 6 * CHUNK : row0_of(rank, rt) * 1024),
          rfl(staged ? par_off + (16 + tct) * 6 * CHUNK : (rank & 1) * 4096 + tct * 256),
          staged ? 16384 : 128, staged ? 16384 : 64 * 8192);
    if (++part == 3) {
      if (tile == 0) { STAMP(4); }
      const bool odd = (c.lane >> 4) & 1;
#pragma unroll
      for (int m = 0; m < 4; ++m) {
        const f32x4 lo = odd ? acc[m][1] : acc[m][0], hi = odd ? acc[m][0] : acc[m][1];
        acc[m][0] = lo; acc[m][1] = hi;
        y[m].x = pack2(lo[0], lo[1]); y[m].y = pack2(lo[2], lo[3]);
        y[m].z = pack2(hi[0], hi[1]); y[m].w = pack2(hi[2], hi[3]);
        if (bad) y[m] = (u32x4){NANS, NANS, NANS, NANS};
      }
      if (!tO) {
        const u32 gp = gpos(tj, tct), gt = (grp >> 1) * 32 + gp;
        const bool fwd = isG && gp < 2 * NFWD, direct = isG && !fwd;
        if (direct) (void)wait_words(c, OFF_GATE + grp * 128, OFF_GATE + grp * 128, NSEND, ALL48, nwait++);
        STAMP(7);
        char* dst = p.ws[(rank ^ (fwd ? 0u : (isG ? grp : 4u))) & (NR - 1)];
        const auto rd = RSRC(dst, WS_NEED);
        const u32 yo = vy + (fwd ? OFF_LY + gt * 32768 : par_off + YS_OFF + (isG ? 64 + gt : j * 16 + ct) * 32768);
#pragma unroll
        for (int m = 0; m < 4; ++m) {
          __builtin_amdgcn_raw_buffer_store_b128(y[m], rd, yo + m * 1024, 0, SYS);
          if (m < 3 && direct) WAIT_VM(0);
        }
        WAIT_VM(0);
        const u32 fo = direct ? OFF_GFLAG + gt * 128 + (c.lane >> 1) * 64 + (c.lane & 1) * 4 : (fwd ? OFF_LYF + gt * 4 : OFF_YFLAG + (j * 16 + ct) * 128);
        if (last_wave(c, tile) && c.lane < (direct ? 4u : 1u)) put_flag(dst + fo, c.call);
        if (tile == 0) { STAMP(6); }
      }
      part = 0;
      if (++tile < ntiles) {
        if (isG) { tj = (u >> 1) & 1; tct = 12 + (u >> 2); }
        else { tO = true; tj = 3; tct = u * 2 + (grp >> 2); }
      }
    }
  }
  STAMP(5);

  const auto rout = RSRC(p.out, IN_BYTES);

  if (tO) {
    const u32 yt = rfl(tj * 16 + tct);
    bad |= wait_words(c, OFF_YFLAG + yt * 128, OFF_YFLAG + yt * 128, 1, ALL48, nwait) & 1;
    STAMP(8);
#pragma unroll
    for (int m = 0; m < 4; ++m) y[m] = __builtin_amdgcn_raw_buffer_load_b128(rws, vy + m * 1024, par_off + YS_OFF + yt * 32768, SYS);
    const u32 vo = (tj * 128 + (c.wave >> 2) * 64 + (c.lane & 15)) * 8192 + (rank & 1) * 4096 + tct * 256 + (c.wave & 3) * 64 + (c.lane >> 4) * 16;
#pragma unroll
    for (int m = 0; m < 4; ++m) {
      u32x4 o;
      o.x = pack2(acc[m][0][0] + bf_lo(y[m].x), acc[m][0][1] + bf_hi(y[m].x));
      o.y = pack2(acc[m][0][2] + bf_lo(y[m].y), acc[m][0][3] + bf_hi(y[m].y));
      o.z = pack2(acc[m][1][0] + bf_lo(y[m].z), acc[m][1][1] + bf_hi(y[m].z));
      o.w = pack2(acc[m][1][2] + bf_lo(y[m].w), acc[m][1][3] + bf_hi(y[m].w));
      if (bad) o = (u32x4){NANS, NANS, NANS, NANS};
      __builtin_amdgcn_raw_buffer_store_b128(o, rout, vo + m * 16 * 8192, 0, SYS);
    }
  } else if (isG) {
    const u32 gt = ((grp >> 1) & 1) * 32 + gpos(j, ct);
    const u32 ya = 64 + gt, yb = ya + 64;
    const u32 fo = OFF_GFLAG + (gt * 2 + (grp >> 2)) * 64;
    bad |= wait_words(c, fo, fo + 128 * 64, 2, ALL48, nwait) & 1;
    STAMP(8);
    const u32 pw = (grp >> 2) * 4 + (c.wave >> 1), m0 = (c.wave & 1) * 2;
    const u32 vr = pw * 4096 + m0 * 1024 + c.lane * 16;
    u32x4 ra[2], rb[2];
#pragma unroll
    for (int m = 0; m < 2; ++m) {
      ra[m] = __builtin_amdgcn_raw_buffer_load_b128(rws, vr + m * 1024, par_off + YS_OFF + ya * 32768, SYS);
      rb[m] = __builtin_amdgcn_raw_buffer_load_b128(rws, vr + m * 1024, par_off + YS_OFF + yb * 32768, SYS);
    }
    const u32 vo = (((grp >> 1) & 1) * 256 + j * 128 + (pw >> 2) * 64 + m0 * 16 + (c.lane & 15)) * 8192 +
                   ((rank & 1) ^ 1) * 4096 + ct * 256 + (pw & 3) * 64 + (c.lane >> 4) * 16;
#pragma unroll
    for (int m = 0; m < 2; ++m) {
      u32x4 o;
      o.x = pack2(bf_lo(ra[m].x) + bf_lo(rb[m].x), bf_hi(ra[m].x) + bf_hi(rb[m].x));
      o.y = pack2(bf_lo(ra[m].y) + bf_lo(rb[m].y), bf_hi(ra[m].y) + bf_hi(rb[m].y));
      o.z = pack2(bf_lo(ra[m].z) + bf_lo(rb[m].z), bf_hi(ra[m].z) + bf_hi(rb[m].z));
      o.w = pack2(bf_lo(ra[m].w) + bf_lo(rb[m].w), bf_hi(ra[m].w) + bf_hi(rb[m].w));
      if (bad) o = (u32x4){NANS, NANS, NANS, NANS};
      __builtin_amdgcn_raw_buffer_store_b128(o, rout, vo + m * 16 * 8192, 0, SYS);
    }
  }
}

}

extern "C" void kernel_launch(int rank, int nranks, ncclComm_t comm, hipStream_t stream,
                              void* const* d_in, const int* in_sizes, int n_in,
                              void* d_out, int out_size,
                              const int* in_dtypes, int out_dtype,
                              void* const* d_out_all,
                              void* const* d_ws_all, size_t ws_bytes) {
  (void)comm; (void)in_dtypes; (void)out_dtype; (void)d_out_all;
  if (nranks != NR || (unsigned)rank >= (unsigned)NR || n_in != 2 || ws_bytes < WS_NEED ||
      in_sizes[0] != 4096 * 512 || in_sizes[1] != 512 * 4096 || out_size != 512 * 4096) {
    fprintf(stderr, "gemm_rs: unexpected problem shape\n");
    return;
  }
  Params p;
  p.x = (const char*)d_in[0];
  p.w = (const char*)d_in[1];
  p.out = (char*)d_out;
  p.wsl = (char*)d_ws_all[rank];
  p.rank = 0;
  for (int i = 0; i < NR; ++i) {
    const int r = (L2P >> (4 * i)) & 7;
    p.ws[i] = (char*)d_ws_all[r];
    if (r == rank) p.rank = i;
  }
  hipLaunchKernelGGL(gemm_rs, dim3(NWG), dim3(NT), 0, stream, p);
}
```

```cpp
#include <hip/hip_runtime.h>
#include <rccl/rccl.h>
#include <cstdio>

namespace {

typedef unsigned u32;
typedef unsigned long long u64;
typedef __attribute__((ext_vector_type(4))) unsigned u32x4;
typedef __attribute__((ext_vector_type(4))) float f32x4;
typedef __attribute__((ext_vector_type(2))) float f32x2;
typedef __attribute__((ext_vector_type(2))) __bf16 bf16x2;
typedef __attribute__((ext_vector_type(8))) __bf16 bf16x8;
typedef __attribute__((ext_vector_type(4))) short s16x4;
#define LAS __attribute__((address_space(3)))
#define GAS __attribute__((address_space(1)))
typedef GAS u32 gu32;

constexpr int NR = 8;
constexpr int NWG = 256;
constexpr int NT = 512;
constexpr u32 MiB = 1u << 20;
constexpr u32 CHUNK = 65536;
constexpr int NSEND = 8;
constexpr u32 PIECE = CHUNK / NSEND;
constexpr u32 IN_BYTES = 4 * MiB;

constexpr u32 OFF_CNT = 0;
constexpr u32 OFF_CLAIM = 131072;
constexpr u32 OFF_TMO = 4096;
constexpr u32 OFF_LYF = 8192;
constexpr u32 OFF_GATE = 12288;
constexpr u32 OFF_IFLAG = 16384;
constexpr u32 OFF_YFLAG = 32768;
constexpr u32 OFF_GFLAG = 40960;
constexpr u32 OFF_STAMP = 65536;
constexpr u32 OFF_STAGE = 1 * MiB;
constexpr u32 YS_OFF = 12 * MiB;
constexpr u32 PAR_BYTES = 18 * MiB;
constexpr u32 OFF_LY = OFF_STAGE + 2 * PAR_BYTES;
constexpr u32 WS_NEED = OFF_LY + 4 * MiB;

constexpr u64 DEADLINE = 40000000ull;

constexpr u32 STAGE_B = 32768;
constexpr u32 LDS_CTL = 4 * STAGE_B;
constexpr u32 LDS_BYTES = LDS_CTL + 2048;

enum { K_X = 0, K_W = 1, K_Y = 2, K_H = 3, K_R = 4 };
#define X(i) (K_X * 16 + (i))
#define W(i) (K_W * 16 + (i))
#define Y(i) (K_Y * 16 + (i))
#define H(i) (K_H * 16 + (i))
#define R(i) (K_R * 16 + (i))
#define NREL 6
#define SEQ_HEAD X(0), W(0), W(1), X(1), W(2), X(2), X(3), W(3), X(4), X(6), W(4), X(8), W(5), X(10), W(6), X(5), \
                 W(7), X(7), W(8), X(9), W(9), X(11)
constexpr int NFWD = 4;
constexpr int NPOS_G = 32 - NREL / 2 + NFWD, NPOS_O = 32 + NREL;
constexpr unsigned char SEQ_G[48] = {SEQ_HEAD,
    H(10), H(12), H(14),
    X(12), X(13), X(14), X(15), Y(0), Y(1), Y(2), Y(3)};
constexpr unsigned char SEQ_O[48] = {SEQ_HEAD,
    W(10), W(11), W(12), W(13), R(10), R(11), W(14), R(12), W(15), R(13), R(14), R(15),
    X(12), X(13), X(14), X(15)};
#undef X
#undef W
#undef Y
#undef H
#undef R
constexpr u64 seq_pack(const unsigned char* s, int from) { u64 r = 0; for (int t = 0; t < 8; ++t) r |= (u64)s[from + t] << (8 * t); return r; }

#define GATE_POS (NPOS_G - 4)
#define PACE 113
#define PACE_T0 100
#define DELAY_GO 600
#define DELAY_P 400

constexpr u32 L2P = 0x53764210u;
constexpr u32 X2G = 0x75314602u;

struct Params {
  const char* x;
  const char* w;
  char* out;
  char* wsl;
  char* ws[NR];
  int rank;
};

#define RSRC(p, n) __builtin_amdgcn_make_buffer_rsrc((void*)(p), 0, (n), 0x00020000)
#define WAIT_VM(n) asm volatile("s_waitcnt vmcnt(%0)" ::"n"(n) : "memory")
#define BARRIER() do { asm volatile("s_waitcnt lgkmcnt(0)" ::: "memory"); __builtin_amdgcn_s_barrier(); asm volatile("" ::: "memory"); } while (0)
#define SB() __builtin_amdgcn_sched_barrier(0)
constexpr int SYS = 17;

__device__ __forceinline__ u32 rfl(u32 v) { return __builtin_amdgcn_readfirstlane(v); }
__device__ __forceinline__ __amdgpu_buffer_rsrc_t srsrc(const char* p, u32 n) {
  return RSRC(((u64)rfl((u32)((u64)p >> 32)) << 32) | rfl((u32)(u64)p), rfl(n));
}
__device__ __forceinline__ u32 rdl(u32 v, int lane) { return __builtin_amdgcn_readlane(v, lane); }
__device__ __forceinline__ void put_flag(char* addr, u32 v) { __hip_atomic_store((gu32*)addr, v, __ATOMIC_RELAXED, __HIP_MEMORY_SCOPE_SYSTEM); }
__device__ __forceinline__ u32 get_flag(const char* addr) { return __hip_atomic_load((gu32*)addr, __ATOMIC_RELAXED, __HIP_MEMORY_SCOPE_SYSTEM); }
__device__ __forceinline__ u32 lds_get(LAS u32* w) { return rfl(__hip_atomic_load(w, __ATOMIC_RELAXED, __HIP_MEMORY_SCOPE_WORKGROUP)); }
__device__ __forceinline__ void lds_inc(LAS u32* w, u32 lane) {
  if (lane == 0) __hip_atomic_fetch_add(w, 1u, __ATOMIC_RELAXED, __HIP_MEMORY_SCOPE_WORKGROUP);
}

__device__ __forceinline__ u32 row0_of(u32 q, u32 rt) {
  const bool g = rt - 4u < 8u;
  const u32 d = g ? ((rt - 4u) | 1u) : (rt < 4u ? 4u : 0u);
  const u32 t = g ? (d & 2u) + (rt & 1u) : (rt & 3u);
  return ((L2P >> ((q ^ d) * 4u)) & 7u) * 512u + t * 128u;
}
__device__ __forceinline__ u32 slot_of(u32 d) { return (d & 1u) ? d >> 1 : 4u + (d >> 2); }
__device__ __forceinline__ u32 gpos(u32 j, u32 ct) { return j * 16u + ct; }

struct Ctx {
  LAS char* lds;
  char* wsl;
  u32 call, lane, wave, t_start;
};
#define STAMP(i)

__device__ __forceinline__ u32 wait_words(const Ctx& c, u32 off0, u32 off1, u32 n, u64 need, u32 slot) {
  LAS u32* res = (LAS u32*)(c.lds + LDS_CTL + 256) + slot;
  if (c.wave == 0) {
    const char* p0 = c.wsl + (c.lane < n ? off0 + c.lane * 4 : OFF_TMO);
    const char* p1 = c.wsl + (c.lane < n ? off1 + c.lane * 4 : OFF_TMO);
    const u64 t0 = __builtin_amdgcn_s_memrealtime();
    u32 r;
    for (;;) {
      const u32 v0 = get_flag(p0), v1 = get_flag(p1);
      if (rdl(v1, 63) == c.call) { r = 3; break; }
      const u64 miss = __ballot(c.lane < n && (v0 < c.call || v1 < c.call));
      if ((miss & need) == 0) { r = miss ? 0 : 2; break; }
      if (__builtin_amdgcn_s_memrealtime() - t0 > DEADLINE) {
        if (c.lane == 0) put_flag(c.wsl + OFF_TMO, c.call);
        r = 3;
        break;
      }
      __builtin_amdgcn_s_sleep(2);
    }
    if (c.lane == 0) *res = r;
  }
  BARRIER();
  return rfl(*res);
}

__device__ __forceinline__ bool last_wave(const Ctx& c, int idx) {
  LAS u32* cnt = (LAS u32*)(c.lds + LDS_CTL) + idx;
  u32 old = 0;
  if (c.lane == 0) old = __hip_atomic_fetch_add(cnt, 1u, __ATOMIC_RELAXED, __HIP_MEMORY_SCOPE_WORKGROUP);
  return rfl(old) == 7u;
}

__device__ __forceinline__ u32 kof_of(u32 q, u32 st) { return (st < 2u ? (q >> 2) : 1u + (((q >> 2) ^ (q >> 1)) & 1u)) * 128u + st * 64u; }

struct Fill { u32 vx[4], vw[4]; };
__device__ __forceinline__ void fill_slot(const Ctx& c, int pos, const u32 (&vo)[4], __amdgpu_buffer_rsrc_t rs, u32 so, u32 sstep) {
  const u32 lb = (u32)(size_t)c.lds + (u32)(pos & 7) * PIECE;
#pragma unroll
  for (int g = 0; g < 2; ++g)
    asm volatile("s_mov_b32 m0, %0\n\ts_nop 3\n\t"
                 "buffer_load_dwordx4 %1, %5, %6 offen sc0 sc1 lds\n\t"
                 "buffer_load_dwordx4 %2, %5, %6 offen offset:1024 sc0 sc1 lds\n\t"
                 "buffer_load_dwordx4 %3, %5, %6 offen offset:2048 sc0 sc1 lds\n\t"
                 "buffer_load_dwordx4 %4, %5, %6 offen offset:3072 sc0 sc1 lds"
                 ::"s"(rfl(lb + g * 4096u)), "v"(vo[0]), "v"(vo[1]), "v"(vo[2]), "v"(vo[3]), "s"(rs), "s"(rfl(so + g * sstep)) : "memory");
}
__device__ __forceinline__ void first_fill(const Ctx& c, const Params& p, u32 q, u32 sidx, Fill& f) {
  static_assert(SEQ_G[0] == 0x00 && SEQ_G[1] == 0x10 && SEQ_G[2] == 0x11, "first positions");
  const u32 l = c.lane, pos = c.wave - 4, pc = (sidx + pos) & 7u, kof = kof_of(q, pc >> 1);
#pragma unroll
  for (int i = 0; i < 4; ++i) {
    const u32 xr = i * 8 + (l >> 3), wk = i * 4 + (l >> 4);
    f.vx[i] = xr * 1024 + (((l & 7) ^ ((xr >> 1) & 7)) << 4) - i * 1024;
    f.vw[i] = wk * 8192 + (q & 1) * 4096 + (((l & 15) ^ (((wk & 3) << 2) | ((wk >> 3) & 1))) << 4) - i * 1024;
  }
  u32 mx = pos - 1;
  asm("s_ashr_i32 %0, %0, 31" : "+s"(mx)::"scc");
  u32 vo[4];
#pragma unroll
  for (int i = 0; i < 4; ++i) vo[i] = (f.vx[i] & mx) | (f.vw[i] & ~mx);
  const u32 sx = row0_of(q, 0) * 1024u + kof * 2u + (pc & 1u) * 65536u, sw = (kof + (pc & 1u) * 32u) * 8192u + (pos - 1) * 256u;
  fill_slot(c, pos, vo, srsrc(mx ? p.x : p.w, IN_BYTES), (sx & mx) | (sw & ~mx),
            (mx & 32768u) | (~mx & 131072u));
}

__device__ __forceinline__ void link_engine(Ctx& c, const Params& p, u32 d, u32 sidx, const Fill& f) {
  const u32 q = ((u32)p.rank ^ d) & (NR - 1);
  char* dst = p.ws[q];
  const bool odd = d & 1;
  const int npos = odd ? NPOS_G : NPOS_O;
  LAS u32* done = (LAS u32*)(c.lds + LDS_CTL + 512);
  LAS u32* full = (LAS u32*)(c.lds + LDS_CTL + 768);
  LAS u32* freed = (LAS u32*)(c.lds + LDS_CTL + 1024);
  LAS u32* dead = (LAS u32*)(c.lds + LDS_CTL + 320);
  const u32 l = c.lane;
  u64 tab = seq_pack(SEQ_G, 0);
  if (l >= 8) tab = seq_pack(SEQ_G, 8);
  if (l >= 16) tab = odd ? seq_pack(SEQ_G, 16) : seq_pack(SEQ_O, 16);
  if (l >= 24) tab = odd ? seq_pack(SEQ_G, 24) : seq_pack(SEQ_O, 24);
  if (l >= 32) tab = odd ? seq_pack(SEQ_G, 32) : seq_pack(SEQ_O, 32);
  if (l >= 40) tab = odd ? seq_pack(SEQ_G, 40) : seq_pack(SEQ_O, 40);
  const u32 ent = (u32)(tab >> (8 * (l & 7))) & 255u, kind = ent >> 4;
  const u32 ts = (sidx + l) & 7u;
  const bool isy = kind == K_Y, isr = kind == K_R, split = kind >= K_H;
  const u32 id = (ent & 15u) + (kind == K_H ? ts >> 2 : 0u);
  const u32 pc = split ? ((isr ? d : d << 1) & 4u) + (ts & 3u) : ts;
  const u32 dorg = isr ? (d >> 1) + (ts & 4u) : 0u;
  const u32 cid = (kind == K_X ? 0u : 16u) + id, yk = (d >> 1) * 16 + id;
  const u32 par = OFF_STAGE + (c.call & 1) * PAR_BYTES;
  if (c.wave >= 4 && c.wave < 7) {
    WAIT_VM(0);
    lds_inc(full + (c.wave - 4), l);
    const u32 kof = kof_of(q, pc >> 1);
    const u32 t_src = isy ? OFF_LY + yk * CHUNK + pc * PIECE
                    : isr ? par + (cid * 6 + slot_of(dorg)) * CHUNK + pc * PIECE
                    : kind == K_X ? row0_of(q, id) * 1024u + kof * 2u + (pc & 1u) * 65536u
                                  : (kof + (pc & 1u) * 32u) * 8192u + id * 256u;
    const u32 t_mx = kind == K_X ? ~0u : 0u, t_ml = (isy || isr) ? ~0u : 0u;
    const u32 t_dep = isy ? OFF_LYF + (yk * 2 + (pc >> 2)) * 4 : OFF_IFLAG + cid * 256 + (slot_of(dorg) * 8 + pc) * 4;
#pragma unroll 1
    for (int pos = (int)c.wave - 1; pos < npos; pos += 3) {
      const u32 mx = rdl(t_mx, pos), ml = rdl(t_ml, pos), mw = ~(mx | ml);
      u32 vo[4];
#pragma unroll
      for (int i = 0; i < 4; ++i) vo[i] = (f.vx[i] & mx) | (f.vw[i] & mw) | ((l * 16) & ml);
      if (pos >= 8) while (lds_get(freed + (pos - 8)) < 4u) __builtin_amdgcn_s_sleep(1);
      if (ml) {
        const char* dep = c.wsl + rdl(t_dep, pos);
        const u64 t0 = __builtin_amdgcn_s_memrealtime();
        while (rfl(get_flag(dep)) < c.call) {
          if (lds_get(dead) || __builtin_amdgcn_s_memrealtime() - t0 > 2 * DEADLINE) {
            if (l == 0) {
              put_flag(c.wsl + OFF_TMO, c.call);
              __hip_atomic_store(dead, 1u, __ATOMIC_RELAXED, __HIP_MEMORY_SCOPE_WORKGROUP);
            }
            break;
          }
          __builtin_amdgcn_s_sleep(2);
        }
      }
      fill_slot(c, pos, vo, srsrc(ml ? c.wsl : (mw ? p.w : p.x), ml ? WS_NEED : IN_BYTES), rdl(t_src, pos),
                (mx & 32768u) | (mw & 131072u) | (ml & 4096u));
      WAIT_VM(0);
      lds_inc(full + pos, l);
    }
  } else {
    const u32 oslot = slot_of(d ^ dorg);
    if (c.wave < 4) {
      const u32 tw = (c.wave + l + (l >> 2)) & 3u;
      const u32 t_dst = par + pc * PIECE + tw * 2048u +
                        (isy ? YS_OFF + 2 * MiB + yk * CHUNK : (cid * 6 + oslot) * CHUNK);
      const u32 t_lds = (l & 7u) * PIECE + tw * 2048u;
      const auto rd = RSRC(dst, WS_NEED);
      int sig = 0;
      const u32 t0 = c.t_start;
      u32 due = 2 * PACE_T0 + (sidx * 4 + c.wave) * PACE / 32;
#pragma unroll 1
      for (int pos = 0; pos < npos; ++pos) {
        if (lds_get(full + pos) == 0u) {
          WAIT_VM(0);
          for (; sig < pos; ++sig) lds_inc(done + sig, l);
          while (lds_get(full + pos) == 0u) __builtin_amdgcn_s_sleep(1);
        }
        asm volatile("" ::: "memory");
        LAS char* src = c.lds + rdl(t_lds, pos) + l * 16;
        u32x4 v0 = *(LAS u32x4*)src, v1 = *(LAS u32x4*)(src + 1024);
        const u32 vo = l * 16 + rdl(t_dst, pos);
        asm volatile("s_waitcnt lgkmcnt(0)" : "+v"(v0), "+v"(v1)::"memory");
        lds_inc(freed + pos, l);
        WAIT_VM(1);
        while (((u32)__builtin_amdgcn_s_memrealtime() - t0) * 2 < due) __builtin_amdgcn_s_sleep(1);
        __builtin_amdgcn_raw_buffer_store_b128(v0, rd, vo, 0, SYS);
        WAIT_VM(1);
        if (sig < pos) { lds_inc(done + sig, l); ++sig; }
        while (((u32)__builtin_amdgcn_s_memrealtime() - t0) * 2 < due + PACE) __builtin_amdgcn_s_sleep(1);
        __builtin_amdgcn_raw_buffer_store_b128(v1, rd, vo + 1024, 0, SYS);
        asm volatile("" ::: "memory");
        due += 2 * PACE;
      }
      WAIT_VM(0);
      lds_inc(done + sig, l);
    } else {
      const u32 t_flag = isy ? OFF_GFLAG + (yk * 4 + (pc >> 1)) * 64 + (pc & 1u) * 4 : OFF_IFLAG + cid * 256 + (oslot * 8 + pc) * 4;
#pragma unroll 1
      for (int pos = 0; pos < npos; ++pos) {
        while (lds_get(done + pos) < 4u) __builtin_amdgcn_s_sleep(1);
        if (l == (u32)pos && !lds_get(dead)) put_flag(dst + t_flag, c.call);
        if (pos == GATE_POS && l == 0) put_flag(c.wsl + OFF_GATE + d * 128 + sidx * 4, c.call);
      }
    }
  }
  BARRIER();
}

__device__ __forceinline__ bool gemm_tile(Ctx& c, const Params& p, f32x4 (&acc)[4][2], u64 order, u64 need, u32 own_a, u32 own_b,
                                          u32 stg_a, u32 stg_b, u32 flag_a, u32 flag_b, u32& nwait, bool stamp) {
  constexpr u64 ALL48 = (1ull << 48) - 1;
  order = ((u64)rfl((u32)(order >> 32)) << 32) | rfl((u32)order);
  const u32 m0a = (u32)(size_t)c.lds + c.wave * 2048;
  const u32 wm = c.wave >> 2, wn = c.wave & 3, fr = c.lane & 15, fq = c.lane >> 4;
  const u32 xa0 = (wm * 64 + fr) * 128 + ((fq ^ (fr >> 1)) << 4);
  const u32 tq = (c.lane >> 2) & 3, tp = c.lane & 3;
  const u32 wa0 = 16384 + (fq * 8 + tq) * 256 + (((wn * 4 + tp) ^ ((tq << 2) | (fq & 1))) << 4) + ((tp & 1) << 3);
  LAS u32* samp = (LAS u32*)(c.lds + LDS_CTL + 1280);
  const auto rws = RSRC(c.wsl, WS_NEED);
  const u32 vfa = flag_a + (c.lane < 48 ? c.lane : 0) * 4, vfb = flag_b - 256 + (c.lane < 48 ? c.lane : 0) * 4;

  const u32 xr = c.wave * 16 + (c.lane >> 3), wk = c.wave * 8 + (c.lane >> 4);
  u32 va0 = xr * 1024 + (((c.lane & 7) ^ ((xr >> 1) & 7)) << 4);
  u32 va1 = (xr + 8) * 1024 + (((c.lane & 7) ^ (((xr + 8) >> 1) & 7)) << 4) - 1024;
  u32 vb0 = wk * 8192 + (((c.lane & 15) ^ (((wk & 3) << 2) | ((wk >> 3) & 1))) << 4);
  u32 vb1 = vb0 + 4 * 8192 - 1024;
  auto desc = [](const char* q) { return (u32x4){rfl((u32)(u64)q), rfl((u32)((u64)q >> 32)) & 0xffffu, 0u, 0x00020000u}; };
  u32x4 ra = desc(p.x), rb = desc(p.w);
  u32 nrec = IN_BYTES, sa = rfl(own_a), sb = rfl(own_b), stepa = 128, stepb = 64 * 8192;
  stg_a = rfl(stg_a); stg_b = rfl(stg_b);
  int limit = 8;

  u32 soa, sob, m0v;
  auto prep = [&](int s) {
    const u32 img = ((u32)(order >> ((2 * (s & ~1)) & 63)) & 15u) * 2u + (u32)(s & 1);
    soa = sa + img * stepa; sob = sb + img * stepb;
    m0v = m0a + (u32)(s & 3) * STAGE_B;
    ra.z = rb.z = s < limit ? nrec : 0u;
    asm volatile("" : "+s"(soa), "+s"(sob), "+s"(m0v), "+s"(ra), "+s"(rb));
  };
  auto dma = [&](int i) {
    if (i == 0) asm volatile("s_mov_b32 m0, %0\n\ts_nop 0\n\tbuffer_load_dwordx4 %1, %2, %3 offen sc0 sc1 lds" ::"s"(m0v), "v"(va0), "s"(ra), "s"(soa) : "memory");
    if (i == 1) asm volatile("s_mov_b32 m0, %0\n\ts_nop 0\n\tbuffer_load_dwordx4 %1, %2, %3 offen offset:1024 sc0 sc1 lds" ::"s"(m0v), "v"(va1), "s"(ra), "s"(soa) : "memory");
    if (i == 2) asm volatile("s_add_u32 m0, %0, 0x4000\n\ts_nop 0\n\tbuffer_load_dwordx4 %1, %2, %3 offen sc0 sc1 lds" ::"s"(m0v), "v"(vb0), "s"(rb), "s"(sob) : "memory", "scc");
    if (i == 3) asm volatile("s_add_u32 m0, %0, 0x4000\n\ts_nop 0\n\tbuffer_load_dwordx4 %1, %2, %3 offen offset:1024 sc0 sc1 lds" ::"s"(m0v), "v"(vb1), "s"(rb), "s"(sob) : "memory", "scc");
  };
  bf16x8 X[2][2][4];
  s16x4 W[2][2][2][2];
  auto mm = [&](int o, int ii) {
    const int h = ii >> 3, m = (ii >> 1) & 3, n = ii & 1;
    const bf16x8 wf = __builtin_bit_cast(bf16x8, __builtin_shufflevector(W[o][h][n][0], W[o][h][n][1], 0, 1, 2, 3, 4, 5, 6, 7));
    acc[m][n] = __builtin_amdgcn_mfma_f32_16x16x32_bf16(wf, X[o][h][m], acc[m][n], 0, 0, 0);
  };
#pragma unroll
  for (int m = 0; m < 4; ++m) acc[m][0] = acc[m][1] = (f32x4){0.f, 0.f, 0.f, 0.f};
#pragma unroll
  for (int h = 0; h < 2; ++h) {
#pragma unroll
    for (int m = 0; m < 4; ++m) X[1][h][m] = (bf16x8)(__bf16)0.0f;
#pragma unroll
    for (int n = 0; n < 2; ++n) W[1][h][n][0] = W[1][h][n][1] = (s16x4){0, 0, 0, 0};
  }
  bool bad = false;
  int t2 = 0;
#pragma unroll 1
  for (;;) {
    SB();
    asm volatile("s_nop 4" ::: "memory");
    SB();
#pragma unroll
    for (int s = 0; s < 3; ++s) {
      prep(2 * t2 + s);
#pragma unroll
      for (int i = 0; i < 4; ++i) dma(i);
    }
    prep(2 * t2 + 3);
#pragma unroll 1
    for (; 2 * t2 < limit; ++t2) {
#pragma unroll
      for (int b = 0; b < 2; ++b) {
        const int s = 2 * t2 + b, o = b ^ 1;
        WAIT_VM(8);
        BARRIER();
#pragma unroll
        for (int m = 0; m < 4; ++m) asm volatile("" : "+a"(acc[m][0]), "+a"(acc[m][1]));
        LAS char* st = c.lds + (u32)(s & 3) * STAGE_B;
        auto rd = [&](int ii) {
          const int h = ii >> 3, i = ii & 7;
          if (i < 4) W[b][h][i >> 1][i & 1] = __builtin_amdgcn_ds_read_tr16_b64_v4i16((LAS s16x4*)(st + (wa0 ^ (u32)((i >> 1) << 3)) + h * 8192 + (i & 1) * 1024));
          else X[b][h][i - 4] = *(LAS bf16x8*)(st + (xa0 ^ (u32)(h << 6)) + (i - 4) * 2048);
        };
        mm(o, 0); SB(); rd(0); rd(1); SB(); mm(o, 1); SB(); rd(4); SB(); mm(o, 2); SB();
        if (b == 1 && __builtin_expect((u32)(limit - s - 3) <= 2u && limit < 32, 0)) {
          if (s + 3 == limit) {
            const u32 f0 = samp[c.lane], f1 = samp[64 + c.lane];
            const u64 miss = __ballot((f0 < f1 ? f0 : f1) < c.call);
            limit = (int)rfl(miss == 0 ? 32u : ((limit == 8 && (miss & need) == 0) ? 24u : (u32)limit));
            if (s == 5) {
              va0 = va1 = vb0 = vb1 = c.wave * 2048 + c.lane * 16;
              ra = rb = desc(c.wsl); nrec = WS_NEED; sa = stg_a; sb = stg_b; stepa = stepb = 16384;
            }
            prep(s + 3);
            asm volatile("s_nop 4" ::: "memory");
          } else if (c.wave == 0) {
            asm volatile("s_mov_b32 m0, %0\n\ts_nop 3\n\tbuffer_load_dword %1, %3, 0 offen sc0 sc1 lds\n\t"
                         "buffer_load_dword %2, %3, 0 offen offset:256 sc0 sc1 lds"
                         ::"s"((u32)(size_t)samp), "v"(vfa), "v"(vfb), "s"(rws) : "memory");
          }
        }
        SB(); dma(0); rd(2); rd(3); SB(); mm(o, 3); SB(); rd(5); SB();
        mm(o, 4); SB(); rd(6); SB(); mm(o, 5); SB(); rd(7); SB(); mm(o, 6); SB(); dma(1); rd(8); rd(9); SB(); mm(o, 7); SB(); rd(12); SB();
        mm(o, 8); SB(); rd(10); rd(11); SB(); mm(o, 9); SB(); rd(13); SB(); mm(o, 10); SB(); dma(2); rd(14); SB(); mm(o, 11); SB(); rd(15); SB();
        mm(o, 12); SB(); mm(o, 13); SB(); dma(3); SB(); mm(o, 14); SB(); prep(s + 4); SB(); mm(o, 15); SB();
      }
    }
    WAIT_VM(0);
    if (limit == 32) break;
    if (stamp) { if (limit == 8) { STAMP(2); } else { STAMP(11); } }
    const u32 r = wait_words(c, flag_a, flag_b, 48, limit == 8 ? need : ALL48, nwait++);
    if (stamp) { if (limit == 8) { STAMP(3); } else { STAMP(10); } }
    bad |= r & 1;
    limit = (r & 2) ? 32 : 24;
  }
#pragma unroll
  for (int i = 0; i < 16; ++i) mm(1, i);
  BARRIER();
  return bad;
}

__device__ __forceinline__ u32 pack2(float a, float b) {
  return __builtin_bit_cast(u32, __builtin_convertvector((f32x2){a, b}, bf16x2));
}
__device__ __forceinline__ float bf_lo(u32 v) { return __builtin_bit_cast(float, v << 16); }
__device__ __forceinline__ float bf_hi(u32 v) { return __builtin_bit_cast(float, v & 0xffff0000u); }

__global__ void __launch_bounds__(NT, 2) gemm_rs(Params p) {
  __shared__ __attribute__((aligned(1024))) char smem[LDS_BYTES];
  Ctx c;
  c.lds = (LAS char*)smem;
  c.lane = threadIdx.x & 63;
  c.wave = rfl(threadIdx.x >> 6);
  const u32 rank = (u32)p.rank & (NR - 1);
  c.wsl = p.wsl;
  STAMP(0);
  u32 role = (__builtin_amdgcn_s_getreg((3 << 11) | 20) & 7u) * 32u + (blockIdx.x >> 3);
  gu32* cnt = (gu32*)(c.wsl + OFF_CNT);
  gu32* claim = (gu32*)(c.wsl + OFF_CLAIM);
  LAS u32* sync = (LAS u32*)(c.lds + LDS_CTL) + 500;
  u32 nv = 0, ov = 0;
  if (threadIdx.x == NT - 64) {
    nv = __hip_atomic_load(cnt + blockIdx.x, __ATOMIC_RELAXED, __HIP_MEMORY_SCOPE_AGENT);
    ov = __hip_atomic_fetch_add(claim + role * 32, 1u, __ATOMIC_RELAXED, __HIP_MEMORY_SCOPE_AGENT);
  }
  c.t_start = (u32)__builtin_amdgcn_s_memrealtime();
  u32 grp, u, j, ct, sidx;
  bool isG, isO, sender;
  Fill fill;
  for (bool retry = false;; retry = true) {
    grp = (X2G >> ((role >> 5) * 4)) & 7; u = role & 31;
    isG = grp & 1; isO = (grp & 3) == 2;
    sender = isG ? u >= 24 : (isO && (u & 3) == 3);
    sidx = (isG ? u : u >> 2) & 7;
    if (sender && c.wave >= 4 && c.wave < 7) first_fill(c, p, (rank ^ grp) & (NR - 1), sidx, fill);
    if (retry) break;
    ((LAS u32*)(c.lds + LDS_CTL))[threadIdx.x] = 0;
    if (c.wave == 7) {
      const u32 n = rfl(nv);
      const bool lost = rfl(ov) != n;
      u32 r = role;
      if (lost) {
        if (c.lane == 0) (void)__hip_atomic_fetch_sub(claim + role * 32, 1u, __ATOMIC_RELAXED, __HIP_MEMORY_SCOPE_AGENT);
        WAIT_VM(0);
      }
      if (c.lane == 0) __hip_atomic_store(cnt + blockIdx.x, n + 1, __ATOMIC_RELAXED, __HIP_MEMORY_SCOPE_AGENT);
      if (lost) {
        const u64 t0 = __builtin_amdgcn_s_memrealtime();
#pragma unroll 1
        for (u32 i = 0; i < NWG;) {
          const u32 v = __hip_atomic_load(cnt + i + c.lane, __ATOMIC_RELAXED, __HIP_MEMORY_SCOPE_AGENT);
          if (__all(v > n) || __builtin_amdgcn_s_memrealtime() - t0 > DEADLINE) i += 64;
        }
#pragma unroll 1
        for (u32 i = 0; i < NWG; ++i) {
          u32 expect = n;
          if (c.lane == 0) (void)__hip_atomic_compare_exchange_strong(claim + i * 32, &expect, n + 1, __ATOMIC_RELAXED, __ATOMIC_RELAXED, __HIP_MEMORY_SCOPE_AGENT);
          if (rfl(expect) == n) { r = i; break; }
        }
      }
      if (c.lane == 0) { sync[0] = n + 1; sync[1] = r; }
    }
    BARRIER();
    c.call = rfl(sync[0]);
    const u32 r = rfl(sync[1]);
    if (r == role) break;
    role = r;
    WAIT_VM(0);
  }
  if (isG) { j = u & 1; ct = u >> 1; }
  else { j = u & 3; ct = (u >> 2) * 2 + (grp >> 2); }
  const int ntiles = sender ? 0 : ((isG ? (u < 16 && j == 0) : (!isO && u < 8)) ? 2 : 1);

  if (sender) {
    link_engine(c, p, grp, sidx, fill);
  } else {
    const u32 hold = (isG || isO) ? DELAY_GO : (ct >= 4 ? DELAY_P : 0);
    while ((u32)__builtin_amdgcn_s_memrealtime() - c.t_start < hold) __builtin_amdgcn_s_sleep(4);
  }
  __builtin_amdgcn_s_waitcnt(0x0F70);
  const u32 par_off = OFF_STAGE + (c.call & 1) * PAR_BYTES;
  STAMP(1);

  const auto rws = RSRC(c.wsl, WS_NEED);
  const u32 NANS = 0x7fc07fc0u;
  bool bad = false;
  u32x4 y[4];
  const u32 vy = c.wave * 4096 + c.lane * 16;
  u32 tj = j, tct = ct, nwait = 0;
  bool tO = isO && !sender;
  f32x4 acc[4][2];
  constexpr u64 ORD_LIN = 0xba98765432103210ull, ORD_REL = 0x65217430ba983210ull;
  constexpr u64 ALL48 = (1ull << 48) - 1, NEED_O = (1ull << 32) - 1, NEED_REL = 0xfffff00ff00full;
#pragma unroll 1
  for (int tile = 0; tile < ntiles;) {
    const u32 rt = (isG ? 3 + grp : (tO ? 12 : 0)) + tj;
    const bool rel = !tO && tct >= 16 - NREL;
    bad |= gemm_tile(c, p, acc, rel ? ORD_REL : ORD_LIN, (tO && NREL > 0) ? NEED_O : (rel ? NEED_REL : ALL48),
                     row0_of(rank, rt) * 1024, (rank & 1) * 4096 + tct * 256, par_off + rt * 6 * CHUNK, par_off + (16 + tct) * 6 * CHUNK,
                     OFF_IFLAG + rt * 256, OFF_IFLAG + (16 + tct) * 256, nwait, tile == 0);
    {
      if (tile == 0) { STAMP(4); }
      const bool odd = (c.lane >> 4) & 1;
#pragma unroll
      for (int m = 0; m < 4; ++m) {
        const f32x4 lo = odd ? acc[m][1] : acc[m][0], hi = odd ? acc[m][0] : acc[m][1];
        acc[m][0] = lo; acc[m][1] = hi;
        y[m].x = pack2(lo[0], lo[1]); y[m].y = pack2(lo[2], lo[3]);
        y[m].z = pack2(hi[0], hi[1]); y[m].w = pack2(hi[2], hi[3]);
        if (bad) y[m] = (u32x4){NANS, NANS, NANS, NANS};
      }
      if (!tO) {
        const u32 gp = gpos(tj, tct), gt = (grp >> 1) * 32 + gp;
        const bool fwd = isG && gp < 2 * NFWD, direct = isG && !fwd;
        if (direct) (void)wait_words(c, OFF_GATE + grp * 128, OFF_GATE + grp * 128, NSEND, ALL48, nwait++);
        STAMP(7);
        char* dst = p.ws[(rank ^ (fwd ? 0u : (isG ? grp : 4u))) & (NR - 1)];
        const auto rd = RSRC(dst, WS_NEED);
        const u32 yo = vy + (fwd ? OFF_LY + gt * 32768 : par_off + YS_OFF + (isG ? 64 + gt : j * 16 + ct) * 32768);
#pragma unroll
        for (int m = 0; m < 4; ++m) {
          __builtin_amdgcn_raw_buffer_store_b128(y[m], rd, yo + m * 1024, 0, SYS);
          if (m < 3 && direct) WAIT_VM(0);
        }
        WAIT_VM(0);
        const u32 fo = direct ? OFF_GFLAG + gt * 128 + (c.lane >> 1) * 64 + (c.lane & 1) * 4 : (fwd ? OFF_LYF + gt * 4 : OFF_YFLAG + (j * 16 + ct) * 128);
        if (last_wave(c, tile) && c.lane < (direct ? 4u : 1u)) put_flag(dst + fo, c.call);
        if (tile == 0) { STAMP(6); }
      }
      if (++tile < ntiles) {
        if (isG) { tj = (u >> 1) & 1; tct = 12 + (u >> 2); }
        else { tO = true; tj = 3; tct = u * 2 + (grp >> 2); }
      }
    }
  }
  STAMP(5);

  const auto rout = RSRC(p.out, IN_BYTES);

  if (tO) {
    const u32 yt = rfl(tj * 16 + tct);
    bad |= wait_words(c, OFF_YFLAG + yt * 128, OFF_YFLAG + yt * 128, 1, ALL48, nwait) & 1;
    STAMP(8);
#pragma unroll
    for (int m = 0; m < 4; ++m) y[m] = __builtin_amdgcn_raw_buffer_load_b128(rws, vy + m * 1024, par_off + YS_OFF + yt * 32768, SYS);
    const u32 vo = (tj * 128 + (c.wave >> 2) * 64 + (c.lane & 15)) * 8192 + (rank & 1) * 4096 + tct * 256 + (c.wave & 3) * 64 + (c.lane >> 4) * 16;
#pragma unroll
    for (int m = 0; m < 4; ++m) {
      u32x4 o;
      o.x = pack2(acc[m][0][0] + bf_lo(y[m].x), acc[m][0][1] + bf_hi(y[m].x));
      o.y = pack2(acc[m][0][2] + bf_lo(y[m].y), acc[m][0][3] + bf_hi(y[m].y));
      o.z = pack2(acc[m][1][0] + bf_lo(y[m].z), acc[m][1][1] + bf_hi(y[m].z));
      o.w = pack2(acc[m][1][2] + bf_lo(y[m].w), acc[m][1][3] + bf_hi(y[m].w));
      if (bad) o = (u32x4){NANS, NANS, NANS, NANS};
      __builtin_amdgcn_raw_buffer_store_b128(o, rout, vo + m * 16 * 8192, 0, SYS);
    }
  } else if (isG) {
    const u32 gt = ((grp >> 1) & 1) * 32 + gpos(j, ct);
    const u32 ya = 64 + gt, yb = ya + 64;
    const u32 fo = OFF_GFLAG + (gt * 2 + (grp >> 2)) * 64;
    bad |= wait_words(c, fo, fo + 128 * 64, 2, ALL48, nwait) & 1;
    STAMP(8);
    const u32 pw = (grp >> 2) * 4 + (c.wave >> 1), m0 = (c.wave & 1) * 2;
    const u32 vr = pw * 4096 + m0 * 1024 + c.lane * 16;
    u32x4 ra[2], rb[2];
#pragma unroll
    for (int m = 0; m < 2; ++m) {
      ra[m] = __builtin_amdgcn_raw_buffer_load_b128(rws, vr + m * 1024, par_off + YS_OFF + ya * 32768, SYS);
      rb[m] = __builtin_amdgcn_raw_buffer_load_b128(rws, vr + m * 1024, par_off + YS_OFF + yb * 32768, SYS);
    }
    const u32 vo = (((grp >> 1) & 1) * 256 + j * 128 + (pw >> 2) * 64 + m0 * 16 + (c.lane & 15)) * 8192 +
                   ((rank & 1) ^ 1) * 4096 + ct * 256 + (pw & 3) * 64 + (c.lane >> 4) * 16;
#pragma unroll
    for (int m = 0; m < 2; ++m) {
      u32x4 o;
      o.x = pack2(bf_lo(ra[m].x) + bf_lo(rb[m].x), bf_hi(ra[m].x) + bf_hi(rb[m].x));
      o.y = pack2(bf_lo(ra[m].y) + bf_lo(rb[m].y), bf_hi(ra[m].y) + bf_hi(rb[m].y));
      o.z = pack2(bf_lo(ra[m].z) + bf_lo(rb[m].z), bf_hi(ra[m].z) + bf_hi(rb[m].z));
      o.w = pack2(bf_lo(ra[m].w) + bf_lo(rb[m].w), bf_hi(ra[m].w) + bf_hi(rb[m].w));
      if (bad) o = (u32x4){NANS, NANS, NANS, NANS};
      __builtin_amdgcn_raw_buffer_store_b128(o, rout, vo + m * 16 * 8192, 0, SYS);
    }
  }
}

}

extern "C" void kernel_launch(int rank, int nranks, ncclComm_t comm, hipStream_t stream,
                              void* const* d_in, const int* in_sizes, int n_in,
                              void* d_out, int out_size,
                              const int* in_dtypes, int out_dtype,
                              void* const* d_out_all,
                              void* const* d_ws_all, size_t ws_bytes) {
  (void)comm; (void)in_dtypes; (void)out_dtype; (void)d_out_all;
  if (nranks != NR || (unsigned)rank >= (unsigned)NR || n_in != 2 || ws_bytes < WS_NEED ||
      in_sizes[0] != 4096 * 512 || in_sizes[1] != 512 * 4096 || out_size != 512 * 4096) {
    fprintf(stderr, "gemm_rs: unexpected problem shape\n");
    return;
  }
  Params p;
  p.x = (const char*)d_in[0];
  p.w = (const char*)d_in[1];
  p.out = (char*)d_out;
  p.wsl = (char*)d_ws_all[rank];
  p.rank = 0;
  for (int i = 0; i < NR; ++i) {
    const int r = (L2P >> (4 * i)) & 7;
    p.ws[i] = (char*)d_ws_all[r];
    if (r == rank) p.rank = i;
  }
  hipLaunchKernelGGL(gemm_rs, dim3(NWG), dim3(NT), 0, stream, p);
}
```

```cpp
#include <hip/hip_runtime.h>
#include <rccl/rccl.h>
#include <cstdio>

namespace {

typedef unsigned u32;
typedef unsigned long long u64;
typedef __attribute__((ext_vector_type(4))) unsigned u32x4;
typedef __attribute__((ext_vector_type(4))) float f32x4;
typedef __attribute__((ext_vector_type(2))) float f32x2;
typedef __attribute__((ext_vector_type(2))) __bf16 bf16x2;
typedef __attribute__((ext_vector_type(8))) __bf16 bf16x8;
typedef __attribute__((ext_vector_type(4))) short s16x4;
#define LAS __attribute__((address_space(3)))
#define GAS __attribute__((address_space(1)))
typedef GAS u32 gu32;

constexpr int NR = 8;
constexpr int NWG = 256;
constexpr int NT = 512;
constexpr u32 MiB = 1u << 20;
constexpr u32 CHUNK = 65536;
constexpr int NSEND = 8;
constexpr u32 PIECE = CHUNK / NSEND;
constexpr u32 IN_BYTES = 4 * MiB;

constexpr u32 OFF_CNT = 0;
constexpr u32 OFF_CLAIM = 131072;
constexpr u32 OFF_TMO = 4096;
constexpr u32 OFF_LYF = 8192;
constexpr u32 OFF_GATE = 12288;
constexpr u32 OFF_IFLAG = 16384;
constexpr u32 OFF_YFLAG = 32768;
constexpr u32 OFF_GFLAG = 40960;
constexpr u32 OFF_STAMP = 65536;
constexpr u32 OFF_STAGE = 1 * MiB;
constexpr u32 YS_OFF = 12 * MiB;
constexpr u32 PAR_BYTES = 18 * MiB;
constexpr u32 OFF_LY = OFF_STAGE + 2 * PAR_BYTES;
constexpr u32 WS_NEED = OFF_LY + 4 * MiB;

constexpr u64 DEADLINE = 40000000ull;

constexpr u32 STAGE_B = 32768;
constexpr u32 LDS_CTL = 4 * STAGE_B;
constexpr u32 LDS_BYTES = LDS_CTL + 3072;

enum { K_X = 0, K_W = 1, K_Y = 2, K_H = 3, K_R = 4 };
#define X(i) (K_X * 16 + (i))
#define W(i) (K_W * 16 + (i))
#define Y(i) (K_Y * 16 + (i))
#define H(i) (K_H * 16 + (i))
#define R(i) (K_R * 16 + (i))
constexpr int NFWD = 4;
#define SEQ_V 3
constexpr int NREL = SEQ_V >= 3 ? 8 : 6;
constexpr int NPOS_G = 32 - NREL / 2 + NFWD, NPOS_O = 32 + NREL;
constexpr unsigned char SEQ_G[48] = {
    X(0), W(0), W(1), X(1), W(2), X(2), X(3), W(3), X(4), W(4), X(6), W(5), X(8), W(6), X(10), W(7),
    X(5), H(8), X(7), X(9), H(10), X(11), H(12), H(14), X(12), X(13), X(14), X(15), Y(0), Y(1), Y(2), Y(3)};
constexpr unsigned char SEQ_O[48] = {
    X(4), X(6), X(8), X(0), W(0), W(1), X(1), W(2), X(2), X(3), W(3), X(10), W(4), X(5), W(5), X(7),
    W(6), X(9), W(7), X(11), W(8), W(9), R(8), W(10), R(9), W(11), R(10), W(12), R(11), W(13), R(12), W(14),
    W(15), R(13), R(14), R(15), X(12), X(13), X(14), X(15)};
constexpr int seq_pos(const unsigned char* q, int e) { for (int i = 0; i < 48; ++i) if (q[i] == e) return i; return 99; }
constexpr int cmax(int a, int b) { return a > b ? a : b; }
constexpr int cmin(int a, int b) { return a < b ? a : b; }
constexpr int w_pos(const unsigned char* q, int ct) { return cmin(seq_pos(q, W(ct)), seq_pos(q, H(ct & ~1))); }
struct FirstPos { unsigned char v[256]; };
constexpr FirstPos first_pos() {
  FirstPos f{};
  for (int rt = 0; rt < 16; ++rt)
    for (int ct = 0; ct < 16; ++ct)
      f.v[rt * 16 + ct] = (unsigned char)cmin(cmax(seq_pos(SEQ_O, X(rt)), w_pos(SEQ_O, ct)), cmax(seq_pos(SEQ_G, X(rt)), w_pos(SEQ_G, ct)));
  return f;
}
__constant__ const FirstPos FIRST_POS = first_pos();
constexpr u64 even_first(int h) {
  u64 r = 0;
  for (int t = 0; t < 4; ++t)
    for (int ct = 0; ct < 16 - NREL; ++ct) {
      const int rt = 4 + 4 * h + t;
      if (cmax(seq_pos(SEQ_O, X(rt)), seq_pos(SEQ_O, W(ct))) < cmax(seq_pos(SEQ_G, X(rt)), seq_pos(SEQ_G, W(ct)))) r |= 1ull << (16 * t + ct);
    }
  return r;
}
#undef X
#undef W
#undef Y
#undef H
#undef R
constexpr u64 seq_pack(const unsigned char* s, int from) { u64 r = 0; for (int t = 0; t < 8; ++t) r |= (u64)s[from + t] << (8 * t); return r; }

#define GATE_POS (NPOS_G - 4)
#define PACE 112
#define KV 1
#define EARLY_M 2
#define LEAD 650
#define G_LATEST 100000
#define T_SECOND 0
#define O_EARLY 1000
#define HOLD 35
#define PACE_T0 120
#define DELAY_GO 600
#define DELAY_P 400

constexpr u32 L2P = 0x53764210u;
constexpr u32 X2G = 0x75314602u;

struct Params {
  const char* x;
  const char* w;
  char* out;
  char* wsl;
  char* ws[NR];
  int rank;
};

#define RSRC(p, n) __builtin_amdgcn_make_buffer_rsrc((void*)(p), 0, (n), 0x00020000)
#define WAIT_VM(n) asm volatile("s_waitcnt vmcnt(%0)" ::"n"(n) : "memory")
#define BARRIER() do { asm volatile("s_waitcnt lgkmcnt(0)" ::: "memory"); __builtin_amdgcn_s_barrier(); asm volatile("" ::: "memory"); } while (0)
#define SB() __builtin_amdgcn_sched_barrier(0)
constexpr int SYS = 17;

__device__ __forceinline__ u32 rfl(u32 v) { return __builtin_amdgcn_readfirstlane(v); }
__device__ __forceinline__ __amdgpu_buffer_rsrc_t srsrc(const char* p, u32 n) {
  return RSRC(((u64)rfl((u32)((u64)p >> 32)) << 32) | rfl((u32)(u64)p), rfl(n));
}
__device__ __forceinline__ u32 rdl(u32 v, int lane) { return __builtin_amdgcn_readlane(v, lane); }
__device__ __forceinline__ void put_flag(char* addr, u32 v) { __hip_atomic_store((gu32*)addr, v, __ATOMIC_RELAXED, __HIP_MEMORY_SCOPE_SYSTEM); }
__device__ __forceinline__ u32 get_flag(const char* addr) { return __hip_atomic_load((gu32*)addr, __ATOMIC_RELAXED, __HIP_MEMORY_SCOPE_SYSTEM); }
__device__ __forceinline__ u32 lds_get(LAS u32* w) { return rfl(__hip_atomic_load(w, __ATOMIC_RELAXED, __HIP_MEMORY_SCOPE_WORKGROUP)); }
__device__ __forceinline__ void lds_inc(LAS u32* w, u32 lane) {
  if (lane == 0) __hip_atomic_fetch_add(w, 1u, __ATOMIC_RELAXED, __HIP_MEMORY_SCOPE_WORKGROUP);
}

__device__ __forceinline__ u32 row0_of(u32 q, u32 rt) {
  const bool g = rt - 4u < 8u;
  const u32 d = g ? ((rt - 4u) | 1u) : (rt < 4u ? 4u : 0u);
  const u32 t = g ? (d & 2u) + (rt & 1u) : (rt & 3u);
  return ((L2P >> ((q ^ d) * 4u)) & 7u) * 512u + t * 128u;
}
__device__ __forceinline__ u32 slot_of(u32 d) { return (d & 1u) ? d >> 1 : 4u + (d >> 2); }
__device__ __forceinline__ u32 gpos(u32 j, u32 ct) { return j * 16u + ct; }

struct Ctx {
  LAS char* lds;
  char* wsl;
  u32 call, lane, wave, t_start;
};
#define STAMP(i)

__device__ __forceinline__ u32 wait_words(const Ctx& c, u32 o0, u32 o1, u32 n, u32 need, u32 slot) {
  LAS u32* res = (LAS u32*)(c.lds + LDS_CTL + 256) + slot;
  if (c.wave == 0) {
    const char* p0 = c.wsl + (c.lane < n ? o0 : OFF_TMO);
    const char* p1 = c.wsl + (c.lane < n ? o1 : OFF_TMO);
    const u64 t0 = __builtin_amdgcn_s_memrealtime();
    u32 r;
    for (;;) {
      const u32 v0 = get_flag(p0), v1 = get_flag(p1);
      if (rdl(v1, 63) == c.call) { r = 129; break; }
      const u64 miss = __ballot(c.lane < n && (v0 < c.call || v1 < c.call));
      r = miss ? 2u * (u32)__builtin_ctzll(miss) : 128u;
      if (r >= 2 * need) break;
      if (__builtin_amdgcn_s_memrealtime() - t0 > DEADLINE) {
        if (c.lane == 0) put_flag(c.wsl + OFF_TMO, c.call);
        r = 129;
        break;
      }
      __builtin_amdgcn_s_sleep(2);
    }
    if (c.lane == 0) *res = r;
  }
  BARRIER();
  return rfl(*res);
}

__device__ __forceinline__ bool last_wave(const Ctx& c, int idx) {
  LAS u32* cnt = (LAS u32*)(c.lds + LDS_CTL) + idx;
  u32 old = 0;
  if (c.lane == 0) old = __hip_atomic_fetch_add(cnt, 1u, __ATOMIC_RELAXED, __HIP_MEMORY_SCOPE_WORKGROUP);
  return rfl(old) == 7u;
}

__device__ __forceinline__ u32 kof_of(u32 q, u32 st) { return (st < 2u ? (q >> 2) : 1u + (((q >> 2) ^ (q >> 1)) & 1u)) * 128u + st * 64u; }

struct Fill { u32 vx[4], vw[4]; };
__device__ __forceinline__ void fill_slot(const Ctx& c, int pos, const u32 (&vo)[4], __amdgpu_buffer_rsrc_t rs, u32 so, u32 sstep) {
  const u32 lb = (u32)(size_t)c.lds + (u32)(pos & 7) * PIECE;
#pragma unroll
  for (int g = 0; g < 2; ++g)
    asm volatile("s_mov_b32 m0, %0\n\ts_nop 3\n\t"
                 "buffer_load_dwordx4 %1, %5, %6 offen sc0 sc1 lds\n\t"
                 "buffer_load_dwordx4 %2, %5, %6 offen offset:1024 sc0 sc1 lds\n\t"
                 "buffer_load_dwordx4 %3, %5, %6 offen offset:2048 sc0 sc1 lds\n\t"
                 "buffer_load_dwordx4 %4, %5, %6 offen offset:3072 sc0 sc1 lds"
                 ::"s"(rfl(lb + g * 4096u)), "v"(vo[0]), "v"(vo[1]), "v"(vo[2]), "v"(vo[3]), "s"(rs), "s"(rfl(so + g * sstep)) : "memory");
}
__device__ __forceinline__ void first_fill(const Ctx& c, const Params& p, u32 q, bool odd, u32 sidx, Fill& f) {
  static_assert(((seq_pack(SEQ_G, 0) | seq_pack(SEQ_O, 0)) & 0xe0e0e0) == 0, "first positions");
  const u32 l = c.lane, pos = c.wave - 4, pc = (sidx + pos) & 7u, kof = kof_of(q, pc >> 1);
  const u32 ent = ((odd ? (u32)seq_pack(SEQ_G, 0) : (u32)seq_pack(SEQ_O, 0)) >> (8 * pos)) & 31u, id = ent & 15u;
#pragma unroll
  for (int i = 0; i < 4; ++i) {
    const u32 xr = i * 8 + (l >> 3), wk = i * 4 + (l >> 4);
    f.vx[i] = xr * 1024 + (((l & 7) ^ ((xr >> 1) & 7)) << 4) - i * 1024;
    f.vw[i] = wk * 8192 + (q & 1) * 4096 + (((l & 15) ^ (((wk & 3) << 2) | ((wk >> 3) & 1))) << 4) - i * 1024;
  }
  u32 mx = rfl(ent - 16);
  asm("s_ashr_i32 %0, %0, 31" : "+s"(mx)::"scc");
  u32 vo[4];
#pragma unroll
  for (int i = 0; i < 4; ++i) vo[i] = (f.vx[i] & mx) | (f.vw[i] & ~mx);
  const u32 sx = row0_of(q, id) * 1024u + kof * 2u + (pc & 1u) * 65536u, sw = (kof + (pc & 1u) * 32u) * 8192u + id * 256u;
  fill_slot(c, pos, vo, srsrc(mx ? p.x : p.w, IN_BYTES), (sx & mx) | (sw & ~mx),
            (mx & 32768u) | (~mx & 131072u));
}

__device__ __forceinline__ void link_engine(Ctx& c, const Params& p, u32 d, u32 sidx, const Fill& f) {
  const u32 q = ((u32)p.rank ^ d) & (NR - 1);
  char* dst = p.ws[q];
  const bool odd = d & 1;
  const int npos = odd ? NPOS_G : NPOS_O;
  LAS u32* done = (LAS u32*)(c.lds + LDS_CTL + 512);
  LAS u32* full = (LAS u32*)(c.lds + LDS_CTL + 768);
  LAS u32* freed = (LAS u32*)(c.lds + LDS_CTL + 1024);
  LAS u32* dead = (LAS u32*)(c.lds + LDS_CTL + 320);
  const u32 l = c.lane;
  u64 tab = odd ? seq_pack(SEQ_G, 0) : seq_pack(SEQ_O, 0);
  if (l >= 8) tab = odd ? seq_pack(SEQ_G, 8) : seq_pack(SEQ_O, 8);
  if (l >= 16) tab = odd ? seq_pack(SEQ_G, 16) : seq_pack(SEQ_O, 16);
  if (l >= 24) tab = odd ? seq_pack(SEQ_G, 24) : seq_pack(SEQ_O, 24);
  if (l >= 32) tab = odd ? seq_pack(SEQ_G, 32) : seq_pack(SEQ_O, 32);
  if (l >= 40) tab = odd ? seq_pack(SEQ_G, 40) : seq_pack(SEQ_O, 40);
  const u32 ent = (u32)(tab >> (8 * (l & 7))) & 255u, kind = ent >> 4;
  const u32 ts = (sidx + l) & 7u;
  const bool isy = kind == K_Y, isr = kind == K_R, split = kind >= K_H;
  const u32 id = (ent & 15u) + (kind == K_H ? ts >> 2 : 0u);
  const u32 pc = split ? ((isr ? d : d << 1) & 4u) + (ts & 3u) : ts;
  const u32 dorg = isr ? (d >> 1) + (ts & 4u) : 0u;
  const u32 cid = (kind == K_X ? 0u : 16u) + id, yk = (d >> 1) * 16 + id;
  const u32 par = OFF_STAGE + (c.call & 1) * PAR_BYTES;
  if (c.wave >= 4 && c.wave < 7) {
    WAIT_VM(0);
    lds_inc(full + (c.wave - 4), l);
    const u32 kof = kof_of(q, pc >> 1);
    const u32 t_src = isy ? OFF_LY + yk * CHUNK + pc * PIECE
                    : isr ? par + (cid * 6 + slot_of(dorg)) * CHUNK + pc * PIECE
                    : kind == K_X ? row0_of(q, id) * 1024u + kof * 2u + (pc & 1u) * 65536u
                                  : (kof + (pc & 1u) * 32u) * 8192u + id * 256u;
    const u32 t_mx = kind == K_X ? ~0u : 0u, t_ml = (isy || isr) ? ~0u : 0u;
    const u32 t_dep = isy ? OFF_LYF + (yk * 2 + (pc >> 2)) * 4 : OFF_IFLAG + cid * 256 + (slot_of(dorg) * 8 + pc) * 4;
#pragma unroll 1
    for (int pos = (int)c.wave - 1; pos < npos; pos += 3) {
      const u32 mx = rdl(t_mx, pos), ml = rdl(t_ml, pos), mw = ~(mx | ml);
      u32 vo[4];
#pragma unroll
      for (int i = 0; i < 4; ++i) vo[i] = (f.vx[i] & mx) | (f.vw[i] & mw) | ((l * 16) & ml);
      if (pos >= 8) while (lds_get(freed + (pos - 8)) < 4u) __builtin_amdgcn_s_sleep(1);
      if (ml) {
        const char* dep = c.wsl + rdl(t_dep, pos);
        const u64 t0 = __builtin_amdgcn_s_memrealtime();
        while (rfl(get_flag(dep)) < c.call) {
          if (lds_get(dead) || __builtin_amdgcn_s_memrealtime() - t0 > 2 * DEADLINE) {
            if (l == 0) {
              put_flag(c.wsl + OFF_TMO, c.call);
              __hip_atomic_store(dead, 1u, __ATOMIC_RELAXED, __HIP_MEMORY_SCOPE_WORKGROUP);
            }
            break;
          }
          __builtin_amdgcn_s_sleep(2);
        }
      }
      fill_slot(c, pos, vo, srsrc(ml ? c.wsl : (mw ? p.w : p.x), ml ? WS_NEED : IN_BYTES), rdl(t_src, pos),
                (mx & 32768u) | (mw & 131072u) | (ml & 4096u));
      WAIT_VM(0);
      lds_inc(full + pos, l);
    }
  } else {
    const u32 oslot = slot_of(d ^ dorg);
    if (c.wave < 4) {
      const u32 tw = (c.wave + l + (l >> 2)) & 3u;
      const u32 t_dst = par + pc * PIECE + tw * 2048u +
                        (isy ? YS_OFF + 2 * MiB + yk * CHUNK : (cid * 6 + oslot) * CHUNK);
      const u32 t_lds = (l & 7u) * PIECE + tw * 2048u;
      const auto rd = RSRC(dst, WS_NEED);
      int sig = 0;
      const u32 t0 = c.t_start;
      u32 due = 2 * PACE_T0 + (sidx * 4 + c.wave) * PACE / 32;
#pragma unroll 1
      for (int pos = 0; pos < npos; ++pos) {
        if (lds_get(full + pos) == 0u) {
          WAIT_VM(0);
          for (; sig < pos; ++sig) lds_inc(done + sig, l);
          while (lds_get(full + pos) == 0u) __builtin_amdgcn_s_sleep(1);
        }
        asm volatile("" ::: "memory");
        LAS char* src = c.lds + rdl(t_lds, pos) + l * 16;
        u32x4 v0 = *(LAS u32x4*)src, v1 = *(LAS u32x4*)(src + 1024);
        const u32 vo = l * 16 + rdl(t_dst, pos);
        asm volatile("s_waitcnt lgkmcnt(0)" : "+v"(v0), "+v"(v1)::"memory");
        lds_inc(freed + pos, l);
        WAIT_VM(1);
        u32 now = ((u32)__builtin_amdgcn_s_memrealtime() - t0) * 2, go = max(due, pos ? now + 2 * HOLD : 0u);
        while (now < go) { __builtin_amdgcn_s_sleep(1); now = ((u32)__builtin_amdgcn_s_memrealtime() - t0) * 2; }
        __builtin_amdgcn_raw_buffer_store_b128(v0, rd, vo, 0, SYS);
        WAIT_VM(1);
        now = ((u32)__builtin_amdgcn_s_memrealtime() - t0) * 2; go = max(due + PACE, pos ? now + 2 * HOLD : 0u);
        if (sig < pos) { lds_inc(done + sig, l); ++sig; }
        while (now < go) { __builtin_amdgcn_s_sleep(1); now = ((u32)__builtin_amdgcn_s_memrealtime() - t0) * 2; }
        __builtin_amdgcn_raw_buffer_store_b128(v1, rd, vo + 1024, 0, SYS);
        asm volatile("" ::: "memory");
        due += 2 * PACE;
      }
      WAIT_VM(0);
      lds_inc(done + sig, l);
    } else {
      const u32 t_flag = isy ? OFF_GFLAG + (yk * 4 + (pc >> 1)) * 64 + (pc & 1u) * 4 : OFF_IFLAG + cid * 256 + (oslot * 8 + pc) * 4;
#pragma unroll 1
      for (int pos = 0; pos < npos; ++pos) {
        while (lds_get(done + pos) < 4u) __builtin_amdgcn_s_sleep(1);
        if (l == (u32)pos && !lds_get(dead)) put_flag(dst + t_flag, c.call);
        if (pos == GATE_POS && l == 0) put_flag(c.wsl + OFF_GATE + d * 128 + sidx * 4, c.call);
      }
    }
  }
  BARRIER();
}

__device__ __forceinline__ bool gemm_tile(Ctx& c, const Params& p, f32x4 (&acc)[4][2], u64 order, u32 own_a, u32 own_b,
                                          u32 stg_a, u32 stg_b, u32 flag_a, u32 flag_b, u32& nwait, bool stamp) {
  order = ((u64)rfl((u32)(order >> 32)) << 32) | rfl((u32)order);
  const u32 m0a = (u32)(size_t)c.lds + c.wave * 2048;
  const u32 wm = c.wave >> 2, wn = c.wave & 3, fr = c.lane & 15, fq = c.lane >> 4;
  const u32 xa0 = (wm * 64 + fr) * 128 + ((fq ^ (fr >> 1)) << 4);
  const u32 tq = (c.lane >> 2) & 3, tp = c.lane & 3;
  const u32 wa0 = 16384 + (fq * 8 + tq) * 256 + (((wn * 4 + tp) ^ ((tq << 2) | (fq & 1))) << 4) + ((tp & 1) << 3);
  LAS u32* samp = (LAS u32*)(c.lds + LDS_CTL + 2048);
  const auto rws = RSRC(c.wsl, WS_NEED);
  const u32 fw = c.lane < 48 ? (((u32)(order >> (16 + (c.lane & ~3u))) & 15u) * 4 + (c.lane & 3)) * 4 : 0u;
  const u32 vfa = flag_a + fw, vfb = flag_b - 256 + fw;

  const u32 xr = c.wave * 16 + (c.lane >> 3), wk = c.wave * 8 + (c.lane >> 4);
  u32 va0 = xr * 1024 + (((c.lane & 7) ^ ((xr >> 1) & 7)) << 4);
  u32 va1 = (xr + 8) * 1024 + (((c.lane & 7) ^ (((xr + 8) >> 1) & 7)) << 4) - 1024;
  u32 vb0 = wk * 8192 + (((c.lane & 15) ^ (((wk & 3) << 2) | ((wk >> 3) & 1))) << 4);
  u32 vb1 = vb0 + 4 * 8192 - 1024;
  auto desc = [](const char* q) { return (u32x4){rfl((u32)(u64)q), rfl((u32)((u64)q >> 32)) & 0xffffu, 0u, 0x00020000u}; };
  u32x4 ra = desc(p.x), rb = desc(p.w);
  u32 nrec = IN_BYTES, sa = rfl(own_a), sb = rfl(own_b), stepa = 128, stepb = 64 * 8192;
  stg_a = rfl(stg_a); stg_b = rfl(stg_b);
  int limit = 8;

  u32 soa, sob, m0v;
  auto prep = [&](int s) {
    const u32 img = ((u32)(order >> ((2 * (s & ~1)) & 63)) & 15u) * 2u + (u32)(s & 1);
    soa = sa + img * stepa; sob = sb + img * stepb;
    m0v = m0a + (u32)(s & 3) * STAGE_B;
    ra.z = rb.z = s < limit ? nrec : 0u;
    asm volatile("" : "+s"(soa), "+s"(sob), "+s"(m0v), "+s"(ra), "+s"(rb));
  };
  auto dma = [&](int i) {
    if (i == 0) asm volatile("s_mov_b32 m0, %0\n\ts_nop 0\n\tbuffer_load_dwordx4 %1, %2, %3 offen sc0 sc1 lds" ::"s"(m0v), "v"(va0), "s"(ra), "s"(soa) : "memory");
    if (i == 1) asm volatile("s_mov_b32 m0, %0\n\ts_nop 0\n\tbuffer_load_dwordx4 %1, %2, %3 offen offset:1024 sc0 sc1 lds" ::"s"(m0v), "v"(va1), "s"(ra), "s"(soa) : "memory");
    if (i == 2) asm volatile("s_add_u32 m0, %0, 0x4000\n\ts_nop 0\n\tbuffer_load_dwordx4 %1, %2, %3 offen sc0 sc1 lds" ::"s"(m0v), "v"(vb0), "s"(rb), "s"(sob) : "memory", "scc");
    if (i == 3) asm volatile("s_add_u32 m0, %0, 0x4000\n\ts_nop 0\n\tbuffer_load_dwordx4 %1, %2, %3 offen offset:1024 sc0 sc1 lds" ::"s"(m0v), "v"(vb1), "s"(rb), "s"(sob) : "memory", "scc");
  };
  bf16x8 X[2][2][4];
  s16x4 W[2][2][2][2];
  auto mm = [&](int o, int ii) {
    const int h = ii >> 3, m = (ii >> 1) & 3, n = ii & 1;
    const bf16x8 wf = __builtin_bit_cast(bf16x8, __builtin_shufflevector(W[o][h][n][0], W[o][h][n][1], 0, 1, 2, 3, 4, 5, 6, 7));
    acc[m][n] = __builtin_amdgcn_mfma_f32_16x16x32_bf16(wf, X[o][h][m], acc[m][n], 0, 0, 0);
  };
#pragma unroll
  for (int m = 0; m < 4; ++m) acc[m][0] = acc[m][1] = (f32x4){0.f, 0.f, 0.f, 0.f};
#pragma unroll
  for (int h = 0; h < 2; ++h) {
#pragma unroll
    for (int m = 0; m < 4; ++m) X[1][h][m] = (bf16x8)(__bf16)0.0f;
#pragma unroll
    for (int n = 0; n < 2; ++n) W[1][h][n][0] = W[1][h][n][1] = (s16x4){0, 0, 0, 0};
  }
  bool bad = false;
  int t2 = 0;
#pragma unroll 1
  for (;;) {
    SB();
    asm volatile("s_nop 4" ::: "memory");
    SB();
#pragma unroll
    for (int s = 0; s < 3; ++s) {
      prep(2 * t2 + s);
#pragma unroll
      for (int i = 0; i < 4; ++i) dma(i);
    }
    prep(2 * t2 + 3);
#pragma unroll 1
    for (; 2 * t2 < limit; ++t2) {
#pragma unroll
      for (int b = 0; b < 2; ++b) {
        const int s = 2 * t2 + b, o = b ^ 1;
#pragma unroll
        for (int m = 0; m < 4; ++m) asm volatile("" : "+a"(acc[m][0]), "+a"(acc[m][1]));
        asm volatile("s_waitcnt lgkmcnt(0)" ::: "memory"); SB();
        mm(o, 0); SB();
        WAIT_VM(8);
        BARRIER();
        SB();
        LAS char* st = c.lds + (u32)(s & 3) * STAGE_B;
        auto rd = [&](int ii) {
          const int h = ii >> 3, i = ii & 7;
          if (i < 4) W[b][h][i >> 1][i & 1] = __builtin_amdgcn_ds_read_tr16_b64_v4i16((LAS s16x4*)(st + (wa0 ^ (u32)((i >> 1) << 3)) + h * 8192 + (i & 1) * 1024));
          else X[b][h][i - 4] = *(LAS bf16x8*)(st + (xa0 ^ (u32)(h << 6)) + (i - 4) * 2048);
        };
        rd(0); rd(1); SB(); mm(o, 1); SB(); rd(4); SB(); mm(o, 2); SB();
        if (b == 1 && __builtin_expect(limit - s <= 5 && limit < 32, 0)) {
          const int ahead = limit - s;
          bool sample = ahead == 5;
          if (ahead < 5) {
            LAS u32* sp = samp + (ahead == 1 ? 128 : 0);
            const u32 f0 = sp[c.lane], f1 = sp[64 + c.lane];
            const u64 miss = __ballot((f0 < f1 ? f0 : f1) < c.call) | (1ull << 48);
            const int nl = (int)rfl(8u + ((u32)__builtin_ctzll(miss) >> 4) * 8u);
            if (s == 5) {
              va0 = va1 = vb0 = vb1 = c.wave * 2048 + c.lane * 16;
              ra = rb = desc(c.wsl); nrec = WS_NEED; sa = stg_a; sb = stg_b; stepa = stepb = 16384;
            }
            sample = ahead == 3 && nl == limit;
            const bool late = ahead == 1 && nl > limit;
            limit = nl;
            asm volatile("s_nop 4" ::: "memory");
            if (late) {
#pragma unroll 1
              for (int k = 1; k < 3; ++k) { prep(s + k); dma(0); dma(1); dma(2); dma(3); }
            }
            prep(s + 3);
          }
          if (sample && c.wave == 0) {
            asm volatile("s_mov_b32 m0, %0\n\ts_nop 3\n\tbuffer_load_dword %1, %3, 0 offen sc0 sc1 lds\n\t"
                         "buffer_load_dword %2, %3, 0 offen offset:256 sc0 sc1 lds"
                         ::"s"(rfl((u32)(size_t)samp + (ahead == 3 ? 512u : 0u))), "v"(vfa), "v"(vfb), "s"(rws) : "memory");
          }
        }
        SB(); dma(0); rd(2); rd(3); SB(); mm(o, 3); SB(); rd(5); SB();
        mm(o, 4); SB(); rd(6); SB(); mm(o, 5); SB(); rd(7); SB(); mm(o, 6); SB(); dma(1); rd(8); rd(9); SB(); mm(o, 7); SB(); rd(12); SB();
        mm(o, 8); SB(); rd(10); rd(11); SB(); mm(o, 9); SB(); rd(13); SB(); mm(o, 10); SB(); dma(2); rd(14); SB(); mm(o, 11); SB(); rd(15); SB();
        mm(o, 12); SB(); mm(o, 13); SB(); dma(3); SB(); mm(o, 14); SB(); prep(s + 4); SB(); mm(o, 15); SB();
      }
    }
    WAIT_VM(0);
    if (limit == 32) break;
    if (stamp) { if (limit == 8) { STAMP(2); } else { STAMP(11); } }
    const u32 r = wait_words(c, vfa, vfb + 256, 48, 2 * limit, nwait++);
    if (stamp) { if (limit == 8) { STAMP(3); } else { STAMP(10); } }
    bad |= r & 1;
    limit = r >= 96 ? 32 : 8 + (int)(r >> 5) * 8;
  }
#pragma unroll
  for (int i = 0; i < 16; ++i) mm(1, i);
  BARRIER();
  return bad;
}

__device__ __forceinline__ u32 pack2(float a, float b) {
  return __builtin_bit_cast(u32, __builtin_convertvector((f32x2){a, b}, bf16x2));
}
__device__ __forceinline__ float bf_lo(u32 v) { return __builtin_bit_cast(float, v << 16); }
__device__ __forceinline__ float bf_hi(u32 v) { return __builtin_bit_cast(float, v & 0xffff0000u); }

__device__ __forceinline__ u32 nz(u32 v) { return v ? v : 0x80000000u; }
template <int N>
__device__ __forceinline__ bool load_y(const Ctx& c, __amdgpu_buffer_rsrc_t rws, const u32 (&off)[N], u32x4 (&v)[N], bool bad) {
  const u64 t0 = __builtin_amdgcn_s_memrealtime();
  for (;;) {
    u32 lo = ~0u;
#pragma unroll
    for (int i = 0; i < N; ++i) v[i] = __builtin_amdgcn_raw_buffer_load_b128(rws, off[i], 0, SYS);
#pragma unroll
    for (int i = 0; i < N; ++i) lo = min(min(min(lo, v[i].x), min(v[i].y, v[i].z)), v[i].w);
    if (bad || __all(lo != 0u)) return bad;
    if (__builtin_amdgcn_s_memrealtime() - t0 > DEADLINE) {
      if (c.lane == 0) put_flag(c.wsl + OFF_TMO, c.call);
      return true;
    }
    asm volatile("" ::: "memory");
  }
}

__global__ void __launch_bounds__(NT, 2) gemm_rs(Params p) {
  __shared__ __attribute__((aligned(1024))) char smem[LDS_BYTES];
  Ctx c;
  c.lds = (LAS char*)smem;
  c.lane = threadIdx.x & 63;
  c.wave = rfl(threadIdx.x >> 6);
  const u32 rank = (u32)p.rank & (NR - 1);
  c.wsl = p.wsl;
  STAMP(0);
  u32 role = (__builtin_amdgcn_s_getreg((3 << 11) | 20) & 7u) * 32u + (blockIdx.x >> 3);
  gu32* cnt = (gu32*)(c.wsl + OFF_CNT);
  gu32* claim = (gu32*)(c.wsl + OFF_CLAIM);
  LAS u32* sync = (LAS u32*)(c.lds + LDS_CTL) + 500;
  u32 nv = 0, ov = 0;
  if (threadIdx.x == NT - 64) {
    nv = __hip_atomic_load(cnt + blockIdx.x, __ATOMIC_RELAXED, __HIP_MEMORY_SCOPE_AGENT);
    ov = __hip_atomic_fetch_add(claim + role * 32, 1u, __ATOMIC_RELAXED, __HIP_MEMORY_SCOPE_AGENT);
  }
  c.t_start = (u32)__builtin_amdgcn_s_memrealtime();
  u32 grp, u, j, ct, sidx;
  bool isG, isO, sender;
  Fill fill;
  for (bool retry = false;; retry = true) {
    grp = (X2G >> ((role >> 5) * 4)) & 7; u = role & 31;
    isG = grp & 1; isO = (grp & 3) == 2;
    sender = isG ? u >= 24 : (isO && (u & 3) == 3);
    sidx = (isG ? u : u >> 2) & 7;
    if (sender && c.wave >= 4 && c.wave < 7) first_fill(c, p, (rank ^ grp) & (NR - 1), isG, sidx, fill);
    if (retry) break;
    ((LAS u32*)(c.lds + LDS_CTL))[threadIdx.x] = 0;
    if (c.wave == 7) {
      const u32 n = rfl(nv);
      const bool lost = rfl(ov) != n;
      u32 r = role;
      if (lost) {
        if (c.lane == 0) (void)__hip_atomic_fetch_sub(claim + role * 32, 1u, __ATOMIC_RELAXED, __HIP_MEMORY_SCOPE_AGENT);
        WAIT_VM(0);
      }
      if (c.lane == 0) __hip_atomic_store(cnt + blockIdx.x, n + 1, __ATOMIC_RELAXED, __HIP_MEMORY_SCOPE_AGENT);
      if (lost) {
        const u64 t0 = __builtin_amdgcn_s_memrealtime();
#pragma unroll 1
        for (u32 i = 0; i < NWG;) {
          const u32 v = __hip_atomic_load(cnt + i + c.lane, __ATOMIC_RELAXED, __HIP_MEMORY_SCOPE_AGENT);
          if (__all(v > n) || __builtin_amdgcn_s_memrealtime() - t0 > DEADLINE) i += 64;
        }
#pragma unroll 1
        for (u32 i = 0; i < NWG; ++i) {
          u32 expect = n;
          if (c.lane == 0) (void)__hip_atomic_compare_exchange_strong(claim + i * 32, &expect, n + 1, __ATOMIC_RELAXED, __ATOMIC_RELAXED, __HIP_MEMORY_SCOPE_AGENT);
          if (rfl(expect) == n) { r = i; break; }
        }
      }
      if (c.lane == 0) { sync[0] = n + 1; sync[1] = r; }
    }
    BARRIER();
    c.call = rfl(sync[0]);
    const u32 r = rfl(sync[1]);
    if (r == role) break;
    role = r;
    WAIT_VM(0);
  }
  if (isG) { j = u & 1; ct = u >> 1; }
  else { j = u & 3; ct = (u >> 2) * 2 + (grp >> 2); }
  const int ntiles = sender ? 0 : ((isG ? (u < 16 && j == 0) : (!isO && u < 8)) ? 2 : 1);

  if (sender) {
    link_engine(c, p, grp, sidx, fill);
  } else {
    const int due = 450 + PACE * FIRST_POS.v[((isG ? 3 + grp : (isO ? 12 : 0)) + j) * 16 + ct] - LEAD - (isO ? O_EARLY : 0);
    const u32 hold = min(due > 0 ? (u32)due : 0u, isG ? (u32)G_LATEST : 100000u);
    while ((u32)__builtin_amdgcn_s_memrealtime() - c.t_start < hold) __builtin_amdgcn_s_sleep(4);
  }
  __builtin_amdgcn_s_waitcnt(0x0F70);
  const u32 par_off = OFF_STAGE + (c.call & 1) * PAR_BYTES;
  STAMP(1);

  const auto rws = RSRC(c.wsl, WS_NEED);
  const u32 NANS = 0x7fc07fc0u;
  bool bad = false;
  u32x4 y[4];
  const u32 vy = c.wave * 4096 + c.lane * 16;
  u32 tj = j, tct = ct, nwait = 0;
  bool tO = isO && !sender;
  f32x4 acc[4][2];
  constexpr u64 ORD_ODD = 0xba98652174303210ull, ORD_EVEN = 0x76543210ba983210ull, ORD_REL = 0x6521ba9874303210ull;
#pragma unroll 1
  for (int tile = 0; tile < ntiles;) {
    const u32 rt = (isG ? 3 + grp : (tO ? 12 : 0)) + tj;
    const bool rel = !tO && tct >= 16 - NREL;
    const u32 gp = gpos(tj, tct), gt = (grp >> 1) * 32 + gp, yslot = YS_OFF + (isG ? 64 + gt : j * 16 + ct) * 32768;
    if (!tO) {
#pragma unroll
      for (int m = 0; m < 4; ++m)
        __builtin_amdgcn_raw_buffer_store_b128((u32x4){0u, 0u, 0u, 0u}, rws, vy + m * 1024 + (2 * OFF_STAGE + PAR_BYTES - par_off) + yslot, 0, SYS);
    }
    const bool evf = isG && (((rt & 8) ? even_first(1) : even_first(0)) >> (16 * (rt & 3) + tct)) & 1;
    bad |= gemm_tile(c, p, acc, rel ? ORD_REL : (evf ? ORD_EVEN : ORD_ODD),
                     row0_of(rank, rt) * 1024, (rank & 1) * 4096 + tct * 256, par_off + rt * 6 * CHUNK, par_off + (16 + tct) * 6 * CHUNK,
                     OFF_IFLAG + rt * 256, OFF_IFLAG + (16 + tct) * 256, nwait, tile == 0);
    {
      if (tile == 0) { STAMP(4); }
      const bool odd = (c.lane >> 4) & 1;
#pragma unroll
      for (int m = 0; m < 4; ++m) {
        const f32x4 lo = odd ? acc[m][1] : acc[m][0], hi = odd ? acc[m][0] : acc[m][1];
        acc[m][0] = lo; acc[m][1] = hi;
        y[m].x = nz(pack2(lo[0], lo[1])); y[m].y = nz(pack2(lo[2], lo[3]));
        y[m].z = nz(pack2(hi[0], hi[1])); y[m].w = nz(pack2(hi[2], hi[3]));
        if (bad) y[m] = (u32x4){NANS, NANS, NANS, NANS};
      }
      if (!tO) {
        const bool fwd = isG && gp < 2 * NFWD, direct = isG && !fwd;
        if (direct) (void)wait_words(c, OFF_GATE + grp * 128 + c.lane * 4, OFF_GATE + grp * 128, NSEND, NSEND, nwait++);
        STAMP(7);
        char* dst = p.ws[(rank ^ (fwd ? 0u : (isG ? grp : 4u))) & (NR - 1)];
        const auto rd = RSRC(dst, WS_NEED);
        const u32 yo = vy + (fwd ? OFF_LY + gt * 32768 : par_off + yslot);
        const u32 fo = direct ? OFF_GFLAG + gt * 128 + (c.lane >> 1) * 64 + (c.lane & 1) * 4 : (fwd ? OFF_LYF + gt * 4 : OFF_YFLAG + (j * 16 + ct) * 128);
#pragma unroll
        for (int m = 0; m < 4; ++m) {
          __builtin_amdgcn_raw_buffer_store_b128(y[m], rd, yo + m * 1024, 0, SYS);
          asm volatile("" ::: "memory");
          if (m == EARLY_M && !fwd && last_wave(c, tile) && c.lane < (direct ? 4u : 1u)) put_flag(dst + fo, c.call);
          if (m < 3 && direct) WAIT_VM(0);
        }
        if (fwd) {
          WAIT_VM(0);
          if (last_wave(c, tile) && c.lane == 0) put_flag(dst + fo, c.call);
        }
        if (tile == 0) { STAMP(6); }
      }
      if (++tile < ntiles) {
        if (isG) {
          tj = (u >> 1) & 1; tct = 12 + (u >> 2);
          while ((u32)__builtin_amdgcn_s_memrealtime() - c.t_start < T_SECOND) __builtin_amdgcn_s_sleep(4);
        }
        else { tO = true; tj = 3; tct = u * 2 + (grp >> 2); }
      }
    }
  }
  STAMP(5);

  const auto rout = RSRC(p.out, IN_BYTES);

  if (tO) {
    const u32 yt = rfl(tj * 16 + tct);
    bad |= wait_words(c, OFF_YFLAG + yt * 128, OFF_YFLAG + yt * 128, 1, 1, nwait) & 1;
    STAMP(8);
    const u32 yb = vy + par_off + YS_OFF + yt * 32768;
    const u32 yoff[4] = {yb, yb + 1024, yb + 2048, yb + 3072};
    bad = load_y<4>(c, rws, yoff, y, bad);
    const u32 vo = (tj * 128 + (c.wave >> 2) * 64 + (c.lane & 15)) * 8192 + (rank & 1) * 4096 + tct * 256 + (c.wave & 3) * 64 + (c.lane >> 4) * 16;
#pragma unroll
    for (int m = 0; m < 4; ++m) {
      u32x4 o;
      o.x = pack2(acc[m][0][0] + bf_lo(y[m].x), acc[m][0][1] + bf_hi(y[m].x));
      o.y = pack2(acc[m][0][2] + bf_lo(y[m].y), acc[m][0][3] + bf_hi(y[m].y));
      o.z = pack2(acc[m][1][0] + bf_lo(y[m].z), acc[m][1][1] + bf_hi(y[m].z));
      o.w = pack2(acc[m][1][2] + bf_lo(y[m].w), acc[m][1][3] + bf_hi(y[m].w));
      if (bad) o = (u32x4){NANS, NANS, NANS, NANS};
      __builtin_amdgcn_raw_buffer_store_b128(o, rout, vo + m * 16 * 8192, 0, SYS);
    }
  } else if (isG) {
    const u32 gt = ((grp >> 1) & 1) * 32 + gpos(j, ct);
    const u32 ya = 64 + gt, yb = ya + 64;
    const u32 fo = OFF_GFLAG + (gt * 2 + (grp >> 2)) * 64;
    bad |= wait_words(c, fo + c.lane * 4, fo + 128 * 64 + c.lane * 4, 2, 2, nwait) & 1;
    STAMP(8);
    const u32 pw = (grp >> 2) * 4 + (c.wave >> 1), m0 = (c.wave & 1) * 2;
    const u32 vr = pw * 4096 + m0 * 1024 + c.lane * 16;
    const u32 oa = vr + par_off + YS_OFF + ya * 32768, ob = vr + par_off + YS_OFF + yb * 32768;
    const u32 yoff[4] = {oa, oa + 1024, ob, ob + 1024};
    bad = load_y<4>(c, rws, yoff, y, bad);
    const u32x4* ra = y;
    const u32x4* rb = y + 2;
    const u32 vo = (((grp >> 1) & 1) * 256 + j * 128 + (pw >> 2) * 64 + m0 * 16 + (c.lane & 15)) * 8192 +
                   ((rank & 1) ^ 1) * 4096 + ct * 256 + (pw & 3) * 64 + (c.lane >> 4) * 16;
#pragma unroll
    for (int m = 0; m < 2; ++m) {
      u32x4 o;
      o.x = pack2(bf_lo(ra[m].x) + bf_lo(rb[m].x), bf_hi(ra[m].x) + bf_hi(rb[m].x));
      o.y = pack2(bf_lo(ra[m].y) + bf_lo(rb[m].y), bf_hi(ra[m].y) + bf_hi(rb[m].y));
      o.z = pack2(bf_lo(ra[m].z) + bf_lo(rb[m].z), bf_hi(ra[m].z) + bf_hi(rb[m].z));
      o.w = pack2(bf_lo(ra[m].w) + bf_lo(rb[m].w), bf_hi(ra[m].w) + bf_hi(rb[m].w));
      if (bad) o = (u32x4){NANS, NANS, NANS, NANS};
      __builtin_amdgcn_raw_buffer_store_b128(o, rout, vo + m * 16 * 8192, 0, SYS);
    }
  }
}

}

extern "C" void kernel_launch(int rank, int nranks, ncclComm_t comm, hipStream_t stream,
                              void* const* d_in, const int* in_sizes, int n_in,
                              void* d_out, int out_size,
                              const int* in_dtypes, int out_dtype,
                              void* const* d_out_all,
                              void* const* d_ws_all, size_t ws_bytes) {
  (void)comm; (void)in_dtypes; (void)out_dtype; (void)d_out_all;
  if (nranks != NR || (unsigned)rank >= (unsigned)NR || n_in != 2 || ws_bytes < WS_NEED ||
      in_sizes[0] != 4096 * 512 || in_sizes[1] != 512 * 4096 || out_size != 512 * 4096) {
    fprintf(stderr, "gemm_rs: unexpected problem shape\n");
    return;
  }
  Params p;
  p.x = (const char*)d_in[0];
  p.w = (const char*)d_in[1];
  p.out = (char*)d_out;
  p.wsl = (char*)d_ws_all[rank];
  p.rank = 0;
  for (int i = 0; i < NR; ++i) {
    const int r = (L2P >> (4 * i)) & 7;
    p.ws[i] = (char*)d_ws_all[r];
    if (r == rank) p.rank = i;
  }
  hipLaunchKernelGGL(gemm_rs, dim3(NWG), dim3(NT), 0, stream, p);
}
```

```cpp
#include <hip/hip_runtime.h>
#include <rccl/rccl.h>
#include <cstdio>

namespace {

typedef unsigned u32;
typedef unsigned long long u64;
typedef __attribute__((ext_vector_type(4))) unsigned u32x4;
typedef __attribute__((ext_vector_type(4))) float f32x4;
typedef __attribute__((ext_vector_type(2))) float f32x2;
typedef __attribute__((ext_vector_type(2))) __bf16 bf16x2;
typedef __attribute__((ext_vector_type(8))) __bf16 bf16x8;
typedef __attribute__((ext_vector_type(4))) short s16x4;
#define LAS __attribute__((address_space(3)))
#define GAS __attribute__((address_space(1)))
typedef GAS u32 gu32;

constexpr int NR = 8;
constexpr int NWG = 256;
constexpr int NT = 512;
constexpr u32 MiB = 1u << 20;
constexpr u32 CHUNK = 65536;
constexpr int NSEND = 8;
constexpr u32 PIECE = CHUNK / NSEND;
constexpr u32 IN_BYTES = 4 * MiB;

constexpr u32 OFF_CNT = 0;
constexpr u32 OFF_CLAIM = 131072;
constexpr u32 OFF_TMO = 4096;
constexpr u32 OFF_LYF = 8192;
constexpr u32 OFF_GATE = 12288;
constexpr u32 OFF_IFLAG = 16384;
constexpr u32 OFF_YFLAG = 32768;
constexpr u32 OFF_GFLAG = 40960;
constexpr u32 OFF_STAGE = 1 * MiB;
constexpr u32 YS_OFF = 12 * MiB;
constexpr u32 PAR_BYTES = 18 * MiB;
constexpr u32 OFF_LY = OFF_STAGE + 2 * PAR_BYTES;
constexpr u32 WS_NEED = OFF_LY + 4 * MiB;

constexpr u64 DEADLINE = 40000000ull;

constexpr u32 STAGE_B = 32768;
constexpr u32 LDS_CTL = 4 * STAGE_B;
constexpr u32 LDS_BYTES = LDS_CTL + 3072;

enum { K_X = 0, K_W = 1, K_Y = 2, K_H = 3, K_R = 4 };
#define X(i) (K_X * 16 + (i))
#define W(i) (K_W * 16 + (i))
#define Y(i) (K_Y * 16 + (i))
#define H(i) (K_H * 16 + (i))
#define R(i) (K_R * 16 + (i))
constexpr int NREL = 8;
constexpr int NFWD = 4;
constexpr int NPOS_G = 32 - NREL / 2 + NFWD, NPOS_O = 32 + NREL;
constexpr unsigned char SEQ_G[48] = {
    X(0), W(0), W(1), X(1), W(2), X(2), X(3), W(3), X(4), W(4), X(6), W(5), X(8), W(6), X(10), W(7),
    X(5), H(8), X(7), X(9), H(10), X(11), H(12), H(14), X(12), X(13), X(14), X(15), Y(0), Y(1), Y(2), Y(3)};
constexpr unsigned char SEQ_O[48] = {
    X(4), X(6), X(8), X(10), X(0), W(0), W(1), X(1), W(2), X(2), X(3), W(3), X(5), W(4), X(7), X(9),
    W(5), X(11), W(6), W(7), W(8), W(9), R(8), W(10), R(9), W(11), R(10), W(12), R(11), W(13), R(12), W(14),
    W(15), R(13), R(14), R(15), X(12), X(13), X(14), X(15)};
constexpr int seq_pos(const unsigned char* q, int e) { for (int i = 0; i < 48; ++i) if (q[i] == e) return i; return 99; }
constexpr int cmax(int a, int b) { return a > b ? a : b; }
constexpr int cmin(int a, int b) { return a < b ? a : b; }
constexpr int pos_odd(int rt, int ct) { return cmax(seq_pos(SEQ_G, X(rt)), cmin(seq_pos(SEQ_G, W(ct)), seq_pos(SEQ_G, H(ct & ~1)))); }
constexpr int pos_even(int rt, int ct) { return cmax(seq_pos(SEQ_O, X(rt)), seq_pos(SEQ_O, W(ct))); }
constexpr bool even_first(int rt, int ct) { return rt >= 4 && rt < 12 && ct < 16 - NREL && pos_even(rt, ct) < pos_odd(rt, ct); }
struct TileTab { u32 v[64]; };
constexpr TileTab tile_tab() {
  TileTab f{};
  for (int rt = 0; rt < 16; ++rt)
    for (int ct = 0; ct < 16; ++ct)
      f.v[rt * 4 + (ct >> 2)] |= (u32)(cmin(pos_odd(rt, ct), pos_even(rt, ct)) | (even_first(rt, ct) ? 64 : 0)) << (8 * (ct & 3));
  return f;
}
constexpr TileTab TILE_TAB = tile_tab();
#undef X
#undef W
#undef Y
#undef H
#undef R
constexpr u64 seq_pack(const unsigned char* s, int from) { u64 r = 0; for (int t = 0; t < 8; ++t) r |= (u64)s[from + t] << (8 * t); return r; }

constexpr int GATE_POS = NPOS_G - 4;
constexpr u32 PACE = 112;
constexpr u32 PACE_T0 = 150;
constexpr u32 HOLD = 35;
constexpr u32 LATE = 200;
constexpr int LEAD = 650;

constexpr u32 L2P = 0x53764210u;
constexpr u32 X2G = 0x75314602u;

struct Params {
  const char* x;
  const char* w;
  char* wsl;
  u64 rank;
  char* out;
  char* ws[NR];
  TileTab tiles;
};

#define RSRC(p, n) __builtin_amdgcn_make_buffer_rsrc((void*)(p), 0, (n), 0x00020000)
#define WAIT_VM(n) asm volatile("s_waitcnt vmcnt(%0)" ::"n"(n) : "memory")
#define BARRIER() do { asm volatile("s_waitcnt lgkmcnt(0)" ::: "memory"); __builtin_amdgcn_s_barrier(); asm volatile("" ::: "memory"); } while (0)
#define SB() __builtin_amdgcn_sched_barrier(0)
constexpr int SYS = 17;

__device__ __forceinline__ u32 rfl(u32 v) { return __builtin_amdgcn_readfirstlane(v); }
__device__ __forceinline__ __amdgpu_buffer_rsrc_t srsrc(const char* p, u32 n) {
  return RSRC(((u64)rfl((u32)((u64)p >> 32)) << 32) | rfl((u32)(u64)p), rfl(n));
}
__device__ __forceinline__ u32 rdl(u32 v, int lane) { return __builtin_amdgcn_readlane(v, lane); }
__device__ __forceinline__ void put_flag(char* addr, u32 v) { __hip_atomic_store((gu32*)addr, v, __ATOMIC_RELAXED, __HIP_MEMORY_SCOPE_SYSTEM); }
__device__ __forceinline__ u32 get_flag(const char* addr) { return __hip_atomic_load((gu32*)addr, __ATOMIC_RELAXED, __HIP_MEMORY_SCOPE_SYSTEM); }
__device__ __forceinline__ u32 lds_get(LAS u32* w) { return rfl(__hip_atomic_load(w, __ATOMIC_RELAXED, __HIP_MEMORY_SCOPE_WORKGROUP)); }
__device__ __forceinline__ void lds_inc(LAS u32* w, u32 lane) {
  if (lane == 0) __hip_atomic_fetch_add(w, 1u, __ATOMIC_RELAXED, __HIP_MEMORY_SCOPE_WORKGROUP);
}

__device__ __forceinline__ u32 row0_of(u32 q, u32 rt) {
  const bool g = rt - 4u < 8u;
  const u32 d = g ? ((rt - 4u) | 1u) : (rt < 4u ? 4u : 0u);
  const u32 t = g ? (d & 2u) + (rt & 1u) : (rt & 3u);
  return ((L2P >> ((q ^ d) * 4u)) & 7u) * 512u + t * 128u;
}
__device__ __forceinline__ u32 slot_of(u32 d) { return (d & 1u) ? d >> 1 : 4u + (d >> 2); }

struct Ctx {
  LAS char* lds;
  char* wsl;
  const char* x;
  const char* w;
  u32 call, lane, wave, t_start;
};

__device__ __forceinline__ u32 wait_words(const Ctx& c, u32 o0, u32 o1, u32 n, u32 need, u32 slot) {
  LAS u32* res = (LAS u32*)(c.lds + LDS_CTL + 256) + slot;
  if (c.wave == 0) {
    const char* p0 = c.wsl + (c.lane < n ? o0 : OFF_TMO);
    const char* p1 = c.wsl + (c.lane < n ? o1 : OFF_TMO);
    const u64 t0 = __builtin_amdgcn_s_memrealtime();
    u32 r;
    for (;;) {
      const u32 v0 = get_flag(p0), v1 = get_flag(p1);
      if (rdl(v1, 63) == c.call) { r = 129; break; }
      const u64 miss = __ballot(c.lane < n && (v0 < c.call || v1 < c.call));
      r = miss ? 2u * (u32)__builtin_ctzll(miss) : 128u;
      if (r >= 2 * need) break;
      if (__builtin_amdgcn_s_memrealtime() - t0 > DEADLINE) {
        if (c.lane == 0) put_flag(c.wsl + OFF_TMO, c.call);
        r = 129;
        break;
      }
      __builtin_amdgcn_s_sleep(2);
    }
    if (c.lane == 0) *res = r;
  }
  BARRIER();
  return rfl(*res);
}

__device__ __forceinline__ bool last_wave(const Ctx& c, int idx) {
  LAS u32* cnt = (LAS u32*)(c.lds + LDS_CTL) + idx;
  u32 old = 0;
  if (c.lane == 0) old = __hip_atomic_fetch_add(cnt, 1u, __ATOMIC_RELAXED, __HIP_MEMORY_SCOPE_WORKGROUP);
  return rfl(old) == 7u;
}

__device__ __forceinline__ u32 kof_of(u32 q, u32 st) { return (st < 2u ? (q >> 2) : 1u + (((q >> 2) ^ (q >> 1)) & 1u)) * 128u + st * 64u; }

__device__ __forceinline__ u32 vx_of(u32 l, int i) { return (i * 8 + (l >> 3)) * 1024 + (((l & 7) ^ ((i * 4 + (l >> 4)) & 7)) << 4) - i * 1024; }
__device__ __forceinline__ u32 vw_of(u32 l, int i) { return (i * 4 + (l >> 4)) * 8192 + (((l & 15) ^ (((l >> 4) << 2) | (i >> 1))) << 4) - i * 1024; }
__device__ __forceinline__ void fill_slot(const Ctx& c, int pos, const u32 (&vo)[4], __amdgpu_buffer_rsrc_t rs, u32 so, u32 sstep) {
  const u32 lb = (u32)(size_t)c.lds + (u32)(pos & 7) * PIECE;
#pragma unroll
  for (int g = 0; g < 2; ++g)
    asm volatile("s_mov_b32 m0, %0\n\ts_nop 3\n\t"
                 "buffer_load_dwordx4 %1, %5, %6 offen sc0 sc1 lds\n\t"
                 "buffer_load_dwordx4 %2, %5, %6 offen offset:1024 sc0 sc1 lds\n\t"
                 "buffer_load_dwordx4 %3, %5, %6 offen offset:2048 sc0 sc1 lds\n\t"
                 "buffer_load_dwordx4 %4, %5, %6 offen offset:3072 sc0 sc1 lds"
                 ::"s"(rfl(lb + g * 4096u)), "v"(vo[0]), "v"(vo[1]), "v"(vo[2]), "v"(vo[3]), "s"(rs), "s"(rfl(so + g * sstep)) : "memory");
}
__device__ __forceinline__ void first_fill(const Ctx& c, u32 q, bool odd, u32 sidx) {
  static_assert((seq_pack(SEQ_G, 0) & 0xffffff) == 0x111000 && (seq_pack(SEQ_O, 0) & 0xffffff) == 0x080604, "first positions");
  const u32 l = c.lane, pos = c.wave - 4, pc = (sidx + pos) & 7u, kof = kof_of(q, pc >> 1);
  u32 vo[4];
  if (!odd || pos == 0) {
#pragma unroll
    for (int i = 0; i < 4; ++i) vo[i] = vx_of(l, i);
    const u32 d = odd ? 4u : 2 * pos + 1;
    const u32 row0 = ((L2P >> ((q ^ d) * 4u)) & 7u) * 512u + (d & 2u) * 128u;
    fill_slot(c, pos, vo, srsrc(c.x, IN_BYTES), row0 * 1024u + kof * 2u + (pc & 1u) * 65536u, 32768u);
  } else {
#pragma unroll
    for (int i = 0; i < 4; ++i) vo[i] = vw_of(l, i);
    fill_slot(c, pos, vo, srsrc(c.w, IN_BYTES), (kof + (pc & 1u) * 32u) * 8192u + (q & 1u) * 4096u + (pos - 1) * 256u, 131072u);
  }
}

struct Eng { u32 a, b, c, e; };
__device__ __forceinline__ void engine_setup(const Ctx& c, u32 q, u32 d, u32 sidx, Eng& t) {
  const bool odd = d & 1;
  const u32 l = c.lane;
  u64 tab = odd ? seq_pack(SEQ_G, 0) : seq_pack(SEQ_O, 0);
  if (l >= 8) tab = odd ? seq_pack(SEQ_G, 8) : seq_pack(SEQ_O, 8);
  if (l >= 16) tab = odd ? seq_pack(SEQ_G, 16) : seq_pack(SEQ_O, 16);
  if (l >= 24) tab = odd ? seq_pack(SEQ_G, 24) : seq_pack(SEQ_O, 24);
  if (l >= 32) tab = odd ? seq_pack(SEQ_G, 32) : seq_pack(SEQ_O, 32);
  if (l >= 40) tab = odd ? seq_pack(SEQ_G, 40) : seq_pack(SEQ_O, 40);
  const u32 ent = (u32)(tab >> (8 * (l & 7))) & 255u, kind = ent >> 4;
  const u32 ts = (sidx + l) & 7u;
  const bool isy = kind == K_Y, isr = kind == K_R, split = kind >= K_H;
  const u32 id = (ent & 15u) + (kind == K_H ? ts >> 2 : 0u);
  const u32 pc = split ? ((isr ? d : d << 1) & 4u) + (ts & 3u) : ts;
  const u32 dorg = isr ? (d >> 1) + (ts & 4u) : 0u;
  const u32 cid = (kind == K_X ? 0u : 16u) + id, yk = (d >> 1) * 16 + id;
  const u32 oslot = slot_of(d ^ dorg);
  u32 ta, tb = 0, tc = 0, te = 0;
  if (c.wave < 4) {
    const u32 tw = (c.wave + l + (l >> 2)) & 3u;
    ta = pc * PIECE + tw * 2048u + (isy ? YS_OFF + 2 * MiB + yk * CHUNK : (cid * 6 + oslot) * CHUNK);
    tb = (l & 7u) * PIECE + tw * 2048u;
  } else if (c.wave < 7) {
    const u32 kof = kof_of(q, pc >> 1);
    ta = isy ? OFF_LY + yk * CHUNK + pc * PIECE
       : isr ? (cid * 6 + slot_of(dorg)) * CHUNK + pc * PIECE
       : kind == K_X ? row0_of(q, id) * 1024u + kof * 2u + (pc & 1u) * 65536u
                     : (kof + (pc & 1u) * 32u) * 8192u + (q & 1u) * 4096u + id * 256u;
    tb = kind == K_X ? ~0u : 0u;
    tc = isr ? 3u : (isy ? 1u : 0u);
    te = isy ? OFF_LYF + (yk * 2 + (pc >> 2)) * 4 : OFF_IFLAG + cid * 256 + (slot_of(dorg) * 8 + pc) * 4;
  } else {
    ta = isy ? OFF_GFLAG + (yk * 4 + (pc >> 1)) * 64 + (pc & 1u) * 4 : OFF_IFLAG + cid * 256 + (oslot * 8 + pc) * 4;
  }
  asm volatile("" : "+v"(ta), "+v"(tb), "+v"(tc), "+v"(te));
  t.a = ta; t.b = tb; t.c = tc; t.e = te;
}

__device__ __forceinline__ void link_engine(Ctx& c, const Params& p, u32 d, u32 sidx, const Eng& t) {
  char* dst = p.ws[((u32)p.rank ^ d) & (NR - 1)];
  const int npos = (d & 1) ? NPOS_G : NPOS_O;
  LAS u32* done = (LAS u32*)(c.lds + LDS_CTL + 512);
  LAS u32* full = (LAS u32*)(c.lds + LDS_CTL + 768);
  LAS u32* freed = (LAS u32*)(c.lds + LDS_CTL + 1024);
  LAS u32* dead = (LAS u32*)(c.lds + LDS_CTL + 320);
  const u32 l = c.lane;
  const u32 par = OFF_STAGE + (c.call & 1) * PAR_BYTES;
  if (c.wave >= 4 && c.wave < 7) {
    WAIT_VM(0);
    lds_inc(full + (c.wave - 4), l);
    const u32 t_src = t.a + ((t.c & 2u) ? par : 0u), t_mx = t.b, t_ml = 0u - (t.c & 1u), t_dep = t.e;
#pragma unroll 1
    for (int pos = (int)c.wave - 1; pos < npos; pos += 3) {
      const u32 mx = rdl(t_mx, pos), ml = rdl(t_ml, pos), mw = ~(mx | ml);
      u32 vo[4];
#pragma unroll
      for (int i = 0; i < 4; ++i) vo[i] = (vx_of(l, i) & mx) | (vw_of(l, i) & mw) | ((l * 16) & ml);
      if (pos >= 8) while (lds_get(freed + (pos - 8)) < 4u) __builtin_amdgcn_s_sleep(8);
      if (ml) {
        const char* dep = c.wsl + (l == 0 ? rdl(t_dep, pos) : OFF_TMO);
        const u64 t0 = __builtin_amdgcn_s_memrealtime();
        for (;;) {
          const u32 v = get_flag(dep);
          if (rdl(v, 0) >= c.call) break;
          if (rdl(v, 1) == c.call || __builtin_amdgcn_s_memrealtime() - t0 > DEADLINE) {
            if (l == 0) {
              put_flag(c.wsl + OFF_TMO, c.call);
              __hip_atomic_store(dead, 1u, __ATOMIC_RELAXED, __HIP_MEMORY_SCOPE_WORKGROUP);
            }
            break;
          }
          __builtin_amdgcn_s_sleep(2);
        }
      }
      fill_slot(c, pos, vo, srsrc(ml ? c.wsl : (mw ? c.w : c.x), ml ? WS_NEED : IN_BYTES), rdl(t_src, pos),
                (mx & 32768u) | (mw & 131072u) | (ml & 4096u));
      WAIT_VM(0);
      lds_inc(full + pos, l);
    }
  } else {
    if (c.wave < 4) {
      const u32 t_dst = par + t.a, t_lds = t.b;
      const auto rd = RSRC(dst, WS_NEED);
      int sig = 0;
      const u32 t0 = c.t_start;
      u32 due = 2 * PACE_T0 + (sidx * 4 + c.wave) * PACE / 32;
#pragma unroll 1
      for (int pos = 0; pos < npos; ++pos) {
        if (lds_get(full + pos) == 0u) {
          WAIT_VM(0);
          for (; sig < pos; ++sig) lds_inc(done + sig, l);
          while (lds_get(full + pos) == 0u) __builtin_amdgcn_s_sleep(1);
        }
        asm volatile("" ::: "memory");
        LAS char* src = c.lds + rdl(t_lds, pos) + l * 16;
        u32x4 v0 = *(LAS u32x4*)src, v1 = *(LAS u32x4*)(src + 1024);
        const u32 vo = l * 16 + rdl(t_dst, pos);
        asm volatile("s_waitcnt lgkmcnt(0)" : "+v"(v0), "+v"(v1)::"memory");
        lds_inc(freed + pos, l);
        WAIT_VM(1);
        u32 now = ((u32)__builtin_amdgcn_s_memrealtime() - t0) * 2, go = max(due, (pos && now < due + 2 * LATE) ? now + 2 * HOLD : 0u);
        while (now < go) { __builtin_amdgcn_s_sleep(1); now = ((u32)__builtin_amdgcn_s_memrealtime() - t0) * 2; }
        __builtin_amdgcn_raw_buffer_store_b128(v0, rd, vo, 0, SYS);
        WAIT_VM(1);
        now = ((u32)__builtin_amdgcn_s_memrealtime() - t0) * 2; go = max(due + PACE, (pos && now < due + 2 * LATE) ? now + 2 * HOLD : 0u);
        if (sig < pos) { lds_inc(done + sig, l); ++sig; }
        while (now < go) { __builtin_amdgcn_s_sleep(1); now = ((u32)__builtin_amdgcn_s_memrealtime() - t0) * 2; }
        __builtin_amdgcn_raw_buffer_store_b128(v1, rd, vo + 1024, 0, SYS);
        asm volatile("" ::: "memory");
        due += 2 * PACE;
      }
      WAIT_VM(0);
      lds_inc(done + sig, l);
    } else {
      const u32 t_flag = t.a;
#pragma unroll 1
      for (int pos = 0; pos < npos; ++pos) {
        while (lds_get(done + pos) < 4u) __builtin_amdgcn_s_sleep(1);
        if (l == (u32)pos && !lds_get(dead)) put_flag(dst + t_flag, c.call);
        if (pos == GATE_POS && l == 0) put_flag(c.wsl + OFF_GATE + d * 128 + sidx * 4, c.call);
      }
    }
  }
  BARRIER();
}

__device__ __forceinline__ bool gemm_tile(Ctx& c, const Params& p, f32x4 (&acc)[4][2], u64 order, u32 own_a, u32 own_b,
                                          u32 stg_a, u32 stg_b, u32 flag_a, u32 flag_b, u32& nwait) {
  order = ((u64)rfl((u32)(order >> 32)) << 32) | rfl((u32)order);
  const u32 m0a = (u32)(size_t)c.lds + c.wave * 2048;
  const u32 wm = c.wave >> 2, wn = c.wave & 3, fr = c.lane & 15, fq = c.lane >> 4;
  const u32 xa0 = (wm * 64 + fr) * 128 + ((fq ^ (fr >> 1)) << 4);
  const u32 tq = (c.lane >> 2) & 3, tp = c.lane & 3;
  const u32 wa0 = 16384 + (fq * 8 + tq) * 256 + (((wn * 4 + tp) ^ ((tq << 2) | (fq & 1))) << 4) + ((tp & 1) << 3);
  LAS u32* samp = (LAS u32*)(c.lds + LDS_CTL + 2048);
  const auto rws = RSRC(c.wsl, WS_NEED);
  const u32 fw = c.lane < 48 ? (((u32)(order >> (16 + (c.lane & ~3u))) & 15u) * 4 + (c.lane & 3)) * 4 : 0u;
  const u32 vfa = flag_a + fw, vfb = flag_b - 256 + fw;

  const u32 xr = c.wave * 16 + (c.lane >> 3), wk = c.wave * 8 + (c.lane >> 4);
  u32 va0 = xr * 1024 + (((c.lane & 7) ^ ((xr >> 1) & 7)) << 4);
  u32 va1 = (xr + 8) * 1024 + (((c.lane & 7) ^ (((xr + 8) >> 1) & 7)) << 4) - 1024;
  u32 vb0 = wk * 8192 + (((c.lane & 15) ^ (((wk & 3) << 2) | ((wk >> 3) & 1))) << 4);
  u32 vb1 = vb0 + 4 * 8192 - 1024;
  auto desc = [](const char* q) { return (u32x4){rfl((u32)(u64)q), rfl((u32)((u64)q >> 32)) & 0xffffu, 0u, 0x00020000u}; };
  u32x4 ra = desc(c.x), rb = desc(c.w);
  u32 nrec = IN_BYTES, sa = rfl(own_a), sb = rfl(own_b), stepa = 128, stepb = 64 * 8192;
  stg_a = rfl(stg_a); stg_b = rfl(stg_b);
  int limit = 8;

  u32 soa, sob, m0v;
  auto prep = [&](int s) {
    const u32 img = ((u32)(order >> ((2 * (s & ~1)) & 63)) & 15u) * 2u + (u32)(s & 1);
    soa = sa + img * stepa; sob = sb + img * stepb;
    m0v = m0a + (u32)(s & 3) * STAGE_B;
    ra.z = rb.z = s < limit ? nrec : 0u;
    asm volatile("" : "+s"(soa), "+s"(sob), "+s"(m0v), "+s"(ra), "+s"(rb));
  };
  auto dma = [&](int i) {
    if (i == 0) asm volatile("s_mov_b32 m0, %0\n\ts_nop 0\n\tbuffer_load_dwordx4 %1, %2, %3 offen sc0 sc1 lds" ::"s"(m0v), "v"(va0), "s"(ra), "s"(soa) : "memory");
    if (i == 1) asm volatile("s_mov_b32 m0, %0\n\ts_nop 0\n\tbuffer_load_dwordx4 %1, %2, %3 offen offset:1024 sc0 sc1 lds" ::"s"(m0v), "v"(va1), "s"(ra), "s"(soa) : "memory");
    if (i == 2) asm volatile("s_add_u32 m0, %0, 0x4000\n\ts_nop 0\n\tbuffer_load_dwordx4 %1, %2, %3 offen sc0 sc1 lds" ::"s"(m0v), "v"(vb0), "s"(rb), "s"(sob) : "memory", "scc");
    if (i == 3) asm volatile("s_add_u32 m0, %0, 0x4000\n\ts_nop 0\n\tbuffer_load_dwordx4 %1, %2, %3 offen offset:1024 sc0 sc1 lds" ::"s"(m0v), "v"(vb1), "s"(rb), "s"(sob) : "memory", "scc");
  };
  bf16x8 X[2][2][4];
  s16x4 W[2][2][2][2];
  auto mm = [&](int o, int ii) {
    const int h = ii >> 3, m = (ii >> 1) & 3, n = ii & 1;
    const bf16x8 wf = __builtin_bit_cast(bf16x8, __builtin_shufflevector(W[o][h][n][0], W[o][h][n][1], 0, 1, 2, 3, 4, 5, 6, 7));
    acc[m][n] = __builtin_amdgcn_mfma_f32_16x16x32_bf16(wf, X[o][h][m], acc[m][n], 0, 0, 0);
  };
#pragma unroll
  for (int m = 0; m < 4; ++m) acc[m][0] = acc[m][1] = (f32x4){0.f, 0.f, 0.f, 0.f};
#pragma unroll
  for (int h = 0; h < 2; ++h) {
#pragma unroll
    for (int m = 0; m < 4; ++m) X[1][h][m] = (bf16x8)(__bf16)0.0f;
#pragma unroll
    for (int n = 0; n < 2; ++n) W[1][h][n][0] = W[1][h][n][1] = (s16x4){0, 0, 0, 0};
  }
  bool bad = false;
  int t2 = 0;
#pragma unroll 1
  for (;;) {
    SB();
    asm volatile("s_nop 4" ::: "memory");
    SB();
#pragma unroll
    for (int s = 0; s < 3; ++s) {
      prep(2 * t2 + s);
#pragma unroll
      for (int i = 0; i < 4; ++i) dma(i);
    }
    prep(2 * t2 + 3);
#pragma unroll 1
    for (; 2 * t2 < limit; ++t2) {
#pragma unroll
      for (int b = 0; b < 2; ++b) {
        const int s = 2 * t2 + b, o = b ^ 1;
#pragma unroll
        for (int m = 0; m < 4; ++m) asm volatile("" : "+a"(acc[m][0]), "+a"(acc[m][1]));
        asm volatile("s_waitcnt lgkmcnt(0)" ::: "memory"); SB();
        mm(o, 0); SB();
        WAIT_VM(8);
        BARRIER();
        SB();
        LAS char* st = c.lds + (u32)(s & 3) * STAGE_B;
        auto rd = [&](int ii) {
          const int h = ii >> 3, i = ii & 7;
          if (i < 4) W[b][h][i >> 1][i & 1] = __builtin_amdgcn_ds_read_tr16_b64_v4i16((LAS s16x4*)(st + (wa0 ^ (u32)((i >> 1) << 3)) + h * 8192 + (i & 1) * 1024));
          else X[b][h][i - 4] = *(LAS bf16x8*)(st + (xa0 ^ (u32)(h << 6)) + (i - 4) * 2048);
        };
        rd(0); rd(1); SB(); mm(o, 1); SB(); rd(4); SB(); mm(o, 2); SB();
        if (b == 1 && __builtin_expect(limit - s <= 5 && limit < 32, 0)) {
          const int ahead = limit - s;
          bool sample = ahead == 5;
          if (ahead < 5) {
            LAS u32* sp = samp + (ahead == 1 ? 128 : 0);
            const u32 f0 = sp[c.lane], f1 = sp[64 + c.lane];
            const u64 miss = __ballot((f0 < f1 ? f0 : f1) < c.call) | (1ull << 48);
            const int nl = (int)rfl(8u + ((u32)__builtin_ctzll(miss) >> 4) * 8u);
            if (s == 5) {
              va0 = va1 = vb0 = vb1 = c.wave * 2048 + c.lane * 16;
              ra = rb = desc(c.wsl); nrec = WS_NEED; sa = stg_a; sb = stg_b; stepa = stepb = 16384;
            }
            sample = ahead == 3 && nl == limit;
            const bool late = ahead == 1 && nl > limit;
            limit = nl;
            asm volatile("s_nop 4" ::: "memory");
            if (late) {
#pragma unroll 1
              for (int k = 1; k < 3; ++k) { prep(s + k); dma(0); dma(1); dma(2); dma(3); }
            }
            prep(s + 3);
          }
          if (sample && c.wave == 0) {
            asm volatile("s_mov_b32 m0, %0\n\ts_nop 3\n\tbuffer_load_dword %1, %3, 0 offen sc0 sc1 lds\n\t"
                         "buffer_load_dword %2, %3, 0 offen offset:256 sc0 sc1 lds"
                         ::"s"(rfl((u32)(size_t)samp + (ahead == 3 ? 512u : 0u))), "v"(vfa), "v"(vfb), "s"(rws) : "memory");
          }
        }
        SB(); dma(0); rd(2); rd(3); SB(); mm(o, 3); SB(); rd(5); SB();
        mm(o, 4); SB(); rd(6); SB(); mm(o, 5); SB(); rd(7); SB(); mm(o, 6); SB(); dma(1); rd(8); rd(9); SB(); mm(o, 7); SB(); rd(12); SB();
        mm(o, 8); SB(); rd(10); rd(11); SB(); mm(o, 9); SB(); rd(13); SB(); mm(o, 10); SB(); dma(2); rd(14); SB(); mm(o, 11); SB(); rd(15); SB();
        mm(o, 12); SB(); mm(o, 13); SB(); dma(3); SB(); mm(o, 14); SB(); prep(s + 4); SB(); mm(o, 15); SB();
      }
    }
    WAIT_VM(0);
    if (limit == 32) break;
    const u32 r = wait_words(c, vfa, vfb + 256, 48, 2 * limit, nwait++);
    bad |= r & 1;
    limit = r >= 96 ? 32 : 8 + (int)(r >> 5) * 8;
  }
#pragma unroll
  for (int i = 0; i < 16; ++i) mm(1, i);
  BARRIER();
  return bad;
}

__device__ __forceinline__ u32 pack2(float a, float b) {
  return __builtin_bit_cast(u32, __builtin_convertvector((f32x2){a, b}, bf16x2));
}
__device__ __forceinline__ float bf_lo(u32 v) { return __builtin_bit_cast(float, v << 16); }
__device__ __forceinline__ float bf_hi(u32 v) { return __builtin_bit_cast(float, v & 0xffff0000u); }

__device__ __forceinline__ u32 nz(u32 v) { return v ? v : 0x80000000u; }
template <int N>
__device__ __forceinline__ bool load_y(const Ctx& c, __amdgpu_buffer_rsrc_t rws, const u32 (&off)[N], u32x4 (&v)[N], bool bad) {
  const u64 t0 = __builtin_amdgcn_s_memrealtime();
  for (;;) {
    u32 lo = ~0u;
#pragma unroll
    for (int i = 0; i < N; ++i) v[i] = __builtin_amdgcn_raw_buffer_load_b128(rws, off[i], 0, SYS);
#pragma unroll
    for (int i = 0; i < N; ++i) lo = min(min(min(lo, v[i].x), min(v[i].y, v[i].z)), v[i].w);
    if (bad || __all(lo != 0u)) return bad;
    if (__builtin_amdgcn_s_memrealtime() - t0 > DEADLINE) {
      if (c.lane == 0) put_flag(c.wsl + OFF_TMO, c.call);
      return true;
    }
    __builtin_amdgcn_s_sleep(1);
    asm volatile("" ::: "memory");
  }
}

__global__ void __launch_bounds__(NT, 2) gemm_rs(Params p) {
  __shared__ __attribute__((aligned(1024))) char smem[LDS_BYTES];
  Ctx c;
  c.lds = (LAS char*)smem;
  c.lane = threadIdx.x & 63;
  c.wave = rfl(threadIdx.x >> 6);
  c.t_start = (u32)__builtin_amdgcn_s_memrealtime();
  u32 rank = (u32)p.rank & (NR - 1);
  c.x = p.x; c.w = p.w; c.wsl = p.wsl;
  u32 role = (__builtin_amdgcn_s_getreg((3 << 11) | 20) & 7u) * 32u + (31u - ((blockIdx.x >> 3) & 31u));
  u32 grp, u, sidx;
  bool isG, isO, sender;
  auto decode = [&]() {
    grp = (X2G >> ((role >> 5) * 4)) & 7; u = role & 31;
    if ((grp & 3) == 0) u ^= 31;
    isG = grp & 1; isO = (grp & 3) == 2;
    sender = isG ? u >= 24 : (isO && (u & 3) == 3);
    sidx = (isG ? u : u >> 2) & 7;
  };
  decode();
  asm volatile("" : "+s"(c.x), "+s"(c.w), "+s"(c.wsl), "+s"(rank), "+s"(c.t_start));
  if (sender && c.wave - 4 < 3) first_fill(c, rank ^ grp, isG, sidx);
  gu32* cnt = (gu32*)(c.wsl + OFF_CNT);
  gu32* claim = (gu32*)(c.wsl + OFF_CLAIM);
  LAS u32* sync = (LAS u32*)(c.lds + LDS_CTL) + 500;
  u32 nv = 0, ov = 0;
  if (c.wave == 7) {
    nv = __hip_atomic_load(cnt + blockIdx.x, __ATOMIC_RELAXED, __HIP_MEMORY_SCOPE_AGENT);
    ov = __builtin_amdgcn_raw_ptr_buffer_atomic_add_i32(1, RSRC(claim + role * 32, 4), c.lane * 4, 0, 0);
  }
  SB();
  Eng eng;
  {
    if (sender) engine_setup(c, rank ^ grp, grp, sidx, eng);
    ((LAS u32*)(c.lds + LDS_CTL))[threadIdx.x] = 0;
    if (c.wave == 7) {
      const u32 n = rfl(nv);
      const bool lost = rfl(ov) != n;
      u32 r = role;
      if (lost) {
        if (c.lane == 0) (void)__hip_atomic_fetch_sub(claim + role * 32, 1u, __ATOMIC_RELAXED, __HIP_MEMORY_SCOPE_AGENT);
        WAIT_VM(0);
      }
      if (c.lane == 0) __hip_atomic_store(cnt + blockIdx.x, n + 1, __ATOMIC_RELAXED, __HIP_MEMORY_SCOPE_AGENT);
      if (lost) {
        const u64 t0 = __builtin_amdgcn_s_memrealtime();
#pragma unroll 1
        for (u32 i = 0; i < NWG;) {
          const u32 v = __hip_atomic_load(cnt + i + c.lane, __ATOMIC_RELAXED, __HIP_MEMORY_SCOPE_AGENT);
          if (__all(v > n) || __builtin_amdgcn_s_memrealtime() - t0 > DEADLINE) i += 64;
        }
#pragma unroll 1
        for (u32 i = 0; i < NWG; ++i) {
          u32 expect = n;
          if (c.lane == 0) (void)__hip_atomic_compare_exchange_strong(claim + i * 32, &expect, n + 1, __ATOMIC_RELAXED, __ATOMIC_RELAXED, __HIP_MEMORY_SCOPE_AGENT);
          if (rfl(expect) == n) { r = i; break; }
        }
      }
      if (c.lane == 0) { sync[0] = n + 1; sync[1] = r; }
    }
    BARRIER();
    c.call = rfl(sync[0]);
    const u32 r = rfl(sync[1]);
    if (__builtin_expect(r != role, 0)) {
      role = r;
      WAIT_VM(0);
      BARRIER();
      decode();
      if (sender && c.wave - 4 < 3) first_fill(c, rank ^ grp, isG, sidx);
      if (sender) engine_setup(c, rank ^ grp, grp, sidx, eng);
    }
  }
  u32 rt = isG ? 3 + grp + (u & 1) : (isO ? 12 : 0) + (u & 3);
  u32 tct = isG ? u >> 1 : (u >> 2) * 2 + (grp >> 2);
  const int ntiles = sender ? 0 : ((isG ? (u < 16 && (u & 1) == 0) : (!isO && u < 8)) ? 2 : 1);
  u32 tinfo = p.tiles.v[(rt * 4 + (tct >> 2)) & 63] >> (8 * (tct & 3));

  if (sender) {
    link_engine(c, p, grp, sidx, eng);
  } else {
    const int due = 480 + (int)PACE * (int)(tinfo & 63) - LEAD - (isO ? 1000 : 0);
    const u32 hold = due > 0 ? (u32)due : 0u;
    while ((u32)__builtin_amdgcn_s_memrealtime() - c.t_start < hold) __builtin_amdgcn_s_sleep(16);
  }
  __builtin_amdgcn_s_waitcnt(0x0F70);
  const u32 par_off = OFF_STAGE + (c.call & 1) * PAR_BYTES;

  const auto rws = RSRC(c.wsl, WS_NEED);
  const u32 NANS = 0x7fc07fc0u;
  bool bad = false;
  u32x4 y[4];
  const u32 vy = c.wave * 4096 + c.lane * 16;
  u32 nwait = 0;
  bool tO = isO && !sender;
  f32x4 acc[4][2];
  constexpr u64 ORD_ODD = 0xba98652174303210ull, ORD_EVEN = 0x76543210ba983210ull, ORD_REL = 0x6521ba9874303210ull;
#pragma unroll 1
  for (int tile = 0; tile < ntiles;) {
    const bool rel = !tO && tct >= 16 - NREL;
    const u32 gt = (rt - 4) * 16 + tct, dl = isG ? (rt - 4) | 1u : 4u, yslot = YS_OFF + (isG ? 64 + gt : rt * 16 + tct) * 32768;
    if (!tO) {
#pragma unroll
      for (int m = 0; m < 4; ++m)
        __builtin_amdgcn_raw_buffer_store_b128((u32x4){0u, 0u, 0u, 0u}, rws, vy + m * 1024 + (2 * OFF_STAGE + PAR_BYTES - par_off) + yslot, 0, SYS);
    }
    bad |= gemm_tile(c, p, acc, rel ? ORD_REL : ((tinfo & 64) ? ORD_EVEN : ORD_ODD),
                     row0_of(rank, rt) * 1024, (rank & 1) * 4096 + tct * 256, par_off + rt * 6 * CHUNK, par_off + (16 + tct) * 6 * CHUNK,
                     OFF_IFLAG + rt * 256, OFF_IFLAG + (16 + tct) * 256, nwait);
    {
      const bool odd = (c.lane >> 4) & 1;
#pragma unroll
      for (int m = 0; m < 4; ++m) {
        const f32x4 lo = odd ? acc[m][1] : acc[m][0], hi = odd ? acc[m][0] : acc[m][1];
        acc[m][0] = lo; acc[m][1] = hi;
        y[m].x = nz(pack2(lo[0], lo[1])); y[m].y = nz(pack2(lo[2], lo[3]));
        y[m].z = nz(pack2(hi[0], hi[1])); y[m].w = nz(pack2(hi[2], hi[3]));
        if (bad) y[m] = (u32x4){NANS, NANS, NANS, NANS};
      }
      if (!tO) {
        const bool fwd = isG && (gt & 31) < 2 * NFWD, direct = isG && !fwd;
        if (direct) (void)wait_words(c, OFF_GATE + dl * 128 + c.lane * 4, OFF_GATE + dl * 128, NSEND, NSEND, nwait++);
        const int depth = direct ? (tct >= 12 ? 2 : 1) : ((!isG && tct >= 12) ? 1 : 4);
        char* dst = p.ws[(rank ^ (fwd ? 0u : dl)) & (NR - 1)];
        const auto rd = RSRC(dst, WS_NEED);
        const u32 yo = vy + (fwd ? OFF_LY + gt * 32768 : par_off + yslot);
        const u32 fo = direct ? OFF_GFLAG + gt * 128 + (c.lane >> 1) * 64 + (c.lane & 1) * 4 : (fwd ? OFF_LYF + gt * 4 : OFF_YFLAG + (rt * 16 + tct) * 128);
#pragma unroll
        for (int m = 0; m < 4; ++m) {
          __builtin_amdgcn_raw_buffer_store_b128(y[m], rd, yo + m * 1024, 0, SYS);
          asm volatile("" ::: "memory");
          if (m == (depth == 1 ? 2 : 1) && !fwd && last_wave(c, tile) && c.lane < (direct ? 4u : 1u)) put_flag(dst + fo, c.call);
          if (m < 3) {
            if (depth == 1) WAIT_VM(0);
            else if (depth == 2) WAIT_VM(1);
          }
        }
        if (fwd) {
          WAIT_VM(0);
          if (last_wave(c, tile) && c.lane == 0) put_flag(dst + fo, c.call);
        }
      }
      if (++tile < ntiles) {
        if (isG) { rt = 3 + grp + ((u >> 1) & 1); tct = 12 + (u >> 2); }
        else { tO = true; rt = 15; tct = u * 2 + (grp >> 2); }
        tinfo = 0;
      }
    }
  }

  const auto rout = RSRC(p.out, IN_BYTES);

  if (tO) {
    const u32 tj = rt - 12, yt = rfl(tj * 16 + tct);
    bad |= wait_words(c, OFF_YFLAG + yt * 128, OFF_YFLAG + yt * 128, 1, 1, nwait) & 1;
    const u32 yb = vy + par_off + YS_OFF + yt * 32768;
    const u32 yoff[4] = {yb, yb + 1024, yb + 2048, yb + 3072};
    bad = load_y<4>(c, rws, yoff, y, bad);
    const u32 vo = (tj * 128 + (c.wave >> 2) * 64 + (c.lane & 15)) * 8192 + (rank & 1) * 4096 + tct * 256 + (c.wave & 3) * 64 + (c.lane >> 4) * 16;
#pragma unroll
    for (int m = 0; m < 4; ++m) {
      u32x4 o;
      o.x = pack2(acc[m][0][0] + bf_lo(y[m].x), acc[m][0][1] + bf_hi(y[m].x));
      o.y = pack2(acc[m][0][2] + bf_lo(y[m].y), acc[m][0][3] + bf_hi(y[m].y));
      o.z = pack2(acc[m][1][0] + bf_lo(y[m].z), acc[m][1][1] + bf_hi(y[m].z));
      o.w = pack2(acc[m][1][2] + bf_lo(y[m].w), acc[m][1][3] + bf_hi(y[m].w));
      if (bad) o = (u32x4){NANS, NANS, NANS, NANS};
      __builtin_amdgcn_raw_buffer_store_b128(o, rout, vo + m * 16 * 8192, 0, SYS);
    }
  } else if (isG) {
    const u32 j = u & 1, ct = u >> 1, la = (grp >> 1) & 1;
    const u32 ga = la * 32 + j * 16 + ct, gb = ga + 64;
    const u32 ya = 64 + ga, yb = 64 + gb;
    const u32 fo = OFF_GFLAG + (grp >> 2) * 64 + c.lane * 4;
    bad |= wait_words(c, fo + ga * 128, fo + gb * 128, 2, 2, nwait) & 1;
    const u32 pw = (grp >> 2) * 4 + (c.wave >> 1), m0 = (c.wave & 1) * 2;
    const u32 vr = pw * 4096 + m0 * 1024 + c.lane * 16;
    const u32 oa = vr + par_off + YS_OFF + ya * 32768, ob = vr + par_off + YS_OFF + yb * 32768;
    const u32 yoff[4] = {oa, oa + 1024, ob, ob + 1024};
    bad = load_y<4>(c, rws, yoff, y, bad);
    const u32x4* ra = y;
    const u32x4* rb = y + 2;
    const u32 vo = (((grp >> 1) & 1) * 256 + j * 128 + (pw >> 2) * 64 + m0 * 16 + (c.lane & 15)) * 8192 +
                   ((rank & 1) ^ 1) * 4096 + ct * 256 + (pw & 3) * 64 + (c.lane >> 4) * 16;
#pragma unroll
    for (int m = 0; m < 2; ++m) {
      u32x4 o;
      o.x = pack2(bf_lo(ra[m].x) + bf_lo(rb[m].x), bf_hi(ra[m].x) + bf_hi(rb[m].x));
      o.y = pack2(bf_lo(ra[m].y) + bf_lo(rb[m].y), bf_hi(ra[m].y) + bf_hi(rb[m].y));
      o.z = pack2(bf_lo(ra[m].z) + bf_lo(rb[m].z), bf_hi(ra[m].z) + bf_hi(rb[m].z));
      o.w = pack2(bf_lo(ra[m].w) + bf_lo(rb[m].w), bf_hi(ra[m].w) + bf_hi(rb[m].w));
      if (bad) o = (u32x4){NANS, NANS, NANS, NANS};
      __builtin_amdgcn_raw_buffer_store_b128(o, rout, vo + m * 16 * 8192, 0, SYS);
    }
  }
}

}

extern "C" void kernel_launch(int rank, int nranks, ncclComm_t comm, hipStream_t stream,
                              void* const* d_in, const int* in_sizes, int n_in,
                              void* d_out, int out_size,
                              const int* in_dtypes, int out_dtype,
                              void* const* d_out_all,
                              void* const* d_ws_all, size_t ws_bytes) {
  (void)comm; (void)in_dtypes; (void)out_dtype; (void)d_out_all;
  if (nranks != NR || (unsigned)rank >= (unsigned)NR || n_in != 2 || ws_bytes < WS_NEED ||
      in_sizes[0] != 4096 * 512 || in_sizes[1] != 512 * 4096 || out_size != 512 * 4096) {
    fprintf(stderr, "gemm_rs: unexpected problem shape\n");
    return;
  }
  Params p;
  p.x = (const char*)d_in[0];
  p.w = (const char*)d_in[1];
  p.out = (char*)d_out;
  p.wsl = (char*)d_ws_all[rank];
  p.tiles = TILE_TAB;
  p.rank = 0;
  for (int i = 0; i < NR; ++i) {
    const int r = (L2P >> (4 * i)) & 7;
    p.ws[i] = (char*)d_ws_all[r];
    if (r == rank) p.rank = i;
  }
  hipLaunchKernelGGL(gemm_rs, dim3(NWG), dim3(NT), 0, stream, p);
}
```

```cpp
#include <hip/hip_runtime.h>
#include <rccl/rccl.h>
#include <cstdio>

namespace {

typedef unsigned u32;
typedef unsigned long long u64;
typedef __attribute__((ext_vector_type(4))) unsigned u32x4;
typedef __attribute__((ext_vector_type(4))) float f32x4;
typedef __attribute__((ext_vector_type(2))) float f32x2;
typedef __attribute__((ext_vector_type(2))) __bf16 bf16x2;
typedef __attribute__((ext_vector_type(8))) __bf16 bf16x8;
typedef __attribute__((ext_vector_type(4))) short s16x4;
#define LAS __attribute__((address_space(3)))
#define GAS __attribute__((address_space(1)))
typedef GAS u32 gu32;

constexpr int NR = 8;
constexpr int NWG = 256;
constexpr int NT = 512;
constexpr u32 MiB = 1u << 20;
constexpr u32 CHUNK = 65536;
constexpr int NSEND = 8;
constexpr u32 PIECE = CHUNK / NSEND;
constexpr u32 IN_BYTES = 4 * MiB;

constexpr u32 OFF_CNT = 0;
constexpr u32 OFF_CLAIM = 131072;
constexpr u32 OFF_TMO = 4096;
constexpr u32 OFF_LYF = 8192;
constexpr u32 OFF_RFLAG = 9216;
constexpr u32 OFF_GATE = 12288;
constexpr u32 OFF_IFLAG = 16384;
constexpr u32 OFF_YFLAG = 32768;
constexpr u32 OFF_GFLAG = 40960;
constexpr u32 OFF_STAGE = 1 * MiB;
constexpr u32 YS_OFF = 12 * MiB;
constexpr u32 RB_OFF = 18 * MiB;
constexpr u32 PAR_BYTES = 19 * MiB;
constexpr u32 OFF_LY = OFF_STAGE + 2 * PAR_BYTES;
constexpr u32 WS_NEED = OFF_LY + 4 * MiB;

constexpr u32 DEADLINE = 150000000u;

constexpr u32 STAGE_B = 32768;
constexpr u32 LDS_CTL = 4 * STAGE_B;
constexpr u32 LDS_BYTES = LDS_CTL + 3072;

enum { K_X = 0, K_W = 1, K_Y = 2, K_R = 3, K_F = 4, K_S = 5 };
#define Xa(i) (K_X * 32 + (i))
#define Xb(i) (K_X * 32 + 16 + (i))
#define Wa(i) (K_W * 32 + (i))
#define Wb(i) (K_W * 32 + 16 + (i))
#define X(i) Xa(i), Xb(i)
#define W(i) Wa(i), Wb(i)
#define R(i) (K_R * 32 + (i)), (K_R * 32 + 16 + (i))
#define Y(i) (K_Y * 32 + (i))
#define F(i) (K_F * 32 + (i))
#define S(i) (K_S * 32 + (i))
constexpr u32 DETOUR = 9;
constexpr unsigned char SEQ_G[] = {
    Xa(0), Wa(0), Wa(1), Xa(1), Xb(0), Wb(0), Wa(2), Xa(2), Wb(1), Xb(1), Xa(3), Wa(3), Wb(2), Xb(2), Xb(3), Wb(3),
    X(4), W(4), X(6), W(5), X(8), W(6), X(10), W(7), X(5),
    W(8), X(7), W(9), X(9), Wa(10), Wa(11), X(11), Wa(12), Wa(13), Wa(14), Wa(15),
    X(12), X(13), X(14), X(15), Y(0), Y(1), Y(2), Y(3), Y(4), Y(5), Y(6), Y(7)};
constexpr unsigned char SEQ_O[] = {
    X(4), X(6), X(8), X(10), X(0), W(0), W(1), X(1), W(2), X(2), X(3), W(3), X(5), W(4), X(7), X(9),
    W(5), X(11), W(6), W(7), W(8), W(9),
    W(10), W(11), R(10), W(12), R(11), W(13), R(12), W(14), R(13), W(15), R(14), R(15),
    Xa(12), Xa(13), F(0), F(1), Xa(14), Xa(15), Xb(12), Xb(13), Xb(14), Xb(15), S(0), S(1)};
#undef Xa
#undef Xb
#undef Wa
#undef Wb
#undef X
#undef W
#undef R
#undef Y
#undef F
#undef S
constexpr int NPOS_G = (int)sizeof(SEQ_G) / 2 - 2, NPOS_O = (int)sizeof(SEQ_O) / 2;
static_assert(sizeof(SEQ_G) % 2 == 0 && sizeof(SEQ_O) % 2 == 0 && NPOS_G <= 48 && NPOS_O <= 48, "schedule size");
constexpr u32 fwd_mask() { u32 m = 0; for (unsigned char e : SEQ_G) if ((e >> 5) == K_Y) m |= 1u << (e & 31); return m; }
constexpr u64 seq_pack(const unsigned char* s) { u64 r = 0; for (int t = 0; t < 6; ++t) r |= (u64)s[t] << (8 * t); return r; }

struct Tables {
  unsigned short seq[5][48];
  u64 tile[256];
};
constexpr Tables build_tables() {
  Tables t{};
  int pos[5][256] = {};
  for (int v = 0; v < 5; ++v) {
    const unsigned char* q = v < 4 ? SEQ_G : SEQ_O;
    const int n = v < 4 ? (int)sizeof(SEQ_G) : (int)sizeof(SEQ_O);
    for (int i = 0; i < 48; ++i) t.seq[v][i] = 0;
    for (int i = 0; i < 256; ++i) pos[v][i] = -1;
    int k = 0;
    for (int i = 0; i < n; ++i) {
      if (v < 4 && (q[i] >> 5) == K_X && ((q[i] & 15) >> 1) == 2 + v) continue;
      t.seq[v][k >> 1] |= (unsigned short)(q[i] << (8 * (k & 1)));
      pos[v][q[i]] = k >> 1;
      ++k;
    }
  }
  for (int rt = 0; rt < 16; ++rt)
    for (int ct = 0; ct < 16; ++ct) {
      int key[12] = {}, n = 0;
      for (int h = 0; h < 12; ++h) {
        const int slot = h >> 1, v = slot < 4 ? slot : 4, sel = slot < 4 ? (h ^ slot) & 1 : h & 1;
        const int px = pos[v][K_X * 32 + sel * 16 + rt];
        int pw = pos[v][K_W * 32 + sel * 16 + ct];
        if (pw < 0) pw = pos[4][K_R * 32 + (((slot >> 1) ^ h) & 1) * 16 + ct];
        if (px < 0) continue;
        int i = n++;
        const int me = (px > pw ? px : pw) * 16 + h;
        for (; i > 0 && key[i - 1] > me; --i) key[i] = key[i - 1];
        key[i] = me;
      }
      u64 o = (u64)(key[1] >> 4) << 48;
      for (int i = 0; i < n; ++i) o |= (u64)(key[i] & 15) << (4 * i);
      t.tile[rt * 16 + ct] = o;
    }
  return t;
}
__constant__ const Tables TABLES = build_tables();

constexpr int GATE_POS = NPOS_G - 4;
constexpr u32 PACE = 112;
constexpr u32 PACE_T0 = 150;
constexpr u32 HOLD = 35;
constexpr u32 LATE = 200;
constexpr int LEAD = 650;

constexpr u32 L2P = 0x53764210u;
constexpr u32 X2G = 0x75314602u;

struct Params {
  const char* x;
  const char* w;
  char* wsl;
  u64 rank;
  char* out;
  char* ws[NR];
};

#define RSRC(p, n) __builtin_amdgcn_make_buffer_rsrc((void*)(p), 0, (n), 0x00020000)
#define WAIT_VM(n) asm volatile("s_waitcnt vmcnt(%0)" ::"n"(n) : "memory")
#define BARRIER() do { asm volatile("s_waitcnt lgkmcnt(0)" ::: "memory"); __builtin_amdgcn_s_barrier(); asm volatile("" ::: "memory"); } while (0)
#define SB() __builtin_amdgcn_sched_barrier(0)
constexpr u32 NANS = 0x7fc07fc0u;
constexpr int SYS = 17;

__device__ __forceinline__ u32 rfl(u32 v) { return __builtin_amdgcn_readfirstlane(v); }
__device__ __forceinline__ __amdgpu_buffer_rsrc_t srsrc(const char* p, u32 n) {
  return RSRC(((u64)rfl((u32)((u64)p >> 32)) << 32) | rfl((u32)(u64)p), rfl(n));
}
__device__ __forceinline__ u32 rdl(u32 v, int lane) { return __builtin_amdgcn_readlane(v, lane); }
__device__ __forceinline__ void put_flag(char* addr, u32 v) { __hip_atomic_store((gu32*)addr, v, __ATOMIC_RELAXED, __HIP_MEMORY_SCOPE_SYSTEM); }
__device__ __forceinline__ u32 get_flag(const char* addr) { return __hip_atomic_load((gu32*)addr, __ATOMIC_RELAXED, __HIP_MEMORY_SCOPE_SYSTEM); }
__device__ __forceinline__ u32 lds_get(LAS u32* w) { return rfl(__hip_atomic_load(w, __ATOMIC_RELAXED, __HIP_MEMORY_SCOPE_WORKGROUP)); }
__device__ __forceinline__ void lds_inc(LAS u32* w, u32 lane) {
  if (lane == 0) __hip_atomic_fetch_add(w, 1u, __ATOMIC_RELAXED, __HIP_MEMORY_SCOPE_WORKGROUP);
}

__device__ __forceinline__ u32 row0_of(u32 q, u32 rt) {
  const bool g = rt - 4u < 8u;
  const u32 d = g ? ((rt - 4u) | 1u) : (rt < 4u ? 4u : 0u);
  const u32 t = g ? (d & 2u) + (rt & 1u) : (rt & 3u);
  return ((L2P >> ((q ^ d) * 4u)) & 7u) * 512u + t * 128u;
}
__device__ __forceinline__ u32 slot_of(u32 d) { return (d & 1u) ? d >> 1 : 4u + (d >> 2); }

struct Ctx {
  LAS char* lds;
  char* wsl;
  const char* x;
  const char* w;
  u32 call, lane, wave, t_start;
};

__device__ __forceinline__ u32 age(const Ctx& c) { return (u32)__builtin_amdgcn_s_memrealtime() - c.t_start; }

__device__ __forceinline__ u32 wait_words(const Ctx& c, u32 o0, u32 o1, u32 n, u32 need, u32 slot) {
  LAS u32* res = (LAS u32*)(c.lds + LDS_CTL + 256) + slot;
  if (c.wave == 0) {
    const char* p0 = c.wsl + (c.lane < n ? o0 : OFF_TMO);
    const char* p1 = c.wsl + (c.lane < n ? o1 : OFF_TMO);
    u32 r;
    for (;;) {
      const u32 v0 = get_flag(p0), v1 = get_flag(p1);
      if (rdl(v1, 63) == c.call) { r = 129; break; }
      const u64 miss = __ballot(c.lane < n && (v0 < c.call || v1 < c.call));
      r = miss ? 2u * (u32)__builtin_ctzll(miss) : 128u;
      if (r >= 2 * need) break;
      if (age(c) > DEADLINE) {
        if (c.lane == 0) put_flag(c.wsl + OFF_TMO, c.call);
        r = 129;
        break;
      }
      __builtin_amdgcn_s_sleep(2);
    }
    if (c.lane == 0) *res = r;
  }
  BARRIER();
  return rfl(*res);
}

__device__ __forceinline__ bool last_wave(const Ctx& c, int idx) {
  LAS u32* cnt = (LAS u32*)(c.lds + LDS_CTL) + idx;
  u32 old = 0;
  if (c.lane == 0) old = __hip_atomic_fetch_add(cnt, 1u, __ATOMIC_RELAXED, __HIP_MEMORY_SCOPE_WORKGROUP);
  return rfl(old) == 7u;
}

__device__ __forceinline__ u32 kof_of(u32 q, u32 st) { return (st < 2u ? (q >> 2) : 1u + (((q >> 2) ^ (q >> 1)) & 1u)) * 128u + st * 64u; }

__device__ __forceinline__ u32 vx_of(u32 l, int i) { return (i * 8 + (l >> 3)) * 1024 + (((l & 7) ^ ((i * 4 + (l >> 4)) & 7)) << 4) - i * 1024; }
__device__ __forceinline__ u32 vw_of(u32 l, int i) { return (i * 4 + (l >> 4)) * 8192 + (((l & 15) ^ (((l >> 4) << 2) | (i >> 1))) << 4) - i * 1024; }
__device__ __forceinline__ void fill_slot(const Ctx& c, int pos, const u32 (&vo)[4], __amdgpu_buffer_rsrc_t rs, u32 so, u32 sstep) {
  const u32 lb = (u32)(size_t)c.lds + (u32)(pos & 7) * PIECE;
#pragma unroll
  for (int g = 0; g < 2; ++g)
    asm volatile("s_mov_b32 m0, %0\n\ts_nop 3\n\t"
                 "buffer_load_dwordx4 %1, %5, %6 offen sc0 sc1 lds\n\t"
                 "buffer_load_dwordx4 %2, %5, %6 offen offset:1024 sc0 sc1 lds\n\t"
                 "buffer_load_dwordx4 %3, %5, %6 offen offset:2048 sc0 sc1 lds\n\t"
                 "buffer_load_dwordx4 %4, %5, %6 offen offset:3072 sc0 sc1 lds"
                 ::"s"(rfl(lb + g * 4096u)), "v"(vo[0]), "v"(vo[1]), "v"(vo[2]), "v"(vo[3]), "s"(rs), "s"(rfl(so + g * sstep)) : "memory");
}
__device__ __forceinline__ void first_fill(const Ctx& c, u32 q, u32 d, u32 sidx) {
  static_assert(((seq_pack(SEQ_G) | seq_pack(SEQ_O)) & 0xc0c0c0c0c0c0ull) == 0 && (seq_pack(SEQ_G) & 0x0c0c0c0c0c0cull) == 0, "first positions");
  const u32 l = c.lane, pos = c.wave - 4, ts = (sidx + pos) & 7u;
  const u32 ent = (u32)(((d & 1) ? seq_pack(SEQ_G) : seq_pack(SEQ_O)) >> (16 * pos + 2 * (ts & 4u))), id = ent & 15u;
  const u32 pc = (((ent >> 2) ^ ((d & 1) ? d << 1 : 0u)) & 4u) + (ts & 3u), kof = kof_of(q, pc >> 1);
  u32 vo[4];
  if ((ent & 32u) == 0) {
#pragma unroll
    for (int i = 0; i < 4; ++i) vo[i] = vx_of(l, i);
    fill_slot(c, pos, vo, srsrc(c.x, IN_BYTES), row0_of(q, id) * 1024u + kof * 2u + (pc & 1u) * 65536u, 32768u);
  } else {
#pragma unroll
    for (int i = 0; i < 4; ++i) vo[i] = vw_of(l, i);
    fill_slot(c, pos, vo, srsrc(c.w, IN_BYTES), (kof + (pc & 1u) * 32u) * 8192u + (q & 1u) * 4096u + id * 256u, 131072u);
  }
}

struct Eng { u32 a, b, c, e; };
__device__ __forceinline__ void engine_setup(const Ctx& c, u32 q, u32 d, u32 sidx, Eng& t) {
  const u32 l = c.lane;
  const u32 ts = (sidx + l) & 7u;
  const u32 ent = ((u32)TABLES.seq[(d & 1) ? d >> 1 : 4u][l < 48 ? l : 0] >> (2 * (ts & 4u))) & 255u, kind = ent >> 5, id = ent & 15u;
  const bool isy = kind == K_Y, isr = kind == K_R, isf = kind == K_F, iss = kind == K_S;
  const u32 prt = (((iss ? ~d : d) >> 1) & 2u) + id, po = (ts & 3u) * PIECE;
  const u32 half = isy ? id << 2 : (isr ? d : (ent >> 2) ^ ((d & 1) ? d << 1 : 0u));
  const u32 pc = (half & 4u) + (ts & 3u);
  const u32 dorg = isr ? (d >> 1) + ((ent >> 2) & 4u) : 0u;
  const u32 cid = (kind == K_X ? 0u : 16u) + id, yk = (d >> 1) * 16 + ((ent & 31u) >> 1);
  const u32 oslot = slot_of(d ^ dorg);
  u32 ta, tb = 0, tc = 0, te = 0;
  if (c.wave < 4) {
    const u32 tw = (c.wave + l + (l >> 2)) & 3u;
    ta = pc * PIECE + (isy ? YS_OFF + 2 * MiB + yk * CHUNK : (cid * 6 + oslot) * CHUNK);
    if (isf) ta = RB_OFF + prt * 32768 + po;
    if (iss) ta = YS_OFF + (prt * 16 + DETOUR) * 32768 + po;
    ta += tw * 2048u;
    tb = (l & 7u) * PIECE + tw * 2048u;
  } else if (c.wave < 7) {
    const u32 kof = kof_of(q, pc >> 1);
    ta = isy ? OFF_LY + yk * CHUNK + pc * PIECE
       : isr ? (cid * 6 + slot_of(dorg)) * CHUNK + pc * PIECE
       : kind == K_X ? row0_of(q, id) * 1024u + kof * 2u + (pc & 1u) * 65536u
                     : (kof + (pc & 1u) * 32u) * 8192u + (q & 1u) * 4096u + id * 256u;
    tb = kind == K_X ? ~0u : 0u;
    tc = isr ? 3u : (isy ? 1u : 0u);
    te = isy ? OFF_LYF + (yk * 2 + (pc >> 2)) * 4 : OFF_IFLAG + cid * 256 + (slot_of(dorg) * 8 + pc) * 4;
    if (isf) { ta = OFF_LY + (28 + prt) * 32768 + po; tc = 1; te = OFF_LYF + (28 + prt) * 4; }
    if (iss) { ta = RB_OFF + prt * 32768 + po; tc = 3; te = OFF_RFLAG + (prt * 4 + (ts & 3u)) * 4; }
  } else {
    ta = isy ? OFF_GFLAG + (yk * 4 + (pc >> 1)) * 64 + (pc & 1u) * 4 : OFF_IFLAG + cid * 256 + (oslot * 8 + pc) * 4;
    if (isf) ta = OFF_RFLAG + (prt * 4 + (ts & 3u)) * 4;
    if (iss) ta = OFF_YFLAG + (prt * 16 + DETOUR) * 128 + (ts & 3u) * 4;
  }
  asm volatile("" : "+v"(ta), "+v"(tb), "+v"(tc), "+v"(te));
  t.a = ta; t.b = tb; t.c = tc; t.e = te;
}

__device__ __forceinline__ void link_engine(Ctx& c, const Params& p, u32 d, u32 sidx, Eng& t) {
  char* dst = p.ws[((u32)p.rank ^ d) & (NR - 1)];
  const int npos = (d & 1) ? NPOS_G : NPOS_O;
  LAS u32* done = (LAS u32*)(c.lds + LDS_CTL + 512);
  LAS u32* full = (LAS u32*)(c.lds + LDS_CTL + 768);
  LAS u32* freed = (LAS u32*)(c.lds + LDS_CTL + 1024);
  const u32 l = c.lane;
  const u32 par = OFF_STAGE + (c.call & 1) * PAR_BYTES;
  if (c.wave >= 4 && c.wave < 7) {
    WAIT_VM(0);
    lds_inc(full + (c.wave - 4), l);
    engine_setup(c, ((u32)p.rank ^ d) & (NR - 1), d, sidx, t);
    const u32 t_src = t.a + ((t.c & 2u) ? par : 0u), t_mx = t.b, t_ml = 0u - (t.c & 1u), t_dep = t.e;
#pragma unroll 1
    for (int pos = (int)c.wave - 1; pos < npos; pos += 3) {
      const u32 mx = rdl(t_mx, pos), ml = rdl(t_ml, pos), mw = ~(mx | ml);
      u32 vo[4];
#pragma unroll
      for (int i = 0; i < 4; ++i) vo[i] = (vx_of(l, i) & mx) | (vw_of(l, i) & mw) | ((l * 16) & ml);
      if (pos >= 8) while (lds_get(freed + (pos - 8)) < 4u) __builtin_amdgcn_s_sleep(8);
      u32 mark = 1;
      if (ml) {
        const char* dep = c.wsl + (l == 0 ? rdl(t_dep, pos) : OFF_TMO);
        for (;;) {
          const u32 v = get_flag(dep);
          if (rdl(v, 0) >= c.call) break;
          if (rdl(v, 1) == c.call || age(c) > DEADLINE) {
            if (l == 0) put_flag(c.wsl + OFF_TMO, c.call);
            mark = 2;
            break;
          }
          __builtin_amdgcn_s_sleep(2);
        }
      }
      fill_slot(c, pos, vo, srsrc((const char*)(((u64)c.wsl & (u64)(long long)(int)ml) | ((u64)c.w & (u64)(long long)(int)mw) | ((u64)c.x & (u64)(long long)(int)mx)), (ml & WS_NEED) | (~ml & IN_BYTES)), rdl(t_src, pos),
                (mx & 32768u) | (mw & 131072u) | (ml & 4096u));
      WAIT_VM(0);
      if (l == 0) __hip_atomic_store(full + pos, mark, __ATOMIC_RELAXED, __HIP_MEMORY_SCOPE_WORKGROUP);
    }
  } else {
    if (c.wave < 4) {
      const u32 t_dst = par + t.a, t_lds = t.b;
      const auto rd = RSRC(dst, WS_NEED);
      int sig = 0;
      u32 due = 2 * PACE_T0 + (sidx * 4 + c.wave) * PACE / 32;
#pragma unroll 1
      for (int pos = 0; pos < npos; ++pos) {
        u32 mark = lds_get(full + pos);
        if (mark == 0u) {
          WAIT_VM(0);
          for (; sig < pos; ++sig) lds_inc(done + sig, l);
          while ((mark = lds_get(full + pos)) == 0u) __builtin_amdgcn_s_sleep(1);
        }
        asm volatile("" ::: "memory");
        LAS char* src = c.lds + rdl(t_lds, pos) + l * 16;
        u32x4 v0 = *(LAS u32x4*)src, v1 = *(LAS u32x4*)(src + 1024);
        const u32 vo = l * 16 + rdl(t_dst, pos);
        asm volatile("s_waitcnt lgkmcnt(0)" : "+v"(v0), "+v"(v1)::"memory");
        lds_inc(freed + pos, l);
        if (mark == 2u) v0 = v1 = (u32x4){NANS, NANS, NANS, NANS};
        WAIT_VM(1);
        u32 now = age(c) * 2, go = max(due, (pos && now < due + 2 * LATE) ? now + 2 * HOLD : 0u);
        while (now < go) { __builtin_amdgcn_s_sleep(1); now = age(c) * 2; }
        __builtin_amdgcn_raw_buffer_store_b128(v0, rd, vo, 0, SYS);
        WAIT_VM(1);
        now = age(c) * 2; go = max(due + PACE, (pos && now < due + 2 * LATE) ? now + 2 * HOLD : 0u);
        if (sig < pos) { lds_inc(done + sig, l); ++sig; }
        while (now < go) { __builtin_amdgcn_s_sleep(1); now = age(c) * 2; }
        __builtin_amdgcn_raw_buffer_store_b128(v1, rd, vo + 1024, 0, SYS);
        asm volatile("" ::: "memory");
        due += 2 * PACE;
      }
      WAIT_VM(0);
      lds_inc(done + sig, l);
    } else {
      const u32 t_flag = t.a;
#pragma unroll 1
      for (int pos = 0; pos < npos; ++pos) {
        while (lds_get(done + pos) < 4u) __builtin_amdgcn_s_sleep(1);
        if (l == (u32)pos) put_flag(dst + t_flag, c.call);
        if (pos == GATE_POS && l == 0) put_flag(c.wsl + OFF_GATE + d * 128 + sidx * 4, c.call);
      }
    }
  }
  BARRIER();
}

__device__ __forceinline__ bool gemm_tile(Ctx& c, const Params& p, f32x4 (&acc)[4][2], u64 order, int end, u32 own_a, u32 own_b,
                                          u32 stg_a, u32 stg_b, u32 flag_a, u32 flag_b, u32& nwait) {
  order = ((u64)rfl((u32)(order >> 32)) << 32) | rfl((u32)order);
  end = (int)rfl((u32)end);
  const u32 nfl = 2 * (end - 8);
  const u32 m0a = (u32)(size_t)c.lds + c.wave * 2048;
  const u32 wm = c.wave >> 2, wn = c.wave & 3, fr = c.lane & 15, fq = c.lane >> 4;
  const u32 xa0 = (wm * 64 + fr) * 128 + ((fq ^ (fr >> 1)) << 4);
  const u32 tq = (c.lane >> 2) & 3, tp = c.lane & 3;
  const u32 wa0 = 16384 + (fq * 8 + tq) * 256 + (((wn * 4 + tp) ^ ((tq << 2) | (fq & 1))) << 4) + ((tp & 1) << 3);
  LAS u32* samp = (LAS u32*)(c.lds + LDS_CTL + 2048);
  const auto rws = RSRC(c.wsl, WS_NEED);
  const u32 fw = c.lane < 48 ? (((u32)(order >> (16 + (c.lane & ~3u))) & 15u) * 4 + (c.lane & 3)) * 4 : 0u;
  const u32 vfa = flag_a + fw, vfb = flag_b - 256 + fw;

  const u32 xr = c.wave * 16 + (c.lane >> 3), wk = c.wave * 8 + (c.lane >> 4);
  u32 va0 = xr * 1024 + (((c.lane & 7) ^ ((xr >> 1) & 7)) << 4);
  u32 va1 = (xr + 8) * 1024 + (((c.lane & 7) ^ (((xr + 8) >> 1) & 7)) << 4) - 1024;
  u32 vb0 = wk * 8192 + (((c.lane & 15) ^ (((wk & 3) << 2) | ((wk >> 3) & 1))) << 4);
  u32 vb1 = vb0 + 4 * 8192 - 1024;
  auto desc = [](const char* q) { return (u32x4){rfl((u32)(u64)q), rfl((u32)((u64)q >> 32)) & 0xffffu, 0u, 0x00020000u}; };
  u32x4 ra = desc(c.x), rb = desc(c.w);
  u32 nrec = IN_BYTES, sa = rfl(own_a), sb = rfl(own_b), stepa = 128, stepb = 64 * 8192;
  stg_a = rfl(stg_a); stg_b = rfl(stg_b);
  int limit = 8;

  u32 soa, sob, m0v;
  auto prep = [&](int s) {
    const u32 img = ((u32)(order >> ((2 * (s & ~1)) & 63)) & 15u) * 2u + (u32)(s & 1);
    soa = sa + img * stepa; sob = sb + img * stepb;
    m0v = m0a + (u32)(s & 3) * STAGE_B;
    ra.z = rb.z = s < limit ? nrec : 0u;
    asm volatile("" : "+s"(soa), "+s"(sob), "+s"(m0v), "+s"(ra), "+s"(rb));
  };
  auto dma = [&](int i) {
    if (i == 0) asm volatile("s_mov_b32 m0, %0\n\ts_nop 0\n\tbuffer_load_dwordx4 %1, %2, %3 offen sc0 sc1 lds" ::"s"(m0v), "v"(va0), "s"(ra), "s"(soa) : "memory");
    if (i == 1) asm volatile("s_mov_b32 m0, %0\n\ts_nop 0\n\tbuffer_load_dwordx4 %1, %2, %3 offen offset:1024 sc0 sc1 lds" ::"s"(m0v), "v"(va1), "s"(ra), "s"(soa) : "memory");
    if (i == 2) asm volatile("s_add_u32 m0, %0, 0x4000\n\ts_nop 0\n\tbuffer_load_dwordx4 %1, %2, %3 offen sc0 sc1 lds" ::"s"(m0v), "v"(vb0), "s"(rb), "s"(sob) : "memory", "scc");
    if (i == 3) asm volatile("s_add_u32 m0, %0, 0x4000\n\ts_nop 0\n\tbuffer_load_dwordx4 %1, %2, %3 offen offset:1024 sc0 sc1 lds" ::"s"(m0v), "v"(vb1), "s"(rb), "s"(sob) : "memory", "scc");
  };
  bf16x8 X[2][2][4];
  s16x4 W[2][2][2][2];
  auto mm = [&](int o, int ii) {
    const int h = ii >> 3, m = (ii >> 1) & 3, n = ii & 1;
    const bf16x8 wf = __builtin_bit_cast(bf16x8, __builtin_shufflevector(W[o][h][n][0], W[o][h][n][1], 0, 1, 2, 3, 4, 5, 6, 7));
    acc[m][n] = __builtin_amdgcn_mfma_f32_16x16x32_bf16(wf, X[o][h][m], acc[m][n], 0, 0, 0);
  };
#pragma unroll
  for (int m = 0; m < 4; ++m) acc[m][0] = acc[m][1] = (f32x4){0.f, 0.f, 0.f, 0.f};
#pragma unroll
  for (int h = 0; h < 2; ++h) {
#pragma unroll
    for (int m = 0; m < 4; ++m) X[1][h][m] = (bf16x8)(__bf16)0.0f;
#pragma unroll
    for (int n = 0; n < 2; ++n) W[1][h][n][0] = W[1][h][n][1] = (s16x4){0, 0, 0, 0};
  }
  bool bad = false, have = false;
  int t2 = 0;
#pragma unroll 1
  for (;;) {
    SB();
    asm volatile("s_nop 4" ::: "memory");
    SB();
#pragma unroll
    for (int s = 0; s < 3; ++s) {
      prep(2 * t2 + s);
#pragma unroll
      for (int i = 0; i < 4; ++i) dma(i);
    }
    prep(2 * t2 + 3);
#pragma unroll 1
    for (; 2 * t2 < limit; ++t2) {
#pragma unroll
      for (int b = 0; b < 2; ++b) {
        const int s = 2 * t2 + b, o = b ^ 1;
#pragma unroll
        for (int m = 0; m < 4; ++m) asm volatile("" : "+a"(acc[m][0]), "+a"(acc[m][1]));
        asm volatile("s_waitcnt lgkmcnt(0)" ::: "memory"); SB();
        mm(o, 0); SB();
        WAIT_VM(8);
        BARRIER();
        SB();
        LAS char* st = c.lds + (u32)(s & 3) * STAGE_B;
        auto rd = [&](int ii) {
          const int h = ii >> 3, i = ii & 7;
          if (i < 4) W[b][h][i >> 1][i & 1] = __builtin_amdgcn_ds_read_tr16_b64_v4i16((LAS s16x4*)(st + (wa0 ^ (u32)((i >> 1) << 3)) + h * 8192 + (i & 1) * 1024));
          else X[b][h][i - 4] = *(LAS bf16x8*)(st + (xa0 ^ (u32)(h << 6)) + (i - 4) * 2048);
        };
        rd(0); rd(1); SB(); mm(o, 1); SB(); rd(4); SB(); mm(o, 2); SB();
        if (b == 1 && __builtin_expect(limit - s <= 5 && limit < end, 0)) {
          const int ahead = limit - s;
          bool sample = ahead > 1;
          if (have) {
            LAS u32* sp = samp + (ahead == 1 ? 128 : 0);
            const u32 f0 = sp[c.lane], f1 = sp[64 + c.lane];
            const u64 miss = __ballot((f0 < f1 ? f0 : f1) < c.call) | (1ull << nfl);
            const int nl = (int)rfl(8u + ((u32)__builtin_ctzll(miss) >> 2) * 2u);
            if (s == 5) {
              va0 = va1 = vb0 = vb1 = c.wave * 2048 + c.lane * 16;
              ra = rb = desc(c.wsl); nrec = WS_NEED; sa = stg_a; sb = stg_b; stepa = stepb = 16384;
            }
            const bool up = nl >= limit + 4 || nl == end;
            if (up) { limit = nl; sample = false; }
            asm volatile("s_nop 4" ::: "memory");
            if (up && ahead == 1) {
#pragma unroll 1
              for (int k = 1; k < 3; ++k) { prep(s + k); dma(0); dma(1); dma(2); dma(3); }
            }
            prep(s + 3);
          }
          have = sample;
          if (sample && c.wave == 0) {
            asm volatile("s_mov_b32 m0, %0\n\ts_nop 3\n\tbuffer_load_dword %1, %3, 0 offen sc0 sc1 lds\n\t"
                         "buffer_load_dword %2, %3, 0 offen offset:256 sc0 sc1 lds"
                         ::"s"(rfl((u32)(size_t)samp + (ahead == 3 ? 512u : 0u))), "v"(vfa), "v"(vfb), "s"(rws) : "memory");
          }
        }
        SB(); dma(0); rd(2); rd(3); SB(); mm(o, 3); SB(); rd(5); SB();
        mm(o, 4); SB(); rd(6); SB(); mm(o, 5); SB(); rd(7); SB(); mm(o, 6); SB(); dma(1); rd(8); rd(9); SB(); mm(o, 7); SB(); rd(12); SB();
        mm(o, 8); SB(); rd(10); rd(11); SB(); mm(o, 9); SB(); rd(13); SB(); mm(o, 10); SB(); dma(2); rd(14); SB(); mm(o, 11); SB(); rd(15); SB();
        mm(o, 12); SB(); mm(o, 13); SB(); dma(3); SB(); mm(o, 14); SB(); prep(s + 4); SB(); mm(o, 15); SB();
      }
    }
    WAIT_VM(0);
    if (limit == end) break;
    const u32 r = wait_words(c, vfa, vfb + 256, nfl, min(2 * (limit - 4), (int)nfl), nwait++ & 15);
    bad |= r & 1;
    limit = min(end, 8 + (int)(r >> 3) * 2);
  }
#pragma unroll
  for (int i = 0; i < 16; ++i) mm(1, i);
  BARRIER();
  return bad;
}

__device__ __forceinline__ u32 pack2(float a, float b) {
  return __builtin_bit_cast(u32, __builtin_convertvector((f32x2){a, b}, bf16x2));
}
__device__ __forceinline__ float bf_lo(u32 v) { return __builtin_bit_cast(float, v << 16); }
__device__ __forceinline__ float bf_hi(u32 v) { return __builtin_bit_cast(float, v & 0xffff0000u); }

__device__ __forceinline__ u32 nz(u32 v) { return v ? v : 0x80000000u; }
template <int N>
__device__ __forceinline__ bool load_y(const Ctx& c, __amdgpu_buffer_rsrc_t rws, const u32 (&off)[N], u32x4 (&v)[N], u32 f0, u32 f1, bool bad) {
  u32 t_end = DEADLINE;
  const char* pf = c.wsl + (c.lane < 4 ? f0 : (c.lane < 8 ? f1 : OFF_TMO));
  bool seen = false;
  for (;;) {
    u32 lo = ~0u;
#pragma unroll
    for (int i = 0; i < N; ++i) v[i] = __builtin_amdgcn_raw_buffer_load_b128(rws, off[i], 0, SYS);
#pragma unroll
    for (int i = 0; i < N; ++i) lo = min(min(min(lo, v[i].x), min(v[i].y, v[i].z)), v[i].w);
    if (bad || __all(lo != 0u)) return bad;
    for (;;) {
      if (age(c) > t_end) {
        if (c.lane == 0) put_flag(c.wsl + OFF_TMO, c.call);
        return true;
      }
      if (seen) break;
      const u32 f = get_flag(pf);
      if (rdl(f, 63) == c.call) return true;
      seen = __all(c.lane >= 8 || f >= c.call);
      if (!seen) __builtin_amdgcn_s_sleep(8);
      else t_end = age(c) + DEADLINE / 32;
    }
    asm volatile("" ::: "memory");
  }
}

__global__ void __launch_bounds__(NT, 2) gemm_rs(Params p) {
  __shared__ __attribute__((aligned(1024))) char smem[LDS_BYTES];
  Ctx c;
  c.lds = (LAS char*)smem;
  c.lane = threadIdx.x & 63;
  c.wave = rfl(threadIdx.x >> 6);
  c.t_start = (u32)__builtin_amdgcn_s_memrealtime();
  u32 rank = (u32)p.rank & (NR - 1);
  c.x = p.x; c.w = p.w; c.wsl = p.wsl;
  u32 role = (__builtin_amdgcn_s_getreg((3 << 11) | 20) & 7u) * 32u + (31u - ((blockIdx.x >> 3) & 31u));
  u32 grp, u, sidx;
  bool isG, isO, sender;
  auto decode = [&]() {
    grp = (X2G >> ((role >> 5) * 4)) & 7; u = role & 31;
    if ((grp & 3) == 0) u ^= 31;
    isG = grp & 1; isO = (grp & 3) == 2;
    sender = isG ? u >= 24 : (isO && (u & 3) == 3);
    sidx = (isG ? u : u >> 2) & 7;
  };
  decode();
  asm volatile("" : "+s"(c.x), "+s"(c.w), "+s"(c.wsl), "+s"(rank), "+s"(c.t_start));
  if (sender && c.wave - 4 < 3) first_fill(c, rank ^ grp, grp, sidx);
  gu32* cnt = (gu32*)(c.wsl + OFF_CNT);
  gu32* claim = (gu32*)(c.wsl + OFF_CLAIM);
  LAS u32* sync = (LAS u32*)(c.lds + LDS_CTL) + 500;
  u32 nv = 0, ov = 0;
  if (c.wave == 7) {
    nv = __hip_atomic_load(cnt + blockIdx.x, __ATOMIC_RELAXED, __HIP_MEMORY_SCOPE_AGENT);
    ov = __builtin_amdgcn_raw_ptr_buffer_atomic_add_i32(1, RSRC(claim + role * 32, 4), c.lane * 4, 0, 0);
  }
  SB();
  Eng eng;
  {
    if (sender && c.wave - 4 >= 3) engine_setup(c, rank ^ grp, grp, sidx, eng);
    ((LAS u32*)(c.lds + LDS_CTL))[threadIdx.x] = 0;
    if (c.wave == 7) {
      const u32 n = rfl(nv);
      const bool lost = rfl(ov) != n;
      u32 r = role;
      if (lost) {
        if (c.lane == 0) (void)__hip_atomic_fetch_sub(claim + role * 32, 1u, __ATOMIC_RELAXED, __HIP_MEMORY_SCOPE_AGENT);
        WAIT_VM(0);
      }
      if (c.lane == 0) __hip_atomic_store(cnt + blockIdx.x, n + 1, __ATOMIC_RELAXED, __HIP_MEMORY_SCOPE_AGENT);
      if (lost) {
#pragma unroll 1
        for (u32 i = 0; i < NWG;) {
          const u32 v = __hip_atomic_load(cnt + i + c.lane, __ATOMIC_RELAXED, __HIP_MEMORY_SCOPE_AGENT);
          if (__all(v > n) || age(c) > DEADLINE) i += 64;
        }
#pragma unroll 1
        for (u32 i = 0; i < NWG; ++i) {
          u32 expect = n;
          if (c.lane == 0) (void)__hip_atomic_compare_exchange_strong(claim + i * 32, &expect, n + 1, __ATOMIC_RELAXED, __ATOMIC_RELAXED, __HIP_MEMORY_SCOPE_AGENT);
          if (rfl(expect) == n) { r = i; break; }
        }
      }
      if (c.lane == 0) { sync[0] = n + 1; sync[1] = r; }
    }
    BARRIER();
    c.call = rfl(sync[0]);
    const u32 r = rfl(sync[1]);
    if (__builtin_expect(r != role, 0)) {
      role = r;
      WAIT_VM(0);
      BARRIER();
      decode();
      if (sender && c.wave - 4 < 3) first_fill(c, rank ^ grp, grp, sidx);
      if (sender && c.wave - 4 >= 3) engine_setup(c, rank ^ grp, grp, sidx, eng);
    }
  }
  u32 rt = isG ? 3 + grp + (u & 1) : (isO ? 12 : 0) + (u & 3);
  u32 tct = isG ? u >> 1 : (u >> 2) * 2 + (grp >> 2);
  const int ntiles = sender ? 0 : ((isG ? (u < 16 && (u & 1) == 0) : (!isO && u < 8)) ? 2 : 1);
  const bool hasQ = isG && ntiles == 1;
  const u32 qrt = (grp & 2) + (grp >> 2), qct = u < 16 ? u >> 1 : u - 8;
  u64 tinfo = TABLES.tile[(rt * 16 + tct) & 255];

  if (sender) {
    link_engine(c, p, grp, sidx, eng);
  } else {
    const int due = 480 + (int)PACE * (int)(tinfo >> 48) - LEAD - (isO ? 1000 : (hasQ ? 600 : 0));
    const u32 hold = due > 0 ? (u32)due : 0u;
    while (age(c) < hold) __builtin_amdgcn_s_sleep(16);
  }
  __builtin_amdgcn_s_waitcnt(0x0F70);
  const u32 par_off = OFF_STAGE + (c.call & 1) * PAR_BYTES;

  const auto rws = RSRC(c.wsl, WS_NEED);
  bool bad = false;
  u32x4 y[4];
  const u32 vy = c.wave * 4096 + c.lane * 16;
  u32 nwait = 0;
  bool tO = isO && !sender;
  f32x4 acc[4][2];
  f32x4 qacc[4][2];
#pragma unroll 1
  for (int tile = hasQ ? -1 : 0; tile < ntiles;) {
    const bool tQ = tile < 0;
    const u32 gt = (rt - 4) * 16 + tct, dl = isG ? (rt - 4) | 1u : 4u, yslot = YS_OFF + (isG ? 64 + gt : rt * 16 + tct) * 32768;
    if (!tO && !tQ) {
#pragma unroll
      for (int m = 0; m < 4; ++m)
        __builtin_amdgcn_raw_buffer_store_b128((u32x4){0u, 0u, 0u, 0u}, rws, vy + m * 1024 + (2 * OFF_STAGE + PAR_BYTES - par_off) + yslot, 0, SYS);
    }
    bad |= gemm_tile(c, p, acc, (tinfo << 16) | 0x3210u, tQ ? 8 : (isG ? 28 : 32),
                     row0_of(rank, tQ ? 12 + qrt : rt) * 1024, ((rank ^ (tQ ? 1u : 0u)) & 1) * 4096 + (tQ ? qct : tct) * 256,
                     par_off + rt * 6 * CHUNK, par_off + (16 + tct) * 6 * CHUNK,
                     OFF_IFLAG + rt * 256, OFF_IFLAG + (16 + tct) * 256, nwait);
    {
      const bool odd = (c.lane >> 4) & 1;
#pragma unroll
      for (int m = 0; m < 4; ++m) {
        const f32x4 lo = odd ? acc[m][1] : acc[m][0], hi = odd ? acc[m][0] : acc[m][1];
        acc[m][0] = lo; acc[m][1] = hi;
      }
      if (tQ) {
#pragma unroll
        for (int m = 0; m < 4; ++m) { qacc[m][0] = acc[m][0]; qacc[m][1] = acc[m][1]; }
      } else if (!tO) {
#pragma unroll
        for (int m = 0; m < 4; ++m) {
          y[m].x = nz(pack2(acc[m][0][0], acc[m][0][1])); y[m].y = nz(pack2(acc[m][0][2], acc[m][0][3]));
          y[m].z = nz(pack2(acc[m][1][0], acc[m][1][1])); y[m].w = nz(pack2(acc[m][1][2], acc[m][1][3]));
          if (bad) y[m] = (u32x4){NANS, NANS, NANS, NANS};
        }
        const bool fwd = isG ? (fwd_mask() >> (gt & 31)) & 1 : tct == DETOUR, direct = isG && !fwd;
        const u32 lt = isG ? gt : 28 + rt;
        if (direct) (void)wait_words(c, OFF_GATE + dl * 128 + c.lane * 4, OFF_GATE + dl * 128, NSEND, NSEND, nwait++ & 15);
        const int depth = direct ? (tct >= 12 ? 2 : 1) : ((!fwd && tct >= 12) ? 1 : 4);
        char* dst = p.ws[(rank ^ (fwd ? 0u : dl)) & (NR - 1)];
        const auto rd = RSRC(dst, WS_NEED);
        const u32 yo = vy + (fwd ? OFF_LY + lt * 32768 : par_off + yslot);
        const u32 fo = direct ? OFF_GFLAG + gt * 128 + (c.lane >> 1) * 64 + (c.lane & 1) * 4 : (fwd ? OFF_LYF + lt * 4 : OFF_YFLAG + (rt * 16 + tct) * 128 + c.lane * 4);
#pragma unroll
        for (int m = 0; m < 4; ++m) {
          __builtin_amdgcn_raw_buffer_store_b128(y[m], rd, yo + m * 1024, 0, SYS);
          asm volatile("" ::: "memory");
          if (m == (depth == 1 ? 2 : 1) && !fwd && last_wave(c, tile) && c.lane < 4u) put_flag(dst + fo, c.call);
          if (m < 3) {
            if (depth == 1) WAIT_VM(0);
            else if (depth == 2) WAIT_VM(1);
          }
        }
        if (fwd) {
          WAIT_VM(0);
          if (last_wave(c, tile) && c.lane == 0) put_flag(dst + fo, c.call);
        }
      }
      if (++tile == 1 && ntiles == 2) {
        if (isG) { rt = 3 + grp + ((u >> 1) & 1); tct = 12 + (u >> 2); }
        else { tO = true; rt = 15; tct = u * 2 + (grp >> 2); }
        tinfo = TABLES.tile[(rt * 16 + tct) & 255];
      }
    }
  }

  const auto rout = RSRC(p.out, IN_BYTES);

  if (tO) {
    const u32 tj = rt - 12, yt = rfl(tj * 16 + tct);
    const u32 fo = OFF_YFLAG + yt * 128 + (c.lane & 3) * 4;
    const u32 yb = vy + par_off + YS_OFF + yt * 32768;
    const u32 yoff[4] = {yb, yb + 1024, yb + 2048, yb + 3072};
    bad = load_y<4>(c, rws, yoff, y, fo, fo, bad);
    const u32 vo = (tj * 128 + (c.wave >> 2) * 64 + (c.lane & 15)) * 8192 + (rank & 1) * 4096 + tct * 256 + (c.wave & 3) * 64 + (c.lane >> 4) * 16;
#pragma unroll
    for (int m = 0; m < 4; ++m) {
      u32x4 o;
      o.x = pack2(acc[m][0][0] + bf_lo(y[m].x), acc[m][0][1] + bf_hi(y[m].x));
      o.y = pack2(acc[m][0][2] + bf_lo(y[m].y), acc[m][0][3] + bf_hi(y[m].y));
      o.z = pack2(acc[m][1][0] + bf_lo(y[m].z), acc[m][1][1] + bf_hi(y[m].z));
      o.w = pack2(acc[m][1][2] + bf_lo(y[m].w), acc[m][1][3] + bf_hi(y[m].w));
      if (bad) o = (u32x4){NANS, NANS, NANS, NANS};
      __builtin_amdgcn_raw_buffer_store_b128(o, rout, vo + m * 16 * 8192, 0, SYS);
    }
  } else if (hasQ) {
    const u32 ga = qrt * 16 + qct, gb = ga + 64;
    const u32 fo = OFF_GFLAG + (c.lane & 2) * 32 + (c.lane & 1) * 4;
    const u32 oa = vy + par_off + YS_OFF + (64 + ga) * 32768, ob = oa + 64 * 32768;
    const u32 yoff[8] = {oa, oa + 1024, oa + 2048, oa + 3072, ob, ob + 1024, ob + 2048, ob + 3072};
    u32x4 z[8];
    bad = load_y<8>(c, rws, yoff, z, fo + ga * 128, fo + gb * 128, bad);
    const u32 vo = (qrt * 128 + (c.wave >> 2) * 64 + (c.lane & 15)) * 8192 + ((rank & 1) ^ 1) * 4096 + qct * 256 + (c.wave & 3) * 64 + (c.lane >> 4) * 16;
#pragma unroll
    for (int m = 0; m < 4; ++m) {
      const u32x4 a = z[m], b = z[4 + m];
      u32x4 o;
      o.x = pack2(qacc[m][0][0] + bf_lo(a.x) + bf_lo(b.x), qacc[m][0][1] + bf_hi(a.x) + bf_hi(b.x));
      o.y = pack2(qacc[m][0][2] + bf_lo(a.y) + bf_lo(b.y), qacc[m][0][3] + bf_hi(a.y) + bf_hi(b.y));
      o.z = pack2(qacc[m][1][0] + bf_lo(a.z) + bf_lo(b.z), qacc[m][1][1] + bf_hi(a.z) + bf_hi(b.z));
      o.w = pack2(qacc[m][1][2] + bf_lo(a.w) + bf_lo(b.w), qacc[m][1][3] + bf_hi(a.w) + bf_hi(b.w));
      if (bad) o = (u32x4){NANS, NANS, NANS, NANS};
      __builtin_amdgcn_raw_buffer_store_b128(o, rout, vo + m * 16 * 8192, 0, SYS);
    }
  }
}

}

extern "C" void kernel_launch(int rank, int nranks, ncclComm_t comm, hipStream_t stream,
                              void* const* d_in, const int* in_sizes, int n_in,
                              void* d_out, int out_size,
                              const int* in_dtypes, int out_dtype,
                              void* const* d_out_all,
                              void* const* d_ws_all, size_t ws_bytes) {
  (void)comm; (void)in_dtypes; (void)out_dtype; (void)d_out_all;
  if (nranks != NR || (unsigned)rank >= (unsigned)NR || n_in != 2 || ws_bytes < WS_NEED ||
      in_sizes[0] != 4096 * 512 || in_sizes[1] != 512 * 4096 || out_size != 512 * 4096) {
    fprintf(stderr, "gemm_rs: unexpected problem shape\n");
    return;
  }
  Params p;
  p.x = (const char*)d_in[0];
  p.w = (const char*)d_in[1];
  p.out = (char*)d_out;
  p.wsl = (char*)d_ws_all[rank];
  p.rank = 0;
  for (int i = 0; i < NR; ++i) {
    const int r = (L2P >> (4 * i)) & 7;
    p.ws[i] = (char*)d_ws_all[r];
    if (r == rank) p.rank = i;
  }
  hipLaunchKernelGGL(gemm_rs, dim3(NWG), dim3(NT), 0, stream, p);
}
```
